```python
import numpy as np

kernel_name = "gpu_gemm_rs_m4096_k4096_n4096_e4m3e5m2_dequant"


ABSMAX_RTOL = 0.05

COMPUTE_DTYPE = "float32"
EPILOGUE = "dequant"
IN_QUANT = None
OUT_QUANT = None
IN_FP8 = "e4m3e5m2"
IN_DTYPES = {"x": "e4m3", "w_mat": "e5m2",
             "scale_x": "float32", "scale_w": "float32"}
WORLD_SIZE = 8

def _snap_fp8(x, mant_bits, max_abs, min_normal):
    a = np.abs(x.astype(np.float64))
    _m, _e = np.frexp(a)
    q = np.where(a >= min_normal, np.ldexp(1.0, _e - 1 - mant_bits),
                 min_normal * 2.0 ** -mant_bits)
    snapped = np.minimum(np.round(a / q) * q, max_abs)
    return (np.sign(x) * snapped).astype(np.float32)

def setup_inputs():

    x = _snap_fp8(np.random.randn(4096, 4096).astype(np.float32), 3, 448.0, 2.0 ** -6)
    w_mat = _snap_fp8(np.random.randn(4096, 4096).astype(np.float32), 2, 57344.0, 2.0 ** -14)
    scale_x = (0.02 * (1.0 + np.random.rand(1))).astype(np.float32)
    scale_w = (0.02 * (1.0 + np.random.rand(1))).astype(np.float32)
    return {"x": x, "w_mat": w_mat, "scale_x": scale_x, "scale_w": scale_w}

def reference(x, w_mat, scale_x, scale_w):
    acc = (x @ w_mat).astype(np.float32)
    y = (acc * (scale_x * scale_w)).astype(np.float32)
    return y
```

```cpp
#include <hip/hip_runtime.h>
#include <rccl/rccl.h>
#include <cstdio>

#define HIP_CHECK(expr) do { hipError_t _e = (expr); if (_e != hipSuccess) { \
    fprintf(stderr, "HIP error %s at %s:%d\n", hipGetErrorString(_e), __FILE__, __LINE__); \
    return; } } while (0)

namespace {

typedef unsigned u32;
typedef unsigned long long u64;
typedef u32 u32x2 __attribute__((ext_vector_type(2)));
typedef u32 u32x4 __attribute__((ext_vector_type(4)));
typedef int i32x8 __attribute__((ext_vector_type(8)));
typedef float f32x2 __attribute__((ext_vector_type(2)));
typedef float f32x16 __attribute__((ext_vector_type(16)));

constexpr int NR = 8;
constexpr int NN = 4096;
constexpr int KS = 512;
constexpr int MS = 512;
constexpr int NWG = 256;
constexpr int NWORK = 4;
constexpr int NPUSH = 2;
constexpr int NTHREADS = 64 * (NWORK + NPUSH);

constexpr size_t WS_W   = 0;
constexpr size_t W_PAR  = (size_t)NR * 8 * 64 * 4096;
constexpr size_t WS_X   = 2 * W_PAR;
constexpr size_t X_PAR  = (size_t)NR * 4 * 4 * 8 * 2048;
constexpr size_t WS_FW  = WS_X + 2 * X_PAR;
constexpr size_t FW_PAR = 64 * 64 * 8;
constexpr size_t WS_FX  = WS_FW + 2 * FW_PAR;
constexpr size_t FX_PAR = 4 * 8 * 16 * 4;
constexpr size_t WS_CNT = WS_FX + 2 * FX_PAR;
constexpr size_t WS_TMO = WS_CNT + NWG * 4;
constexpr size_t WS_END = WS_TMO + 128;

constexpr u64 DEADLINE_TICKS = 30000000ull;

constexpr int L_B    = 0;
constexpr int L_CMD  = 16384;
constexpr int L_BAR  = L_CMD + 64;
constexpr int L_CALL = L_BAR + 64;
constexpr int L_BYTES = 84 * 1024;

struct Params {
    const unsigned char* x;
    const unsigned char* w;
    const float* sx;
    const float* sw;
    float* out;
    unsigned char* ws[NR];
    int rank;
    int pad;
};

__device__ __forceinline__ void st16_sys(void* p, u32x4 v) {
    asm volatile("global_store_dwordx4 %0, %1, off sc0 sc1\n\ts_nop 1" :: "v"(p), "v"(v) : "memory");
}
__device__ __forceinline__ __amdgpu_buffer_rsrc_t make_rsrc(const void* base, u32 bytes) {
    return __builtin_amdgcn_make_buffer_rsrc(const_cast<void*>(base), 0, bytes, 0x00020000);
}
__device__ __forceinline__ u32 rfl(u32 v) { return __builtin_amdgcn_readfirstlane(v); }

__device__ __forceinline__ void worker_barrier(unsigned char* lds, u32& target, int lane) {
    target += NWORK;
    u32* ctr = (u32*)(lds + L_BAR);
    asm volatile("s_waitcnt lgkmcnt(0)" ::: "memory");
    if (lane == 0) __hip_atomic_fetch_add(ctr, 1u, __ATOMIC_RELAXED, __HIP_MEMORY_SCOPE_WORKGROUP);
    while (rfl(__hip_atomic_load(ctr, __ATOMIC_RELAXED, __HIP_MEMORY_SCOPE_WORKGROUP)) < target) { }
    asm volatile("" ::: "memory");
}

__global__ void __launch_bounds__(NTHREADS, 2) gemm_ag_fused(Params p) {
    __shared__ __attribute__((aligned(16))) unsigned char lds[L_BYTES];
    const int tid = threadIdx.x;
    const int lane = tid & 63;
    const int wave = rfl(tid >> 6);
    const int b = blockIdx.x;
    const int xcd = b & 7, a = b >> 3;
    unsigned char* my = p.ws[p.rank];

    if (tid == 0) {
        u32* cnt = (u32*)(my + WS_CNT) + b;
        const u32 c = __hip_atomic_load(cnt, __ATOMIC_RELAXED, __HIP_MEMORY_SCOPE_AGENT) + 1u;
        __hip_atomic_store(cnt, c, __ATOMIC_RELAXED, __HIP_MEMORY_SCOPE_AGENT);
        *(u32*)(lds + L_CALL) = c;
        *(u32*)(lds + L_BAR) = 0u;
    }

    if (wave >= NWORK) {
        const int dd = a & 7;
        const int jb = (a >> 3) * 16 + xcd * 2 + (wave - NWORK);
        const int dst = (p.rank + dd) & (NR - 1);
        unsigned char* rws = p.ws[dst];

        const int xi = jb >> 4, xw = (jb >> 2) & 3, xk2 = jb & 3;
        const unsigned char* xs = p.x + (size_t)(dst * MS + xi * 128 + xw * 32 + (lane & 31)) * KS + xk2 * 128 + (lane >> 5) * 32;
        u32x4 xv[4];
#pragma unroll
        for (int j = 0; j < 4; ++j) xv[j] = *(const u32x4*)(xs + (j >> 1) * 64 + (j & 1) * 16);

        const __amdgpu_buffer_rsrc_t wr = make_rsrc(p.w, KS * NN);
        const u32 woff = (u32)((lane >> 4) * 16 * NN + jb * 64 + (lane & 15) * 4);
        u32 r[16];
#pragma unroll
        for (int t = 0; t < 16; ++t) r[t] = __builtin_amdgcn_raw_buffer_load_b32(wr, woff, t * NN, 0);

        __syncthreads();
        const u32 call = rfl(*(const u32*)(lds + L_CALL));
        const u32 par = call & 1u;

        {
            unsigned char* xd = rws + WS_X + par * X_PAR + (size_t)p.rank * (X_PAR / NR) + (size_t)((xi * 4 + xw) * 8 + xk2 * 2) * 2048 + lane * 16;
#pragma unroll
            for (int j = 0; j < 4; ++j) st16_sys(xd + j * 1024, xv[j]);
        }
        unsigned char* wd = rws + WS_W + par * W_PAR + (size_t)p.rank * (W_PAR / NR) + (size_t)jb * 4096 + lane * 16;
        u64* fw = (u64*)(rws + WS_FW + par * FW_PAR) + jb * 64 + p.rank * 8;
        u32* fx = (u32*)(rws + WS_FX + par * FX_PAR) + (xi * 8 + p.rank) * 16 + (jb & 15);

#pragma unroll 1
        for (int kc = 0; kc < 8; ++kc) {
            u32 T[4][4];
#pragma unroll
            for (int t4 = 0; t4 < 4; ++t4) {
                const u32 r0 = r[4 * t4], r1 = r[4 * t4 + 1], r2 = r[4 * t4 + 2], r3 = r[4 * t4 + 3];
                const u32 aa = __builtin_amdgcn_perm(r1, r0, 0x05010400u), bb = __builtin_amdgcn_perm(r1, r0, 0x07030602u);
                const u32 cc = __builtin_amdgcn_perm(r3, r2, 0x05010400u), dd2 = __builtin_amdgcn_perm(r3, r2, 0x07030602u);
                T[0][t4] = __builtin_amdgcn_perm(cc, aa, 0x05040100u);
                T[1][t4] = __builtin_amdgcn_perm(cc, aa, 0x07060302u);
                T[2][t4] = __builtin_amdgcn_perm(dd2, bb, 0x05040100u);
                T[3][t4] = __builtin_amdgcn_perm(dd2, bb, 0x07060302u);
            }
            if (kc == 0) {
                asm volatile("s_waitcnt vmcnt(0)" ::: "memory");
                if (lane == 0) __hip_atomic_store(fx, call, __ATOMIC_RELAXED, __HIP_MEMORY_SCOPE_SYSTEM);
            }
#pragma unroll
            for (int q = 0; q < 4; ++q) {
                u32x4 v = {T[q][0], T[q][1], T[q][2], T[q][3]};
                st16_sys(wd + (size_t)kc * (64 * 4096) + q * 1024, v);
            }
            if (kc < 7) {
#pragma unroll
                for (int t = 0; t < 16; ++t) r[t] = __builtin_amdgcn_raw_buffer_load_b32(wr, woff + (kc + 1) * 64 * NN, t * NN, 0);
            }
            asm volatile("s_waitcnt vmcnt(0)" ::: "memory");
            if (lane == 0) __hip_atomic_store(fw + kc, (u64)call, __ATOMIC_RELAXED, __HIP_MEMORY_SCOPE_SYSTEM);
        }
        return;
    }

    __syncthreads();
    const u32 call = rfl(*(const u32*)(lds + L_CALL));
    const u32 par = call & 1u;
    const int wv = wave;
    const int jb = xcd * 8 + (a & 7), ti = a >> 3;
    u32* tmo = (u32*)(my + WS_TMO);

    const __amdgpu_buffer_rsrc_t rw = make_rsrc(my + WS_W + par * W_PAR, (u32)W_PAR);
    const __amdgpu_buffer_rsrc_t rx = make_rsrc(my + WS_X + par * X_PAR, (u32)X_PAR);
    const u64* fwp = (const u64*)(my + WS_FW + par * FW_PAR) + jb * 64 + lane;
    const u64* fxp = (const u64*)(my + WS_FX + par * FX_PAR) + ti * 64 + lane;

    f32x16 acc0 = {}, acc1 = {};
    u64 done = 0, ready = 0;
    u32 bar = 0;
    bool dead = false;
    const u32 b_rd = (u32)(((lane >> 4) & 1) * 2048 + (lane >> 5) * 512 + (lane & 15) * 16);
    const u32 a_off = (u32)((ti * 4 + wv) * 8 * 2048 + lane * 16);
    const u32 b_off = (u32)(jb * 4096 + lane * 16);

    while (done != ~0ull) {
        if (wv == 0) {
            u64 avail = ready & ~done;
            if (avail == 0) {
                const u64 t0 = __builtin_amdgcn_s_memrealtime();
                while (true) {
                    const u64 f = __hip_atomic_load(fwp, __ATOMIC_RELAXED, __HIP_MEMORY_SCOPE_SYSTEM);
                    const u64 g = __hip_atomic_load(fxp, __ATOMIC_RELAXED, __HIP_MEMORY_SCOPE_SYSTEM);
                    const u64 mw = __ballot((u32)f >= call);
                    u64 mx = __ballot((u32)g >= call && (u32)(g >> 32) >= call);
                    mx &= (mx >> 4) & 0x0f0f0f0f0f0f0f0full; mx &= mx >> 2; mx &= mx >> 1; mx &= 0x0101010101010101ull;
                    mx *= 0xffull;
                    ready = mw & mx;
                    avail = ready & ~done;
                    if (avail) break;
                    if (__hip_atomic_load(tmo, __ATOMIC_RELAXED, __HIP_MEMORY_SCOPE_AGENT) == call
                        || __builtin_amdgcn_s_memrealtime() - t0 > DEADLINE_TICKS) {
                        __hip_atomic_store(tmo, call, __ATOMIC_RELAXED, __HIP_MEMORY_SCOPE_AGENT);
                        dead = true;
                        break;
                    }
                    __builtin_amdgcn_s_sleep(1);
                }
            }
            u32 cnt = 0, u[4];
            if (dead) {
                cnt = 0x80000000u;
                u[0] = u[1] = u[2] = u[3] = 0;
            } else {
                u[0] = (u32)__builtin_ctzll(avail); avail &= avail - 1; cnt = 1;
#pragma unroll
                for (int j = 1; j < 4; ++j) {
                    if (avail) { u[j] = (u32)__builtin_ctzll(avail); avail &= avail - 1; ++cnt; }
                    else u[j] = u[0];
                }
            }
            if (lane == 0) {
                u32x4 cu = {u[0], u[1], u[2], u[3]};
                *(u32x4*)(lds + L_CMD) = cu;
                *(u32*)(lds + L_CMD + 16) = cnt;
            }
        }
        worker_barrier(lds, bar, lane);
        const u32x4 cu = *(const u32x4*)(lds + L_CMD);
        const u32 cnt = rfl(*(const u32*)(lds + L_CMD + 16));
        if (cnt == 0x80000000u) break;
        u32 u[4] = {rfl(cu.x), rfl(cu.y), rfl(cu.z), rfl(cu.w)};

        const u32 ub = wv == 0 ? u[0] : wv == 1 ? u[1] : wv == 2 ? u[2] : u[3];
        u32x4 bv[4];
#pragma unroll
        for (int q = 0; q < 4; ++q)
            bv[q] = __builtin_amdgcn_raw_buffer_load_b128(rw, b_off + q * 1024, ub * (64 * 4096), 17);
        u32x4 av[4][2];
#pragma unroll
        for (int j = 0; j < 4; ++j) {
            const u32 so = (u[j] >> 3) * (u32)(X_PAR / NR) + (u[j] & 7) * 2048;
#pragma unroll
            for (int s = 0; s < 2; ++s) av[j][s] = __builtin_amdgcn_raw_buffer_load_b128(rx, a_off + s * 1024, so, 17);
        }
#pragma unroll
        for (int q = 0; q < 4; ++q) *(u32x4*)(lds + L_B + wv * 4096 + q * 1024 + lane * 16) = bv[q];
        worker_barrier(lds, bar, lane);
#pragma unroll
        for (int j = 0; j < 4; ++j) {
            if ((u32)j < cnt) {
                const unsigned char* bb = lds + L_B + j * 4096 + b_rd;
                const u32x4 b00 = *(const u32x4*)(bb), b01 = *(const u32x4*)(bb + 256);
                const u32x4 b10 = *(const u32x4*)(bb + 1024), b11 = *(const u32x4*)(bb + 1280);
                const i32x8 af = {(int)av[j][0].x, (int)av[j][0].y, (int)av[j][0].z, (int)av[j][0].w, (int)av[j][1].x, (int)av[j][1].y, (int)av[j][1].z, (int)av[j][1].w};
                const i32x8 bf0 = {(int)b00.x, (int)b00.y, (int)b00.z, (int)b00.w, (int)b01.x, (int)b01.y, (int)b01.z, (int)b01.w};
                const i32x8 bf1 = {(int)b10.x, (int)b10.y, (int)b10.z, (int)b10.w, (int)b11.x, (int)b11.y, (int)b11.z, (int)b11.w};
                acc0 = __builtin_amdgcn_mfma_scale_f32_32x32x64_f8f6f4(af, bf0, acc0, 0, 1, 0, 0, 0, 0);
                acc1 = __builtin_amdgcn_mfma_scale_f32_32x32x64_f8f6f4(af, bf1, acc1, 0, 1, 0, 0, 0, 0);
                done |= 1ull << u[j];
            }
        }
    }

    const bool bad = __hip_atomic_load(tmo, __ATOMIC_RELAXED, __HIP_MEMORY_SCOPE_AGENT) == call;
    const float s = bad ? __builtin_nanf("") : p.sx[0] * p.sw[0];
    float* o = p.out + (size_t)(ti * 128 + wv * 32 + (lane >> 5) * 4) * NN + jb * 64 + (lane & 15) * 4 + ((lane >> 4) & 1) * 2;
#pragma unroll
    for (int r = 0; r < 16; ++r) {
        f32x2 v = {acc0[r] * s, acc1[r] * s};
        *(f32x2*)(o + (size_t)((r & 3) + 8 * (r >> 2)) * NN) = v;
    }
}

}

extern "C" void kernel_launch(int rank, int nranks, ncclComm_t comm, hipStream_t stream,
                              void* const* d_in, const int* in_sizes, int n_in,
                              void* d_out, int out_size,
                              const int* in_dtypes, int out_dtype,
                              void* const* d_out_all,
                              void* const* d_ws_all, size_t ws_bytes) {
    (void)comm; (void)in_sizes; (void)n_in; (void)out_size; (void)in_dtypes; (void)out_dtype; (void)d_out_all;
    if (nranks != NR || ws_bytes < WS_END || (unsigned)rank >= (unsigned)NR) {
        fprintf(stderr, "unexpected configuration\n");
        return;
    }
    Params p;
    p.x = (const unsigned char*)d_in[0];
    p.w = (const unsigned char*)d_in[1];
    p.sx = (const float*)d_in[2];
    p.sw = (const float*)d_in[3];
    p.out = (float*)d_out;
    for (int i = 0; i < NR; ++i) p.ws[i] = (unsigned char*)d_ws_all[i];
    p.rank = rank;
    p.pad = 0;
    hipLaunchKernelGGL(gemm_ag_fused, dim3(NWG), dim3(NTHREADS), 0, stream, p);
    HIP_CHECK(hipGetLastError());
}
```

```cpp
#include <hip/hip_runtime.h>
#include <rccl/rccl.h>
#include <cstdio>

#define HIP_CHECK(expr) do { hipError_t _e = (expr); if (_e != hipSuccess) { \
    fprintf(stderr, "HIP error %s at %s:%d\n", hipGetErrorString(_e), __FILE__, __LINE__); \
    return; } } while (0)

namespace {

typedef unsigned u32;
typedef unsigned long long u64;
typedef u32 u32x2 __attribute__((ext_vector_type(2)));
typedef u32 u32x4 __attribute__((ext_vector_type(4)));
typedef int i32x8 __attribute__((ext_vector_type(8)));
typedef float f32x16 __attribute__((ext_vector_type(16)));

#define ROT 0
#define POLLSLEEP 2
#define XSLEEP 2
#define NSEND_ 6
#define NSTAGE_ 7
#define PACK 1

constexpr int NR = 8;
constexpr int NN = 4096;
constexpr int KS = 512;
constexpr int MS = 512;
constexpr int NWG = 256;
constexpr int NCWG = 128;
constexpr int NWORK = 8;
constexpr int NSTAGE = NSTAGE_;
constexpr int W_STG  = NWORK;
constexpr int W_POLL = W_STG + NSTAGE;
constexpr int NWAVES = W_POLL + 1;
constexpr int NTHREADS = 64 * NWAVES;
constexpr int NSEND = NSEND_;
#define MAPPING 0
constexpr int WPD = MAPPING == 0 ? (NWG - NCWG) * NSEND / (NR - 1) : MAPPING == 1 ? 18 * NSEND : 16 * NSEND;
constexpr u32 NXB = 64;
constexpr u32 NBLK = NXB + 512;

constexpr size_t WS_W   = 0;
constexpr size_t W_PAR  = (size_t)NR * 8 * 64 * 4096;
constexpr size_t WS_X   = 2 * W_PAR;
constexpr size_t X_PAR  = (size_t)NR * 16 * 8 * 2048;
constexpr size_t WS_FW  = WS_X + 2 * X_PAR;
constexpr size_t FW_PAR = 64 * 64 * 8;
constexpr size_t WS_FX  = WS_FW + 2 * FW_PAR;
constexpr size_t FX_PAR = 2 * 8 * 32 * 4;
constexpr size_t WS_Q   = WS_FX + 2 * FX_PAR;
constexpr size_t WS_CNT = WS_Q + NR * 128;
constexpr size_t WS_TMO = WS_CNT + NWG * 4;
constexpr size_t WS_END = WS_TMO + 128;
#define STAMP(slot) do {} while (0)
#define STAMPV(slot, v) do {} while (0)
#define STAMPA(area, idx, v) do {} while (0)

constexpr u64 DEADLINE_TICKS = 30000000ull;

constexpr u32 H_RAW = 1u << 17;
constexpr u32 MAXFIX = 512;

constexpr int NSLOT  = 32;
constexpr int L_B    = 0;
constexpr int L_ORD  = NSLOT * 4096;
constexpr int L_STG  = L_ORD + 256;
constexpr int L_TAIL = L_STG + 256;
constexpr int L_CALL = L_TAIL + 16;
constexpr int L_CONS = L_CALL + 16;
constexpr int L_BYTES = L_CONS + 32;
constexpr u32 DEAD = 0x80000000u;

struct Params {
    const unsigned char* x;
    const unsigned char* w;
    const float* sx;
    const float* sw;
    float* out;
    unsigned char* ws[NR];
    int rank;
    int pad;
};

__device__ __forceinline__ void st16_sys(void* p, u32x4 v) {
    asm volatile("global_store_dwordx4 %0, %1, off sc0 sc1\n\ts_nop 1" :: "v"(p), "v"(v) : "memory");
}
__device__ __forceinline__ __amdgpu_buffer_rsrc_t make_rsrc(const void* base, u32 bytes) {
    return __builtin_amdgcn_make_buffer_rsrc(const_cast<void*>(base), 0, bytes, 0x00020000);
}
__device__ __forceinline__ u32 rfl(u32 v) { return __builtin_amdgcn_readfirstlane(v); }
__device__ __forceinline__ u32 lds_ld(const unsigned char* lds, int off) {
    return __hip_atomic_load((const u32*)(lds + off), __ATOMIC_RELAXED, __HIP_MEMORY_SCOPE_WORKGROUP);
}
__device__ __forceinline__ void lds_st(unsigned char* lds, int off, u32 v) {
    __hip_atomic_store((u32*)(lds + off), v, __ATOMIC_RELAXED, __HIP_MEMORY_SCOPE_WORKGROUP);
}
__device__ __forceinline__ u32 lanes_below(u64 m) {
    return __builtin_amdgcn_mbcnt_hi((u32)(m >> 32), __builtin_amdgcn_mbcnt_lo((u32)m, 0u));
}
__device__ __forceinline__ u32 wave_sum(u32 v) {
    v += __builtin_amdgcn_update_dpp(0u, v, 0x111, 0xf, 0xf, false);
    v += __builtin_amdgcn_update_dpp(0u, v, 0x112, 0xf, 0xf, false);
    v += __builtin_amdgcn_update_dpp(0u, v, 0x114, 0xf, 0xf, false);
    v += __builtin_amdgcn_update_dpp(0u, v, 0x118, 0xf, 0xf, false);
    return __builtin_amdgcn_readlane(v, 15) + __builtin_amdgcn_readlane(v, 31) + __builtin_amdgcn_readlane(v, 47) + __builtin_amdgcn_readlane(v, 63);
}
#define LGKM0() asm volatile("s_waitcnt lgkmcnt(0)" ::: "memory")
#define VM0() asm volatile("s_waitcnt vmcnt(0)" ::: "memory")
#define CFENCE() asm volatile("" ::: "memory")

__device__ __forceinline__ bool timed_out(u32* tmo, u32 call, u64 t0) {
    if (__hip_atomic_load(tmo, __ATOMIC_RELAXED, __HIP_MEMORY_SCOPE_AGENT) == call
        || __builtin_amdgcn_s_memrealtime() - t0 > DEADLINE_TICKS) {
        __hip_atomic_store(tmo, call, __ATOMIC_RELAXED, __HIP_MEMORY_SCOPE_AGENT);
        return true;
    }
    return false;
}

__global__ void __launch_bounds__(NTHREADS, (NTHREADS + 255) / 256) gemm_ag_fused(Params p) {
    __shared__ __attribute__((aligned(16))) unsigned char lds[L_BYTES];
    const int tid = threadIdx.x;
    const int lane = tid & 63;
    const int wave = rfl(tid >> 6);
    const int b = blockIdx.x;
    const int xcd = b & 7, a = b >> 3;
    unsigned char* my = p.ws[p.rank];
    u32* tmo = (u32*)(my + WS_TMO);

    if (wave == 0) STAMP(0);
    if (tid == 0) {
        u32* cnt = (u32*)(my + WS_CNT) + b;
        const u32 c = __hip_atomic_load(cnt, __ATOMIC_RELAXED, __HIP_MEMORY_SCOPE_AGENT) + 1u;
        __hip_atomic_store(cnt, c, __ATOMIC_RELAXED, __HIP_MEMORY_SCOPE_AGENT);
        *(u32*)(lds + L_CALL) = c;
    }
    if (wave == 1) {
        *(u32*)(lds + L_STG + lane * 4) = 0u;
        if (lane < 4) *(u32*)(lds + L_TAIL + lane * 4) = 0u;
        if (lane < 8) *(u32*)(lds + L_CONS + lane * 4) = 0u;
    }
    const u32 xlane = (u32)((lane & 31) * KS + (lane >> 5) * 32);

    if (a >= NCWG / 8) {
        const int g = ((a - NCWG / 8) * 8 + xcd) * NSEND + wave;
        if (wave >= NSEND || g >= WPD * (NR - 1)) { __syncthreads(); return; }
        const int dd = 1 + g % (NR - 1);
        const u32 wi = (u32)(g / (NR - 1));
        const int dst = (p.rank + dd) & (NR - 1);
        unsigned char* rws = p.ws[dst];
        u32* queue = (u32*)(my + WS_Q + dd * 128);
        const __amdgpu_buffer_rsrc_t xr = make_rsrc(p.x + (size_t)dst * MS * KS, MS * KS);
        u32x4 r[4];
        auto load_x = [&](u32 t) {
            const u32 so = (t >> 2) * (32 * KS) + (t & 3) * 128;
#pragma unroll
            for (int j = 0; j < 4; ++j) r[j] = __builtin_amdgcn_raw_buffer_load_b128(xr, xlane + (j >> 1) * 64 + (j & 1) * 16, so, 0);
        };
        u32 t0 = wi, t1 = wi + WPD, t2 = wi + 2 * WPD;
        if (t0 < NXB) load_x(t0);
        __syncthreads();
        const u32 call = rfl(*(const u32*)(lds + L_CALL));
        const u32 par = call & 1u;
        const u32 qbase = (call - 1u) * NBLK - 3u * WPD;
        const __amdgpu_buffer_rsrc_t wl = make_rsrc(my + WS_W + par * W_PAR + (size_t)p.rank * (W_PAR / NR), (u32)(W_PAR / NR));
        const u64* fl = (const u64*)(my + WS_FW + par * FW_PAR) + p.rank * 8;
        unsigned char* xd = rws + WS_X + par * X_PAR + (size_t)p.rank * (X_PAR / NR) + lane * 16;
        unsigned char* wd = rws + WS_W + par * W_PAR + (size_t)p.rank * (W_PAR / NR) + lane * 16;
        u64* fw = (u64*)(rws + WS_FW + par * FW_PAR) + p.rank * 8;
        u32* fx = (u32*)(rws + WS_FX + par * FX_PAR) + p.rank * 32;
        auto kc_of = [&](u32 t) { return (t - NXB) >> 6; };
        auto jb_of = [&](u32 t) { return (t - NXB + ROT * p.rank) & 63u; };
        auto load_w = [&](u32 t) {
            const u32 so = (kc_of(t) * 64 + jb_of(t)) * 4096;
#pragma unroll
            for (int j = 0; j < 4; ++j) r[j] = __builtin_amdgcn_raw_buffer_load_b128(wl, lane * 16 + j * 1024, so, 17);
        };
        auto wait_hdr = [&](u32 t, u32& h) -> bool {
            const u64* f = fl + jb_of(t) * 64 + kc_of(t);
            u64 v = __hip_atomic_load(f, __ATOMIC_RELAXED, __HIP_MEMORY_SCOPE_SYSTEM);
            if ((u32)v < call) {
                const u64 ts = __builtin_amdgcn_s_memrealtime();
                do {
                    if (timed_out(tmo, call, ts)) return false;
                    __builtin_amdgcn_s_sleep(2);
                    v = __hip_atomic_load(f, __ATOMIC_RELAXED, __HIP_MEMORY_SCOPE_SYSTEM);
                } while ((u32)v < call);
            }
            CFENCE();
            h = rfl((u32)(v >> 32));
            return true;
        };
        u32 h0 = H_RAW, h1 = H_RAW;
        u64 f1 = 0;
        if (t0 >= NXB) { if (!wait_hdr(t0, h0)) return; load_w(t0); }
        int nb = 0;
#pragma unroll 1
        while (t0 < NBLK) {
            u32 tk = 0;
            if (lane == 0) tk = __hip_atomic_fetch_add(queue, 1u, __ATOMIC_RELAXED, __HIP_MEMORY_SCOPE_AGENT);
            unsigned char* d = t0 < NXB ? xd + (size_t)t0 * 4096 : wd + (size_t)(kc_of(t0) * 64 + jb_of(t0)) * 4096;
            const u32 nl = (h0 & H_RAW) ? 64u : ((h0 & 1023u) + 7u) >> 3;
#pragma unroll
            for (int j = 0; j < 3; ++j) st16_sys(d + j * 1024, r[j]);
            if ((u32)lane < nl) st16_sys(d + 3 * 1024, r[3]);
            h1 = H_RAW;
            if (t1 < NXB) load_x(t1);
            else if (t1 < NBLK) {
                if (rfl((u32)f1) >= call) h1 = rfl((u32)(f1 >> 32));
                else if (!wait_hdr(t1, h1)) return;
                load_w(t1);
            }
            u64 f2 = 0;
            const bool w2 = t2 >= NXB && t2 < NBLK;
            if (w2) f2 = __hip_atomic_load(fl + jb_of(t2) * 64 + kc_of(t2), __ATOMIC_RELAXED, __HIP_MEMORY_SCOPE_SYSTEM);
            VM0();
            if (lane == 0) {
                if (t0 < NXB) __hip_atomic_store(fx + (t0 >> 5) * 256 + (t0 & 31), call, __ATOMIC_RELAXED, __HIP_MEMORY_SCOPE_SYSTEM);
                else __hip_atomic_store(fw + jb_of(t0) * 64 + kc_of(t0), (u64)call | ((u64)h0 << 32), __ATOMIC_RELAXED, __HIP_MEMORY_SCOPE_SYSTEM);
            }
            t0 = t1; h0 = h1;
            t1 = t2; f1 = f2;
            t2 = rfl(tk) - qbase;
        }
        (void)nb;
        return;
    }

    const int jb = xcd * 8 + (a & 7), ti = a >> 3;

    if (wave >= W_STG && wave < W_POLL) {
        const int sg = wave - W_STG;
        {
            const __amdgpu_buffer_rsrc_t wr = make_rsrc(p.w, KS * NN);
            const u32 wlane = (u32)((lane >> 4) * 16 * NN + (lane & 15) * 4);
            u32 id = (u32)((a * 8 + xcd) * NSTAGE + sg);
            u32 r[16];
            auto load_w = [&](u32 t) {
                const u32 so = (t >> 6) * 64 * NN + ((t + ROT * p.rank) & 63u) * 64;
#pragma unroll
                for (int j = 0; j < 16; ++j) r[j] = __builtin_amdgcn_raw_buffer_load_b32(wr, wlane + j * NN, so, 0);
            };
            if (id < 512) load_w(id);
            __syncthreads();
            const u32 call = rfl(*(const u32*)(lds + L_CALL));
            const u32 par = call & 1u;
            unsigned char* scratch = lds + L_B + sg * 4096;
            for (; id < 512; id += NCWG * NSTAGE) {
                const u32 kc = id >> 6, pj = (id + ROT * p.rank) & 63u;
                u32 T[4][4];
#pragma unroll
                for (int t4 = 0; t4 < 4; ++t4) {
                    const u32 r0 = r[4 * t4], r1 = r[4 * t4 + 1], r2 = r[4 * t4 + 2], r3 = r[4 * t4 + 3];
                    const u32 aa = __builtin_amdgcn_perm(r1, r0, 0x05010400u), bb = __builtin_amdgcn_perm(r1, r0, 0x07030602u);
                    const u32 cc = __builtin_amdgcn_perm(r3, r2, 0x05010400u), ee = __builtin_amdgcn_perm(r3, r2, 0x07030602u);
                    T[0][t4] = __builtin_amdgcn_perm(cc, aa, 0x05040100u);
                    T[1][t4] = __builtin_amdgcn_perm(cc, aa, 0x07060302u);
                    T[2][t4] = __builtin_amdgcn_perm(ee, bb, 0x05040100u);
                    T[3][t4] = __builtin_amdgcn_perm(ee, bb, 0x07060302u);
                }
                if (id + NCWG * NSTAGE < 512) load_w(id + NCWG * NSTAGE);
                u32 hdr = H_RAW;
                u32x4 o[4];
                u32 sum = 0;
#pragma unroll
                for (int q = 0; q < 4; ++q)
#pragma unroll
                    for (int t4 = 0; t4 < 4; ++t4) sum = __builtin_amdgcn_sad_u8(T[q][t4] & 0x7f7f7f7fu, 0u, sum);
                const int bs = ((int)wave_sum(sum) - 82944) >> 12;
                const u32 base = (u32)min(max(bs, 0), 96);
                const u32 basex = base * 0x01010101u;
                u32 P[4][3];
                u32 total = 0;
#pragma unroll
                for (int q = 0; q < 4; ++q) {
                    u32 D[4];
#pragma unroll
                    for (int t4 = 0; t4 < 4; ++t4) {
                        const u32 d = T[q][t4];
                        const u32 code = ((d | 0x80808080u) - basex) & 0x1f1f1f1fu;
                        D[t4] = (d & 0x80808080u) | code;
                        u32 fix = (d ^ (code + basex)) & 0x60606060u;
                        u64 m = __ballot(fix != 0u);
                        while (m) {
                            if (fix != 0u) {
                                const u32 sh = (u32)__builtin_ctz(fix) & 24u;
                                const u32 idx = total + lanes_below(m);
                                const u32 ent = (u32)(q * 1024 + t4 * 4) + (u32)lane * 16u + (sh >> 3) + (((fix >> (sh + 5u)) & 3u) << 12);
                                if (idx < MAXFIX) *(unsigned short*)(scratch + idx * 2) = (unsigned short)ent;
                                fix &= ~(0x60u << sh);
                            }
                            total += (u32)__builtin_popcountll(m);
                            m = __ballot(fix != 0u);
                        }
                    }
                    P[q][0] = D[0] | ((D[3] & 0x03030303u) << 5);
                    P[q][1] = D[1] | ((D[3] & 0x0c0c0c0cu) << 3);
                    P[q][2] = D[2] | ((D[3] & 0x10101010u) << 1) | ((D[3] & 0x80808080u) >> 1);
                }
                if (total <= MAXFIX) {
                    hdr = total | (base << 10);
                    o[0] = (u32x4){P[0][0], P[0][1], P[0][2], P[1][0]};
                    o[1] = (u32x4){P[1][1], P[1][2], P[2][0], P[2][1]};
                    o[2] = (u32x4){P[2][2], P[3][0], P[3][1], P[3][2]};
                    o[3] = *(const u32x4*)(scratch + lane * 16);
                } else
                {
#pragma unroll
                    for (int q = 0; q < 4; ++q) o[q] = (u32x4){T[q][0], T[q][1], T[q][2], T[q][3]};
                }
                unsigned char* d = my + WS_W + par * W_PAR + (size_t)p.rank * (W_PAR / NR) + (size_t)(kc * 64 + pj) * 4096 + lane * 16;
#pragma unroll
                for (int j = 0; j < 4; ++j) st16_sys(d + j * 1024, o[j]);
                VM0();
                if (lane == 0) __hip_atomic_store((u64*)(my + WS_FW + par * FW_PAR) + pj * 64 + p.rank * 8 + kc, (u64)call | ((u64)hdr << 32), __ATOMIC_RELAXED, __HIP_MEMORY_SCOPE_SYSTEM);
            }
            const __amdgpu_buffer_rsrc_t xr = make_rsrc(p.x + (size_t)p.rank * MS * KS, MS * KS);
            for (u32 t = (u32)((a * 8 + xcd) * NSTAGE + sg); t < NXB; t += NCWG * NSTAGE) {
                const u32 so = (t >> 2) * (32 * KS) + (t & 3) * 128;
                u32x4 o[4];
#pragma unroll
                for (int j = 0; j < 4; ++j) o[j] = __builtin_amdgcn_raw_buffer_load_b128(xr, xlane + (j >> 1) * 64 + (j & 1) * 16, so, 0);
                unsigned char* d = my + WS_X + par * X_PAR + (size_t)p.rank * (X_PAR / NR) + (size_t)t * 4096 + lane * 16;
#pragma unroll
                for (int j = 0; j < 4; ++j) st16_sys(d + j * 1024, o[j]);
                VM0();
                if (lane == 0) __hip_atomic_store((u32*)(my + WS_FX + par * FX_PAR) + p.rank * 32 + (t >> 5) * 256 + (t & 31), call, __ATOMIC_RELAXED, __HIP_MEMORY_SCOPE_SYSTEM);
            }
            if (sg == 0) STAMP(6);
        }
        const u32 call = rfl(*(const u32*)(lds + L_CALL));
        const u32 par = call & 1u;
        const __amdgpu_buffer_rsrc_t rw = make_rsrc(my + WS_W + par * W_PAR, (u32)W_PAR);
        const u32 b_off = (u32)(jb * 4096);
        for (u32 n = (u32)sg; n < 64; n += NSTAGE) {
            u32 t;
            while ((t = rfl(lds_ld(lds, L_TAIL))) <= n) __builtin_amdgcn_s_sleep(1);
            if (t == DEAD) break;
            CFENCE();
            const u32 ent = rfl(*(const u32*)(lds + L_ORD + n * 4));
            const u32 u = ent & 63u, hdr = ent >> 6;
            const u32 so = u * (64 * 4096) + b_off;
            u32x4 v[4];
#pragma unroll
            for (int q = 0; q < 3; ++q) v[q] = __builtin_amdgcn_raw_buffer_load_b128(rw, lane * 16 + q * 1024, so, 17);
            const bool raw = (hdr & H_RAW) != 0u;
            u32 e = 0;
            if (raw) v[3] = __builtin_amdgcn_raw_buffer_load_b128(rw, lane * 16 + 3072, so, 17);
            else e = __builtin_amdgcn_raw_buffer_load_b16(rw, lane * 2 + 3072, so, 17);
            if (n >= NSLOT) {
                bool dead = false;
                while (true) {
                    const u32 c = lds_ld(lds, L_CONS + (lane & 7) * 4);
                    if (__all(c > n - NSLOT)) break;
                    if (rfl(lds_ld(lds, L_TAIL)) == DEAD) { dead = true; break; }
                    __builtin_amdgcn_s_sleep(1);
                }
                if (dead) break;
                CFENCE();
            }
            unsigned char* slot = lds + L_B + (n & (NSLOT - 1)) * 4096;
            if (raw) {
#pragma unroll
                for (int q = 0; q < 4; ++q) *(u32x4*)(slot + q * 1024 + lane * 16) = v[q];
            } else {
                const u32 basex = ((hdr >> 10) & 127u) * 0x01010101u;
                const u32 P[4][3] = {{v[0].x, v[0].y, v[0].z}, {v[0].w, v[1].x, v[1].y}, {v[1].z, v[1].w, v[2].x}, {v[2].y, v[2].z, v[2].w}};
#pragma unroll
                for (int q = 0; q < 4; ++q) {
                    const u32 d3 = ((P[q][0] >> 5) & 0x03030303u) | ((P[q][1] >> 3) & 0x0c0c0c0cu) | ((P[q][2] >> 1) & 0x10101010u) | ((P[q][2] << 1) & 0x80808080u);
                    *(u32x4*)(slot + q * 1024 + lane * 16) =
                        (u32x4){(P[q][0] & 0x9f9f9f9fu) + basex, (P[q][1] & 0x9f9f9f9fu) + basex, (P[q][2] & 0x9f9f9f9fu) + basex, d3 + basex};
                }
                const u32 cnt = hdr & 1023u;
                for (u32 e0 = 0; e0 < cnt; e0 += 64) {
                    if (e0) e = __builtin_amdgcn_raw_buffer_load_b16(rw, lane * 2 + 3072 + e0 * 2, so, 17);
                    if (e0 + (u32)lane < cnt)
                        __hip_atomic_fetch_xor((u32*)(slot + (e & 0xffcu)), ((e >> 12) & 3u) << ((e & 3u) * 8u + 5u), __ATOMIC_RELAXED, __HIP_MEMORY_SCOPE_WORKGROUP);
                }
            }
            LGKM0();
            if (lane == 0) lds_st(lds, L_STG + n * 4, 1u);
        }
        return;
    }

    __syncthreads();
    const u32 call = rfl(*(const u32*)(lds + L_CALL));
    const u32 par = call & 1u;

    if (wave == W_POLL) {
        const u64* fwp = (const u64*)(my + WS_FW + par * FW_PAR) + jb * 64 + lane;
        const __amdgpu_buffer_rsrc_t rf = make_rsrc(my + WS_FX + par * FX_PAR + ti * 1024, 1024);
        u64 seen = 0, xok = 0;
        u32 tail = 0;
        const u64 t0 = __builtin_amdgcn_s_memrealtime();
        while (seen != ~0ull) {
            const u64 f = __hip_atomic_load(fwp, __ATOMIC_RELAXED, __HIP_MEMORY_SCOPE_SYSTEM);
            if (xok != ~0ull) {
                const u32x4 g = __builtin_amdgcn_raw_buffer_load_b128(rf, lane * 16, 0, 17);
                u64 mx = __ballot(g.x >= call && g.y >= call && g.z >= call && g.w >= call);
                mx &= (mx >> 4) & 0x0f0f0f0f0f0f0f0full; mx &= mx >> 2; mx &= mx >> 1; mx &= 0x0101010101010101ull;
                xok = mx * 0xffull;
            }
            const u64 fresh = __ballot((u32)f >= call) & xok & ~seen;
            if (fresh) {
                if ((fresh >> lane) & 1) {
                    const u32 pos = tail + lanes_below(fresh);
                    *(u32*)(lds + L_ORD + pos * 4) = (u32)lane | ((u32)(f >> 32) << 6);
                }
                tail += (u32)__builtin_popcountll(fresh);
                seen |= fresh;
                LGKM0();
                if (lane == 0) lds_st(lds, L_TAIL, tail);
            } else {
                if (timed_out(tmo, call, t0)) {
                    VM0();
                    if (lane == 0) lds_st(lds, L_TAIL, DEAD);
                    break;
                }
                if (xok != ~0ull) __builtin_amdgcn_s_sleep(XSLEEP); else __builtin_amdgcn_s_sleep(POLLSLEEP);
            }
            CFENCE();
        }
        return;
    }

    const int wv = wave;
    const __amdgpu_buffer_rsrc_t rx = make_rsrc(my + WS_X + par * X_PAR, (u32)X_PAR);
    f32x16 acc0 = {}, acc1 = {};
    const u32 b_rd = (u32)(((lane >> 4) & 1) * 2048 + (lane >> 5) * 512 + (lane & 15) * 16);
    const u32 a_off = (u32)((ti * 8 + wv) * 8 * 2048 + lane * 16);
    int iters = 0;
    u32 n = 0;
    while (n < 64) {
        u32 t;
        while ((t = rfl(lds_ld(lds, L_TAIL))) <= n) __builtin_amdgcn_s_sleep(1);
        if (t == DEAD) break;
        CFENCE();
        ++iters;
        const u32 m = min(t - n, 4u);
        const u32 ent = *(const u32*)(lds + L_ORD + (n + min((u32)(lane & 3), m - 1u)) * 4);
        u32x4 av[4][2];
#pragma unroll
        for (int j = 0; j < 4; ++j) {
            const u32 u = (u32)__builtin_amdgcn_readlane(ent, j) & 63u;
            const u32 so = (u >> 3) * (u32)(X_PAR / NR) + (u & 7) * 2048;
#pragma unroll
            for (int s = 0; s < 2; ++s) av[j][s] = __builtin_amdgcn_raw_buffer_load_b128(rx, a_off + s * 1024, so, 17);
        }
        bool dead = false;
#pragma unroll
        for (int j = 0; j < 4; ++j) {
            if ((u32)j < m && !dead) {
                while (rfl(lds_ld(lds, L_STG + (n + j) * 4)) == 0u) {
                    if (rfl(lds_ld(lds, L_TAIL)) == DEAD) { dead = true; break; }
                    __builtin_amdgcn_s_sleep(1);
                }
                CFENCE();
                const unsigned char* bb = lds + L_B + ((n + j) & (NSLOT - 1)) * 4096 + b_rd;
                const u32x4 b00 = *(const u32x4*)(bb), b01 = *(const u32x4*)(bb + 256);
                const u32x4 b10 = *(const u32x4*)(bb + 1024), b11 = *(const u32x4*)(bb + 1280);
                const i32x8 af = {(int)av[j][0].x, (int)av[j][0].y, (int)av[j][0].z, (int)av[j][0].w, (int)av[j][1].x, (int)av[j][1].y, (int)av[j][1].z, (int)av[j][1].w};
                const i32x8 bf0 = {(int)b00.x, (int)b00.y, (int)b00.z, (int)b00.w, (int)b01.x, (int)b01.y, (int)b01.z, (int)b01.w};
                const i32x8 bf1 = {(int)b10.x, (int)b10.y, (int)b10.z, (int)b10.w, (int)b11.x, (int)b11.y, (int)b11.z, (int)b11.w};
                acc0 = __builtin_amdgcn_mfma_scale_f32_32x32x64_f8f6f4(af, bf0, acc0, 0, 1, 0, 0, 0, 0);
                acc1 = __builtin_amdgcn_mfma_scale_f32_32x32x64_f8f6f4(af, bf1, acc1, 0, 1, 0, 0, 0, 0);
            }
        }
        if (dead) break;
        n += m;
        LGKM0();
        if (lane == 0) lds_st(lds, L_CONS + wv * 4, n);
    }

    const bool bad = __hip_atomic_load(tmo, __ATOMIC_RELAXED, __HIP_MEMORY_SCOPE_AGENT) == call;
    const float s = bad ? __builtin_nanf("") : p.sx[0] * p.sw[0];
    float* o = p.out + (size_t)(ti * 256 + wv * 32 + (lane >> 5) * 4 + ((lane >> 4) & 1)) * NN + jb * 64 + (lane & 15) * 4;
#pragma unroll
    for (int r = 0; r < 16; r += 2) {
        const auto s0 = __builtin_amdgcn_permlane16_swap(__float_as_uint(acc0[r] * s), __float_as_uint(acc0[r + 1] * s), false, false);
        const auto s1 = __builtin_amdgcn_permlane16_swap(__float_as_uint(acc1[r] * s), __float_as_uint(acc1[r + 1] * s), false, false);
        st16_sys(o + (size_t)((r & 3) + 8 * (r >> 2)) * NN, (u32x4){s0[0], s1[0], s0[1], s1[1]});
    }
}

}

extern "C" void kernel_launch(int rank, int nranks, ncclComm_t comm, hipStream_t stream,
                              void* const* d_in, const int* in_sizes, int n_in,
                              void* d_out, int out_size,
                              const int* in_dtypes, int out_dtype,
                              void* const* d_out_all,
                              void* const* d_ws_all, size_t ws_bytes) {
    (void)comm; (void)in_sizes; (void)n_in; (void)out_size; (void)in_dtypes; (void)out_dtype; (void)d_out_all;
    if (nranks != NR || ws_bytes < WS_END || (unsigned)rank >= (unsigned)NR) {
        fprintf(stderr, "unexpected configuration\n");
        return;
    }
    Params p;
    p.x = (const unsigned char*)d_in[0];
    p.w = (const unsigned char*)d_in[1];
    p.sx = (const float*)d_in[2];
    p.sw = (const float*)d_in[3];
    p.out = (float*)d_out;
    for (int i = 0; i < NR; ++i) p.ws[i] = (unsigned char*)d_ws_all[i];
    p.rank = rank;
    p.pad = 0;
    hipLaunchKernelGGL(gemm_ag_fused, dim3(NWG), dim3(NTHREADS), 0, stream, p);
    HIP_CHECK(hipGetLastError());
}
```

```cpp
#include <hip/hip_runtime.h>
#include <rccl/rccl.h>
#include <cstdio>

#define HIP_CHECK(expr) do { hipError_t _e = (expr); if (_e != hipSuccess) { \
    fprintf(stderr, "HIP error %s at %s:%d\n", hipGetErrorString(_e), __FILE__, __LINE__); \
    return; } } while (0)

namespace {

typedef unsigned u32;
typedef unsigned long long u64;
typedef u32 u32x2 __attribute__((ext_vector_type(2)));
typedef u32 u32x4 __attribute__((ext_vector_type(4)));
typedef int i32x8 __attribute__((ext_vector_type(8)));
typedef float f32x16 __attribute__((ext_vector_type(16)));

#define ROT 0
#define POLLSLEEP 4
#define F_BAD 1
#define F_SPIN 1
#define PSLEEP 0
#define BFIRST 1
#define USEPRIO 1
#define MSLEEP 1
#define SSLEEP 1
#define APREF 0
#define PAIRS 0
#define RIDE 1
#define NSEND_ 6
#define NSTAGE_ 7
#define PACK 1

constexpr int NR = 8;
constexpr int NN = 4096;
constexpr int KS = 512;
constexpr int MS = 512;
constexpr int NWG = 256;
constexpr int NCWG = 128;
constexpr int NWORK = 8;
constexpr int NSTAGE = NSTAGE_;
constexpr int W_STG  = NWORK;
constexpr int W_POLL = W_STG + NSTAGE;
constexpr int NWAVES = W_POLL + 1;
constexpr int NTHREADS = 64 * NWAVES;
constexpr int NSEND = NSEND_;
#define WPDCAP 1000
constexpr int WPD = (NWG - NCWG) * NSEND / (NR - 1) < WPDCAP ? (NWG - NCWG) * NSEND / (NR - 1) : WPDCAP;
constexpr u32 NXB = 64;
constexpr u32 NBLK = NXB + 512;

constexpr size_t WS_W   = 0;
constexpr size_t W_PAR  = (size_t)NR * 8 * 64 * 4096;
constexpr size_t WS_X   = 2 * W_PAR;
constexpr size_t X_PAR  = (size_t)NR * 16 * 8 * 2048;
constexpr size_t WS_FL  = WS_X + 2 * X_PAR;
constexpr size_t FL_PAR = 64 * 64 * 128;
#define FLAY 0
#define FLIDX(jb, src, kc) (((jb) * 8u + (src)) * 8u + (kc))
constexpr size_t WS_FW  = WS_FL + 2 * FL_PAR;
constexpr size_t FW_PAR = 64 * 8 * 8;
constexpr size_t WS_FX  = WS_FW + 2 * FW_PAR;
constexpr size_t FX_PAR = 2 * 8 * 32 * 4;
constexpr size_t WS_Q   = WS_FX + 2 * FX_PAR;
constexpr size_t WS_CNT = WS_Q + NR * 128;
constexpr size_t WS_TMO = WS_CNT + NWG * 4;
constexpr size_t WS_END = WS_TMO + 128;
#define STAMP(slot) do {} while (0)
#define STAMPV(slot, v) do {} while (0)
#define STAMPA(area, idx, v) do {} while (0)

constexpr u64 DEADLINE_TICKS = 30000000ull;

constexpr u32 H_RAW = 1u << 15;
constexpr u32 MAXFIX = 255;
constexpr u32 MAXRIDE = 60;
constexpr u32 LOC_OWN = 1023;

constexpr int NSLOT  = 32;
constexpr int L_B    = 0;
constexpr int L_ORD  = NSLOT * 4096;
constexpr int L_STG  = L_ORD + 256;
constexpr int L_TAIL = L_STG + 8;
constexpr int L_BAD  = L_STG + 12;
constexpr int L_CALL = L_STG + 16;
constexpr int L_CONS = L_CALL + 16;
constexpr int L_BYTES = L_CONS + 32;
constexpr u32 DEAD = 0x80000000u;

struct Params {
    const unsigned char* x;
    const unsigned char* w;
    const float* sx;
    const float* sw;
    float* out;
    unsigned char* ws[NR];
    int rank;
    int pad;
};

__device__ __forceinline__ void st16_sys(void* p, u32x4 v) {
    asm volatile("global_store_dwordx4 %0, %1, off sc0 sc1\n\ts_nop 1" :: "v"(p), "v"(v) : "memory");
}
__device__ __forceinline__ void glds16_sys(const void* src, u32 dst) {
    u32 keep;
    asm volatile("s_mov_b32 %0, m0\n\ts_mov_b32 m0, %2\n\ts_nop 0\n\tglobal_load_lds_dwordx4 %1, off sc0 sc1\n\ts_mov_b32 m0, %0"
                 : "=&s"(keep) : "v"(src), "s"(dst) : "memory");
}
__device__ __forceinline__ __amdgpu_buffer_rsrc_t make_rsrc(const void* base, u32 bytes) {
    return __builtin_amdgcn_make_buffer_rsrc(const_cast<void*>(base), 0, bytes, 0x00020000);
}
template <int V> struct IC { static constexpr int value = V; };
__device__ __forceinline__ u32 rfl(u32 v) { return __builtin_amdgcn_readfirstlane(v); }
__device__ __forceinline__ u32 lds_ld(const unsigned char* lds, int off) {
    return __hip_atomic_load((const u32*)(lds + off), __ATOMIC_RELAXED, __HIP_MEMORY_SCOPE_WORKGROUP);
}
__device__ __forceinline__ void lds_st(unsigned char* lds, int off, u32 v) {
    __hip_atomic_store((u32*)(lds + off), v, __ATOMIC_RELAXED, __HIP_MEMORY_SCOPE_WORKGROUP);
}
__device__ __forceinline__ u32 lanes_below(u64 m) {
    return __builtin_amdgcn_mbcnt_hi((u32)(m >> 32), __builtin_amdgcn_mbcnt_lo((u32)m, 0u));
}
__device__ __forceinline__ u32 wave_sum(u32 v) {
    v += __builtin_amdgcn_update_dpp(0u, v, 0x111, 0xf, 0xf, false);
    v += __builtin_amdgcn_update_dpp(0u, v, 0x112, 0xf, 0xf, false);
    v += __builtin_amdgcn_update_dpp(0u, v, 0x114, 0xf, 0xf, false);
    v += __builtin_amdgcn_update_dpp(0u, v, 0x118, 0xf, 0xf, false);
    return __builtin_amdgcn_readlane(v, 15) + __builtin_amdgcn_readlane(v, 31) + __builtin_amdgcn_readlane(v, 47) + __builtin_amdgcn_readlane(v, 63);
}
#define PRIO(x) __builtin_amdgcn_s_setprio(x)
#define LGKM0() asm volatile("s_waitcnt lgkmcnt(0)" ::: "memory")
#define VM0() asm volatile("s_waitcnt vmcnt(0)" ::: "memory")
#define CFENCE() asm volatile("" ::: "memory")

__device__ __forceinline__ bool timed_out(u32* tmo, u32 call, u64 t0) {
    if (__hip_atomic_load(tmo, __ATOMIC_RELAXED, __HIP_MEMORY_SCOPE_AGENT) == call
        || __builtin_amdgcn_s_memrealtime() - t0 > DEADLINE_TICKS) {
        __hip_atomic_store(tmo, call, __ATOMIC_RELAXED, __HIP_MEMORY_SCOPE_AGENT);
        return true;
    }
    return false;
}

__global__ void __launch_bounds__(NTHREADS, (NTHREADS + 255) / 256) gemm_ag_fused(Params p) {
    __shared__ __attribute__((aligned(16))) unsigned char lds[L_BYTES];
    const int tid = threadIdx.x;
    const int lane = tid & 63;
    const int wave = rfl(tid >> 6);
    const int b = blockIdx.x;
    const int xcd = b & 7, a = b >> 3;
    unsigned char* my = p.ws[p.rank];
    u32* tmo = (u32*)(my + WS_TMO);

    if (wave == 0) STAMP(0);
    if (tid == 0) {
        u32* cnt = (u32*)(my + WS_CNT) + b;
        const u32 c = __hip_atomic_load(cnt, __ATOMIC_RELAXED, __HIP_MEMORY_SCOPE_AGENT) + 1u;
        __hip_atomic_store(cnt, c, __ATOMIC_RELAXED, __HIP_MEMORY_SCOPE_AGENT);
        *(u32*)(lds + L_CALL) = c;
    }
    if (wave == 1) {
        if (lane < 4) *(u32*)(lds + L_STG + lane * 4) = 0u;
        if (lane < 8) *(u32*)(lds + L_CONS + lane * 4) = 0u;
    }
    const u32 xlane = (u32)((lane & 31) * KS + (lane >> 5) * 32);

    if (a >= NCWG / 8) {
        const int g = ((a - NCWG / 8) * 8 + xcd) * NSEND + wave;
        if (wave >= NSEND || g >= WPD * (NR - 1)) { __syncthreads(); return; }
        const int dd = 1 + g % (NR - 1);
        const u32 wi = (u32)(g / (NR - 1));
        const int dst = (p.rank + dd) & (NR - 1);
        unsigned char* rws = p.ws[dst];
        u32* queue = (u32*)(my + WS_Q + dd * 128);
        const __amdgpu_buffer_rsrc_t xr = make_rsrc(p.x + (size_t)dst * MS * KS, MS * KS);
        u32x4 r[4];
        auto load_x = [&](u32 t) {
            const u32 so = (t >> 2) * (32 * KS) + (t & 3) * 128;
#pragma unroll
            for (int j = 0; j < 4; ++j) r[j] = __builtin_amdgcn_raw_buffer_load_b128(xr, xlane + (j >> 1) * 64 + (j & 1) * 16, so, 0);
        };
        u32 t0 = wi, t1 = wi + WPD, t2 = wi + 2 * WPD;
        if (t0 < NXB) load_x(t0);
        __syncthreads();
        const u32 call = rfl(*(const u32*)(lds + L_CALL));
        const u32 par = call & 1u;
        const u32 qbase = (call - 1u) * NBLK - 3u * WPD;
        const __amdgpu_buffer_rsrc_t wl = make_rsrc(my + WS_W + par * W_PAR + (size_t)p.rank * (W_PAR / NR), (u32)(W_PAR / NR));
        const u64* fl = (const u64*)(my + WS_FW + par * FW_PAR);
        unsigned char* xd = rws + WS_X + par * X_PAR + (size_t)p.rank * (X_PAR / NR) + lane * 16;
        unsigned char* wd = rws + WS_W + par * W_PAR + (size_t)p.rank * (W_PAR / NR) + lane * 16;
        unsigned char* fline = rws + WS_FL + par * FL_PAR + (lane & 7) * 16;
        u32* fx = (u32*)(rws + WS_FX + par * FX_PAR) + p.rank * 32;
        auto kc_of = [&](u32 t) { return (t - NXB) >> 6; };
        auto jb_of = [&](u32 t) { return (t - NXB + ROT * p.rank) & 63u; };
        auto load_w = [&](u32 t) {
            const u32 so = (kc_of(t) * 64 + jb_of(t)) * 4096;
#pragma unroll
            for (int j = 0; j < 4; ++j) r[j] = __builtin_amdgcn_raw_buffer_load_b128(wl, lane * 16 + j * 1024, so, 17);
        };
        auto wait_hdr = [&](u32 t, u32& h) -> bool {
            const u64* f = fl + jb_of(t) * 8 + kc_of(t);
            u64 v = __hip_atomic_load(f, __ATOMIC_RELAXED, __HIP_MEMORY_SCOPE_SYSTEM);
            if ((u32)v < call) {
                const u64 ts = __builtin_amdgcn_s_memrealtime();
                do {
                    if (timed_out(tmo, call, ts)) return false;
                    __builtin_amdgcn_s_sleep(2);
                    v = __hip_atomic_load(f, __ATOMIC_RELAXED, __HIP_MEMORY_SCOPE_SYSTEM);
                } while ((u32)v < call);
            }
            CFENCE();
            h = rfl((u32)(v >> 32)) & 0xffffu;
            return true;
        };
        u32 h0 = H_RAW, h1 = H_RAW, loc0 = LOC_OWN;
        u64 f1 = 0;
        if (t0 >= NXB) { if (!wait_hdr(t0, h0)) return; load_w(t0); }
        int nb = 0;
#pragma unroll 1
        while (t0 < NBLK) {
            u32 tk = 0;
            if (lane == 0) tk = __hip_atomic_fetch_add(queue, 1u, __ATOMIC_RELAXED, __HIP_MEMORY_SCOPE_AGENT);
            unsigned char* d = t0 < NXB ? xd + (size_t)t0 * 4096 : wd + (size_t)(kc_of(t0) * 64 + jb_of(t0)) * 4096;
            const u32 nl = (h0 & H_RAW) ? 64u : loc0 != LOC_OWN ? 0u : ((h0 & 255u) + 7u) >> 3;
#pragma unroll
            for (int j = 0; j < 3; ++j) st16_sys(d + j * 1024, r[j]);
            if ((u32)lane < nl) st16_sys(d + 3 * 1024, r[3]);
            h1 = H_RAW;
            if (t1 < NXB) load_x(t1);
            else if (t1 < NBLK) {
                if (rfl((u32)f1) >= call) h1 = rfl((u32)(f1 >> 32)) & 0xffffu;
                else if (!wait_hdr(t1, h1)) return;
                load_w(t1);
            }
            u64 f2 = 0;
            const bool w2 = t2 >= NXB && t2 < NBLK;
            if (w2) f2 = __hip_atomic_load(fl + jb_of(t2) * 8 + kc_of(t2), __ATOMIC_RELAXED, __HIP_MEMORY_SCOPE_SYSTEM);
            VM0();
            u32 loc1 = LOC_OWN;
            if (t0 < NXB) {
                if (lane == 0) __hip_atomic_store(fx + (t0 >> 5) * 256 + (t0 & 31), call, __ATOMIC_RELAXED, __HIP_MEMORY_SCOPE_SYSTEM);
            } else {
                const bool ride = RIDE && t1 < NBLK && !(h1 & H_RAW) && (h1 & 255u) <= MAXRIDE;
                if (ride) loc1 = t0 - NXB;
                u32x4 v = r[3];
                if (lane == 7) { v.z = call; v.w = h0 | (loc0 << 16); }
                if (lane == 7 || (ride && lane < 7)) st16_sys(fline + (size_t)FLIDX(jb_of(t0), (u32)p.rank, kc_of(t0)) * 128, v);
            }
            t0 = t1; h0 = h1; loc0 = loc1;
            t1 = t2; f1 = f2;
            t2 = rfl(tk) - qbase;
        }
        (void)nb;
        return;
    }

    const int jb = xcd * 8 + (a & 7), ti = a >> 3;

    if (wave >= W_STG && wave < W_POLL) {
        const int sg = wave - W_STG;
        PRIO(3);
        {
            const __amdgpu_buffer_rsrc_t wr = make_rsrc(p.w, KS * NN);
            const u32 wlane = (u32)((lane >> 4) * 16 * NN + (lane & 15) * 4);
            u32 id = (u32)((a * 8 + xcd) * NSTAGE + sg);
            u32 r[16];
            auto load_w = [&](u32 t) {
                const u32 so = (t >> 6) * 64 * NN + ((t + ROT * p.rank) & 63u) * 64;
#pragma unroll
                for (int j = 0; j < 16; ++j) r[j] = __builtin_amdgcn_raw_buffer_load_b32(wr, wlane + j * NN, so, 0);
            };
            if (PSLEEP) __builtin_amdgcn_s_sleep(PSLEEP);
            if (id < 512) load_w(id);
            __syncthreads();
            const u32 call = rfl(*(const u32*)(lds + L_CALL));
            const u32 par = call & 1u;
            unsigned char* scratch = lds + L_B + sg * 4096;
            for (; id < 512; id += NCWG * NSTAGE) {
                const u32 kc = id >> 6, pj = (id + ROT * p.rank) & 63u;
                u32 T[4][4];
#pragma unroll
                for (int t4 = 0; t4 < 4; ++t4) {
                    const u32 r0 = r[4 * t4], r1 = r[4 * t4 + 1], r2 = r[4 * t4 + 2], r3 = r[4 * t4 + 3];
                    const u32 aa = __builtin_amdgcn_perm(r1, r0, 0x05010400u), bb = __builtin_amdgcn_perm(r1, r0, 0x07030602u);
                    const u32 cc = __builtin_amdgcn_perm(r3, r2, 0x05010400u), ee = __builtin_amdgcn_perm(r3, r2, 0x07030602u);
                    T[0][t4] = __builtin_amdgcn_perm(cc, aa, 0x05040100u);
                    T[1][t4] = __builtin_amdgcn_perm(cc, aa, 0x07060302u);
                    T[2][t4] = __builtin_amdgcn_perm(ee, bb, 0x05040100u);
                    T[3][t4] = __builtin_amdgcn_perm(ee, bb, 0x07060302u);
                }
                if (id + NCWG * NSTAGE < 512) load_w(id + NCWG * NSTAGE);
                u32 hdr = H_RAW;
                u32x4 o[4];
                u32 sum = 0;
#pragma unroll
                for (int q = 0; q < 4; ++q)
#pragma unroll
                    for (int t4 = 0; t4 < 4; ++t4) sum = __builtin_amdgcn_sad_u8(T[q][t4] & 0x7f7f7f7fu, 0u, sum);
                const int bs = ((int)wave_sum(sum) - 82944) >> 12;
                const u32 base = (u32)min(max(bs, 0), 96);
                const u32 basex = base * 0x01010101u;
                u32 P[4][3];
                u32 total = 0;
#pragma unroll
                for (int q = 0; q < 4; ++q) {
                    u32 D[4];
#pragma unroll
                    for (int t4 = 0; t4 < 4; ++t4) {
                        const u32 d = T[q][t4];
                        const u32 code = ((d | 0x80808080u) - basex) & 0x1f1f1f1fu;
                        D[t4] = (d & 0x80808080u) | code;
                        u32 fix = (d ^ (code + basex)) & 0x60606060u;
                        u64 m = __ballot(fix != 0u);
                        while (m) {
                            if (fix != 0u) {
                                const u32 sh = (u32)__builtin_ctz(fix) & 24u;
                                const u32 idx = total + lanes_below(m);
                                const u32 ent = (u32)(q * 1024 + t4 * 4) + (u32)lane * 16u + (sh >> 3) + (((fix >> (sh + 5u)) & 3u) << 12);
                                if (idx < MAXFIX) *(unsigned short*)(scratch + idx * 2) = (unsigned short)ent;
                                fix &= ~(0x60u << sh);
                            }
                            total += (u32)__builtin_popcountll(m);
                            m = __ballot(fix != 0u);
                        }
                    }
                    P[q][0] = D[0] | ((D[3] & 0x03030303u) << 5);
                    P[q][1] = D[1] | ((D[3] & 0x0c0c0c0cu) << 3);
                    P[q][2] = D[2] | ((D[3] & 0x10101010u) << 1) | ((D[3] & 0x80808080u) >> 1);
                }
                if (total <= MAXFIX) {
                    hdr = total | (base << 8);
                    o[0] = (u32x4){P[0][0], P[0][1], P[0][2], P[1][0]};
                    o[1] = (u32x4){P[1][1], P[1][2], P[2][0], P[2][1]};
                    o[2] = (u32x4){P[2][2], P[3][0], P[3][1], P[3][2]};
                    o[3] = *(const u32x4*)(scratch + lane * 16);
                } else
                {
#pragma unroll
                    for (int q = 0; q < 4; ++q) o[q] = (u32x4){T[q][0], T[q][1], T[q][2], T[q][3]};
                }
                unsigned char* d = my + WS_W + par * W_PAR + (size_t)p.rank * (W_PAR / NR) + (size_t)(kc * 64 + pj) * 4096 + lane * 16;
#pragma unroll
                for (int j = 0; j < 4; ++j) st16_sys(d + j * 1024, o[j]);
                VM0();
                if (lane == 0) __hip_atomic_store((u64*)(my + WS_FW + par * FW_PAR) + pj * 8 + kc, (u64)call | (((u64)hdr | ((u64)LOC_OWN << 16)) << 32), __ATOMIC_RELAXED, __HIP_MEMORY_SCOPE_SYSTEM);
            }
            const __amdgpu_buffer_rsrc_t xr = make_rsrc(p.x + (size_t)p.rank * MS * KS, MS * KS);
            for (u32 t = (u32)((a * 8 + xcd) * NSTAGE + sg); t < NXB; t += NCWG * NSTAGE) {
                const u32 so = (t >> 2) * (32 * KS) + (t & 3) * 128;
                u32x4 o[4];
#pragma unroll
                for (int j = 0; j < 4; ++j) o[j] = __builtin_amdgcn_raw_buffer_load_b128(xr, xlane + (j >> 1) * 64 + (j & 1) * 16, so, 0);
                unsigned char* d = my + WS_X + par * X_PAR + (size_t)p.rank * (X_PAR / NR) + (size_t)t * 4096 + lane * 16;
#pragma unroll
                for (int j = 0; j < 4; ++j) st16_sys(d + j * 1024, o[j]);
                VM0();
                if (lane == 0) __hip_atomic_store((u32*)(my + WS_FX + par * FX_PAR) + p.rank * 32 + (t >> 5) * 256 + (t & 31), call, __ATOMIC_RELAXED, __HIP_MEMORY_SCOPE_SYSTEM);
            }
            if (sg == 0) STAMP(6);
        }
        const u32 call = rfl(*(const u32*)(lds + L_CALL));
        const u32 par = call & 1u;
        const __amdgpu_buffer_rsrc_t rw = make_rsrc(my + WS_W + par * W_PAR, (u32)W_PAR);
        const __amdgpu_buffer_rsrc_t rall = make_rsrc(my, (u32)WS_END);
        const u32 b_off = (u32)(jb * 4096);
        for (u32 n = (u32)sg; n < 64; n += NSTAGE) {
            u32 t;
            PRIO(0);
            while ((t = rfl(lds_ld(lds, L_TAIL))) <= n) __builtin_amdgcn_s_sleep(SSLEEP);
            PRIO(3);
            if (t == DEAD) break;
            CFENCE();
            const u32 ent = rfl(*(const u32*)(lds + L_ORD + n * 4));
            const u32 u = ent & 63u, hdr = (ent >> 6) & 0xffffu, loc = ent >> 22;
            const u32 so = u * (64 * 4096) + b_off;
            u32x4 v[4];
#pragma unroll
            for (int q = 0; q < 3; ++q) v[q] = __builtin_amdgcn_raw_buffer_load_b128(rw, lane * 16 + q * 1024, so, 17);
            const bool raw = (hdr & H_RAW) != 0u;
            u32 e = 0;
            if (raw) v[3] = __builtin_amdgcn_raw_buffer_load_b128(rw, lane * 16 + 3072, so, 17);
            const u32 lo = loc == LOC_OWN ? (u32)(WS_W + par * W_PAR) + so + 3072u
                                          : (u32)(WS_FL + par * FL_PAR) + FLIDX((loc + ROT * (u >> 3)) & 63u, u >> 3, loc >> 6) * 128u;
            if (!raw) e = __builtin_amdgcn_raw_buffer_load_b16(rall, lane * 2, lo, 17);
            if (n >= NSLOT) {
                bool dead = false;
                while (true) {
                    const u32 c = lds_ld(lds, L_CONS + (lane & 7) * 4);
                    if (__all(c > n - NSLOT)) break;
                    if (rfl(lds_ld(lds, L_TAIL)) == DEAD) { dead = true; break; }
                    __builtin_amdgcn_s_sleep(1);
                }
                if (dead) break;
                CFENCE();
            }
            unsigned char* slot = lds + L_B + (n & (NSLOT - 1)) * 4096;
            if (raw) {
#pragma unroll
                for (int q = 0; q < 4; ++q) *(u32x4*)(slot + q * 1024 + lane * 16) = v[q];
            } else {
                const u32 basex = ((hdr >> 8) & 127u) * 0x01010101u;
                const u32 P[4][3] = {{v[0].x, v[0].y, v[0].z}, {v[0].w, v[1].x, v[1].y}, {v[1].z, v[1].w, v[2].x}, {v[2].y, v[2].z, v[2].w}};
#pragma unroll
                for (int q = 0; q < 4; ++q) {
                    const u32 d3 = ((P[q][0] >> 5) & 0x03030303u) | ((P[q][1] >> 3) & 0x0c0c0c0cu) | ((P[q][2] >> 1) & 0x10101010u) | ((P[q][2] << 1) & 0x80808080u);
                    *(u32x4*)(slot + q * 1024 + lane * 16) =
                        (u32x4){(P[q][0] & 0x9f9f9f9fu) + basex, (P[q][1] & 0x9f9f9f9fu) + basex, (P[q][2] & 0x9f9f9f9fu) + basex, d3 + basex};
                }
                const u32 cnt = hdr & 255u;
                for (u32 e0 = 0; e0 < cnt; e0 += 64) {
                    if (e0) e = __builtin_amdgcn_raw_buffer_load_b16(rall, lane * 2 + e0 * 2, lo, 17);
                    if (e0 + (u32)lane < cnt)
                        __hip_atomic_fetch_xor((u32*)(slot + (e & 0xffcu)), ((e >> 12) & 3u) << ((e & 3u) * 8u + 5u), __ATOMIC_RELAXED, __HIP_MEMORY_SCOPE_WORKGROUP);
                }
            }
            LGKM0();
            if (lane == 0) __hip_atomic_fetch_or((u32*)__builtin_assume_aligned(lds + L_STG + (n >> 5) * 4, 4), 1u << (n & 31u), __ATOMIC_RELAXED, __HIP_MEMORY_SCOPE_WORKGROUP);
        }
        return;
    }

    __syncthreads();
    const u32 call = rfl(*(const u32*)(lds + L_CALL));
    const u32 par = call & 1u;

    if (wave == W_POLL) {
        PRIO(2);
        const u64* fwp = (lane >> 3) == p.rank ? (const u64*)(my + WS_FW + par * FW_PAR) + jb * 8 + (lane & 7)
                                               : (const u64*)(my + WS_FL + par * FL_PAR + (size_t)FLIDX((u32)jb, (u32)(lane >> 3), (u32)(lane & 7)) * 128 + 120);
        const __amdgpu_buffer_rsrc_t rf = make_rsrc(my + WS_FX + par * FX_PAR + ti * 1024, 1024);
        u64 seen = 0, xok = 0;
        u32 tail = 0, spins = 0;
        const u64 t0 = __builtin_amdgcn_s_memrealtime();
        while (seen != ~0ull) {
            u64 f = 0;
            if (!((seen >> lane) & 1ull)) f = __hip_atomic_load(fwp, __ATOMIC_RELAXED, __HIP_MEMORY_SCOPE_SYSTEM);
            if (xok != ~0ull) {
                const u32x4 g = __builtin_amdgcn_raw_buffer_load_b128(rf, lane * 16, 0, 17);
                u64 mx = __ballot(g.x >= call && g.y >= call && g.z >= call && g.w >= call);
                mx &= (mx >> 4) & 0x0f0f0f0f0f0f0f0full; mx &= mx >> 2; mx &= mx >> 1; mx &= 0x0101010101010101ull;
                xok = mx * 0xffull;
            }
            const u64 fresh = __ballot((u32)f >= call) & xok & ~seen;
            if (fresh) {
                if ((fresh >> lane) & 1) {
                    const u32 pos = tail + lanes_below(fresh);
                    *(u32*)(lds + L_ORD + pos * 4) = (u32)lane | ((u32)(f >> 32) << 6);
                }
                tail += (u32)__builtin_popcountll(fresh);
                seen |= fresh;
                LGKM0();
                if (lane == 0) lds_st(lds, L_TAIL, tail);
            } else {
                if ((!F_SPIN || (++spins & 15u) == 0u) && timed_out(tmo, call, t0)) {
                    VM0();
                    if (lane == 0) { lds_st(lds, L_BAD, 2u); lds_st(lds, L_TAIL, DEAD); }
                    return;
                }
                __builtin_amdgcn_s_sleep(POLLSLEEP);
            }
            CFENCE();
        }
        {
            const u32 tv = __hip_atomic_load(tmo, __ATOMIC_RELAXED, __HIP_MEMORY_SCOPE_AGENT);
            if (lane == 0) lds_st(lds, L_BAD, tv == call ? 2u : 1u);
        }
        return;
    }

    const int wv = wave;
    const __amdgpu_buffer_rsrc_t rx = make_rsrc(my + WS_X + par * X_PAR, (u32)X_PAR);
    const float scale = p.sx[0] * p.sw[0];
    f32x16 acc0 = {}, acc1 = {};
    const u32 b_rd = (u32)(((lane >> 4) & 1) * 2048 + (lane >> 5) * 512 + (lane & 15) * 16);
    const u32 a_off = (u32)((ti * 8 + wv) * 8 * 2048 + lane * 16);
    const u32 lds0 = (u32)(size_t)(__attribute__((address_space(3))) unsigned char*)lds;
    int iters = 0;
    u32 n = 0, t = 0, ord = 0;
    u64 staged = 0;
    auto snapshot = [&]() {
        u32x4 c;
        asm volatile("ds_read_b128 %0, %2\n\tds_read_b32 %1, %3\n\ts_waitcnt lgkmcnt(0)"
                     : "=&v"(c), "=&v"(ord) : "v"(lds0 + (u32)L_STG), "v"(lds0 + (u32)L_ORD + (u32)lane * 4u) : "memory");
        t = rfl(c.z);
        staged = ((u64)rfl(c.y) << 32) | rfl(c.x);
    };
    struct BF { u32x4 b00, b01, b10, b11; };
    auto read_b = [&](u32 i) {
        const unsigned char* bb = lds + L_B + (i & (NSLOT - 1)) * 4096 + b_rd;
        BF f;
        f.b00 = *(const u32x4*)(bb); f.b01 = *(const u32x4*)(bb + 256);
        f.b10 = *(const u32x4*)(bb + 1024); f.b11 = *(const u32x4*)(bb + 1280);
        return f;
    };
    auto mma = [&](const u32x4 (&av)[2], const BF& f) {
        const i32x8 af = {(int)av[0].x, (int)av[0].y, (int)av[0].z, (int)av[0].w, (int)av[1].x, (int)av[1].y, (int)av[1].z, (int)av[1].w};
        const i32x8 bf0 = {(int)f.b00.x, (int)f.b00.y, (int)f.b00.z, (int)f.b00.w, (int)f.b01.x, (int)f.b01.y, (int)f.b01.z, (int)f.b01.w};
        const i32x8 bf1 = {(int)f.b10.x, (int)f.b10.y, (int)f.b10.z, (int)f.b10.w, (int)f.b11.x, (int)f.b11.y, (int)f.b11.z, (int)f.b11.w};
        acc0 = __builtin_amdgcn_mfma_scale_f32_32x32x64_f8f6f4(af, bf0, acc0, 0, 1, 0, 0, 0, 0);
        acc1 = __builtin_amdgcn_mfma_scale_f32_32x32x64_f8f6f4(af, bf1, acc1, 0, 1, 0, 0, 0, 0);
    };
    while (n < 64) {
        if (t <= n) {
            PRIO(0);
            while (true) { snapshot(); if (t > n) break; __builtin_amdgcn_s_sleep(MSLEEP); }
            PRIO(1);
        }
        if (t == DEAD) break;
        ++iters;
        const u32 m = min(t - n, 4u);
        u32x4 av[4][2];
#pragma unroll
        for (int j = 0; j < 4; ++j) {
            const u32 u = (u32)__builtin_amdgcn_readlane(ord, n + min((u32)j, m - 1u)) & 63u;
            const u32 so = (u >> 3) * (u32)(X_PAR / NR) + (u & 7) * 2048;
#pragma unroll
            for (int s = 0; s < 2; ++s) av[j][s] = __builtin_amdgcn_raw_buffer_load_b128(rx, a_off + s * 1024, so, 17);
        }
        bool dead = false;
        u32 j = 0;
        auto ready = [&](u32 i) { return ((staged >> i) & 1ull) != 0ull; };
        auto handle = [&](auto JC) {
            constexpr int J = decltype(JC)::value;
            while (!ready(n + J)) {
                snapshot();
                if (t == DEAD) { dead = true; return; }
                if (!ready(n + J)) __builtin_amdgcn_s_sleep(MSLEEP);
            }
            BF f0 = read_b(n + J);
            asm volatile("" : "+v"(f0.b00), "+v"(f0.b01), "+v"(f0.b10), "+v"(f0.b11));
            mma(av[J], f0);
            j = J + 1;
        };
        if (j == 0u) handle(IC<0>{});
        if (!dead && j == 1u && j < m) handle(IC<1>{});
        if (!dead && j == 2u && j < m) handle(IC<2>{});
        if (!dead && j == 3u && j < m) handle(IC<3>{});
        if (dead) break;
        n += m;
        LGKM0();
        if (lane == 0) lds_st(lds, L_CONS + wv * 4, n);
        snapshot();
    }

    u32 bad;
    while ((bad = rfl(lds_ld(lds, L_BAD))) == 0u) __builtin_amdgcn_s_sleep(1);
    const float s = bad == 2u ? __builtin_nanf("") : scale;
    float* o = p.out + (size_t)(ti * 256 + wv * 32 + (lane >> 5) * 4 + ((lane >> 4) & 1)) * NN + jb * 64 + (lane & 15) * 4;
#pragma unroll
    for (int r = 0; r < 16; r += 2) {
        const auto s0 = __builtin_amdgcn_permlane16_swap(__float_as_uint(acc0[r] * s), __float_as_uint(acc0[r + 1] * s), false, false);
        const auto s1 = __builtin_amdgcn_permlane16_swap(__float_as_uint(acc1[r] * s), __float_as_uint(acc1[r + 1] * s), false, false);
        st16_sys(o + (size_t)((r & 3) + 8 * (r >> 2)) * NN, (u32x4){s0[0], s1[0], s0[1], s1[1]});
    }
}

}

extern "C" void kernel_launch(int rank, int nranks, ncclComm_t comm, hipStream_t stream,
                              void* const* d_in, const int* in_sizes, int n_in,
                              void* d_out, int out_size,
                              const int* in_dtypes, int out_dtype,
                              void* const* d_out_all,
                              void* const* d_ws_all, size_t ws_bytes) {
    (void)comm; (void)in_sizes; (void)n_in; (void)out_size; (void)in_dtypes; (void)out_dtype; (void)d_out_all;
    if (nranks != NR || ws_bytes < WS_END || (unsigned)rank >= (unsigned)NR) {
        fprintf(stderr, "unexpected configuration\n");
        return;
    }
    Params p;
    p.x = (const unsigned char*)d_in[0];
    p.w = (const unsigned char*)d_in[1];
    p.sx = (const float*)d_in[2];
    p.sw = (const float*)d_in[3];
    p.out = (float*)d_out;
    for (int i = 0; i < NR; ++i) p.ws[i] = (unsigned char*)d_ws_all[i];
    p.rank = rank;
    p.pad = 0;
    hipLaunchKernelGGL(gemm_ag_fused, dim3(NWG), dim3(NTHREADS), 0, stream, p);
    HIP_CHECK(hipGetLastError());
}
```

```cpp
#include <hip/hip_runtime.h>
#include <rccl/rccl.h>
#include <cstdio>

#define HIP_CHECK(expr) do { hipError_t _e = (expr); if (_e != hipSuccess) { \
    fprintf(stderr, "HIP error %s at %s:%d\n", hipGetErrorString(_e), __FILE__, __LINE__); \
    return; } } while (0)

namespace {

typedef unsigned u32;
typedef unsigned long long u64;
typedef u32 u32x2 __attribute__((ext_vector_type(2)));
typedef u32 u32x4 __attribute__((ext_vector_type(4)));
typedef int i32x8 __attribute__((ext_vector_type(8)));
typedef float f32x16 __attribute__((ext_vector_type(16)));

#define ROT 0
#define POLLSLEEP 4
#define F_BAD 1
#define F_SPIN 1
#define PSLEEP 0
#define BFIRST 1
#define USEPRIO 1
#define MSLEEP 1
#define SSLEEP 1
#define APREF 0
#define PAIRS 0
#define RIDE 1
#define STAGD 60
#define STAG0 32
#define STAGOFF 130
#define POLL2 0
#define FMT5 1
#define F5FROM 2
#define LATE 0
#define NSEND_ 5
#define NSTAGE_ 7
#define PACK 1

constexpr int NR = 8;
constexpr int NN = 4096;
constexpr int KS = 512;
constexpr int MS = 512;
constexpr int NWG = 256;
constexpr int NCWG = 128;
constexpr int NWORK = 8;
constexpr int NSTAGE = NSTAGE_;
constexpr int W_STG  = NWORK;
constexpr int W_POLL = W_STG + NSTAGE;
constexpr int NWAVES = W_POLL + 1;
constexpr int NTHREADS = 64 * NWAVES;
constexpr int NSEND = NSEND_;
#define WPDCAP 1000
constexpr int WPD = (NWG - NCWG) * NSEND / (NR - 1) < WPDCAP ? (NWG - NCWG) * NSEND / (NR - 1) : WPDCAP;
constexpr u32 NXB = 64;
constexpr u32 NBLK = NXB + 512;

constexpr size_t WS_W   = 0;
constexpr size_t W_PAR  = (size_t)NR * 8 * 64 * 4096;
constexpr size_t WS_X   = 2 * W_PAR;
constexpr size_t X_PAR  = (size_t)NR * 16 * 8 * 2048;
constexpr size_t WS_FL  = WS_X + 2 * X_PAR;
constexpr size_t FL_PAR = 64 * 64 * 128;
#define FLAY 0
#define FLIDX(jb, src, kc) (((jb) * 8u + (src)) * 8u + (kc))
constexpr size_t WS_FW  = WS_FL + 2 * FL_PAR;
constexpr size_t FW_PAR = 64 * 8 * 8;
constexpr size_t WS_FX  = WS_FW + 2 * FW_PAR;
constexpr size_t FX_PAR = 2 * 8 * 32 * 4;
constexpr size_t WS_Q   = WS_FX + 2 * FX_PAR;
constexpr size_t WS_CNT = WS_Q + NR * 128;
constexpr size_t WS_TMO = WS_CNT + NWG * 4;
constexpr size_t WS_END = WS_TMO + 128;
#define STAMP(slot) do {} while (0)
#define STAMPV(slot, v) do {} while (0)
#define STAMPA(area, idx, v) do {} while (0)

constexpr u64 DEADLINE_TICKS = 30000000ull;

constexpr u32 H_RAW = 127u << 8;
constexpr u32 H_F5 = 1u << 15;
constexpr u32 MAXFIX = 255;
constexpr u32 MAXRIDE = 60;
constexpr u32 LOC_OWN = 1023;

constexpr int NSLOT  = 32;
constexpr int L_B    = 0;
constexpr int L_ORD  = NSLOT * 4096;
constexpr int L_STG  = L_ORD + 256;
constexpr int L_TAIL = L_STG + 8;
constexpr int L_BAD  = L_STG + 12;
constexpr int L_CALL = L_STG + 16;
constexpr int L_CONS = L_CALL + 16;
constexpr int L_BYTES = L_CONS + 32;
constexpr u32 DEAD = 0x80000000u;

struct Params {
    const unsigned char* x;
    const unsigned char* w;
    const float* sx;
    const float* sw;
    float* out;
    unsigned char* ws[NR];
    int rank;
    int pad;
};

__device__ __forceinline__ void st16_sys(void* p, u32x4 v) {
    asm volatile("global_store_dwordx4 %0, %1, off sc0 sc1\n\ts_nop 1" :: "v"(p), "v"(v) : "memory");
}
__device__ __forceinline__ void glds16_sys(const void* src, u32 dst) {
    u32 keep;
    asm volatile("s_mov_b32 %0, m0\n\ts_mov_b32 m0, %2\n\ts_nop 0\n\tglobal_load_lds_dwordx4 %1, off sc0 sc1\n\ts_mov_b32 m0, %0"
                 : "=&s"(keep) : "v"(src), "s"(dst) : "memory");
}
__device__ __forceinline__ __amdgpu_buffer_rsrc_t make_rsrc(const void* base, u32 bytes) {
    return __builtin_amdgcn_make_buffer_rsrc(const_cast<void*>(base), 0, bytes, 0x00020000);
}
template <int V> struct IC { static constexpr int value = V; };
__device__ __forceinline__ u32 rfl(u32 v) { return __builtin_amdgcn_readfirstlane(v); }
__device__ __forceinline__ u32 lds_ld(const unsigned char* lds, int off) {
    return __hip_atomic_load((const u32*)(lds + off), __ATOMIC_RELAXED, __HIP_MEMORY_SCOPE_WORKGROUP);
}
__device__ __forceinline__ void lds_st(unsigned char* lds, int off, u32 v) {
    __hip_atomic_store((u32*)(lds + off), v, __ATOMIC_RELAXED, __HIP_MEMORY_SCOPE_WORKGROUP);
}
__device__ __forceinline__ u32 lanes_below(u64 m) {
    return __builtin_amdgcn_mbcnt_hi((u32)(m >> 32), __builtin_amdgcn_mbcnt_lo((u32)m, 0u));
}
__device__ __forceinline__ bool is_raw(u32 h) { return ((h >> 8) & 127u) == 127u; }
__device__ __forceinline__ u32 scan_add(u32 v) {
    v += __builtin_amdgcn_update_dpp(0u, v, 0x111, 0xf, 0xf, false);
    v += __builtin_amdgcn_update_dpp(0u, v, 0x112, 0xf, 0xf, false);
    v += __builtin_amdgcn_update_dpp(0u, v, 0x114, 0xf, 0xf, false);
    v += __builtin_amdgcn_update_dpp(0u, v, 0x118, 0xf, 0xf, false);
    v += __builtin_amdgcn_update_dpp(0u, v, 0x142, 0xa, 0xf, false);
    v += __builtin_amdgcn_update_dpp(0u, v, 0x143, 0xc, 0xf, false);
    return v;
}
__device__ __forceinline__ u32 scan_max(u32 v) {
    v = max(v, (u32)__builtin_amdgcn_update_dpp(0u, v, 0x111, 0xf, 0xf, false));
    v = max(v, (u32)__builtin_amdgcn_update_dpp(0u, v, 0x112, 0xf, 0xf, false));
    v = max(v, (u32)__builtin_amdgcn_update_dpp(0u, v, 0x114, 0xf, 0xf, false));
    v = max(v, (u32)__builtin_amdgcn_update_dpp(0u, v, 0x118, 0xf, 0xf, false));
    v = max(v, (u32)__builtin_amdgcn_update_dpp(0u, v, 0x142, 0xa, 0xf, false));
    v = max(v, (u32)__builtin_amdgcn_update_dpp(0u, v, 0x143, 0xc, 0xf, false));
    return v;
}
__device__ __forceinline__ u32 lane_before(u32 v) { return __builtin_amdgcn_update_dpp(0u, v, 0x138, 0xf, 0xf, false); }
__device__ __forceinline__ u32 wave_sum(u32 v) {
    v += __builtin_amdgcn_update_dpp(0u, v, 0x111, 0xf, 0xf, false);
    v += __builtin_amdgcn_update_dpp(0u, v, 0x112, 0xf, 0xf, false);
    v += __builtin_amdgcn_update_dpp(0u, v, 0x114, 0xf, 0xf, false);
    v += __builtin_amdgcn_update_dpp(0u, v, 0x118, 0xf, 0xf, false);
    return __builtin_amdgcn_readlane(v, 15) + __builtin_amdgcn_readlane(v, 31) + __builtin_amdgcn_readlane(v, 47) + __builtin_amdgcn_readlane(v, 63);
}
#define PRIO(x) __builtin_amdgcn_s_setprio(x)
#define LGKM0() asm volatile("s_waitcnt lgkmcnt(0)" ::: "memory")
#define VM0() asm volatile("s_waitcnt vmcnt(0)" ::: "memory")
#define CFENCE() asm volatile("" ::: "memory")

__device__ __forceinline__ bool timed_out(u32* tmo, u32 call, u64 t0) {
    if (__hip_atomic_load(tmo, __ATOMIC_RELAXED, __HIP_MEMORY_SCOPE_AGENT) == call
        || __builtin_amdgcn_s_memrealtime() - t0 > DEADLINE_TICKS) {
        __hip_atomic_store(tmo, call, __ATOMIC_RELAXED, __HIP_MEMORY_SCOPE_AGENT);
        return true;
    }
    return false;
}

__global__ void __launch_bounds__(NTHREADS, (NTHREADS + 255) / 256) gemm_ag_fused(Params p) {
    __shared__ __attribute__((aligned(16))) unsigned char lds[L_BYTES];
    const int tid = threadIdx.x;
    const int lane = tid & 63;
    const int wave = rfl(tid >> 6);
    const int b = blockIdx.x;
    const int xcd = b & 7, a = b >> 3;
    unsigned char* my = p.ws[p.rank];
    u32* tmo = (u32*)(my + WS_TMO);

    if (wave == 0) STAMP(0);
    if (tid == 0) {
        u32* cnt = (u32*)(my + WS_CNT) + b;
        const u32 c = __hip_atomic_load(cnt, __ATOMIC_RELAXED, __HIP_MEMORY_SCOPE_AGENT) + 1u;
        __hip_atomic_store(cnt, c, __ATOMIC_RELAXED, __HIP_MEMORY_SCOPE_AGENT);
        *(u32*)(lds + L_CALL) = c;
    }
    if (wave == 1) {
        if (lane < 4) *(u32*)(lds + L_STG + lane * 4) = 0u;
        if (lane < 8) *(u32*)(lds + L_CONS + lane * 4) = 0u;
    }
    const u32 xlane = (u32)((lane & 31) * KS + (lane >> 5) * 32);

    if (a >= NCWG / 8) {
        const int g = ((a - NCWG / 8) * 8 + xcd) * NSEND + wave;
        if (wave >= NSEND || g >= WPD * (NR - 1)) { __syncthreads(); return; }
        const int dd = 1 + g % (NR - 1);
        const u32 wi = (u32)(g / (NR - 1));
        const int dst = (p.rank + dd) & (NR - 1);
        unsigned char* rws = p.ws[dst];
        u32* queue = (u32*)(my + WS_Q + dd * 128);
        const __amdgpu_buffer_rsrc_t xr = make_rsrc(p.x + (size_t)dst * MS * KS, MS * KS);
        u32x4 r[4];
        auto load_x = [&](u32 t) {
            const u32 so = (t >> 2) * (32 * KS) + (t & 3) * 128;
#pragma unroll
            for (int j = 0; j < 4; ++j) r[j] = __builtin_amdgcn_raw_buffer_load_b128(xr, xlane + (j >> 1) * 64 + (j & 1) * 16, so, 0);
        };
        const u64 tstart = __builtin_amdgcn_s_memrealtime() + STAGOFF;
        u32 t0 = wi, t1 = wi + WPD, t2 = wi + 2 * WPD;
        if (t0 < NXB) load_x(t0);
        __syncthreads();
        const u32 call = rfl(*(const u32*)(lds + L_CALL));
        const u32 par = call & 1u;
        const u32 qbase = (call - 1u) * NBLK - 3u * WPD;
        const __amdgpu_buffer_rsrc_t wl = make_rsrc(my + WS_W + par * W_PAR + (size_t)p.rank * (W_PAR / NR), (u32)(W_PAR / NR));
        const u64* fl = (const u64*)(my + WS_FW + par * FW_PAR);
        unsigned char* xd = rws + WS_X + par * X_PAR + (size_t)p.rank * (X_PAR / NR) + lane * 16;
        unsigned char* wd = rws + WS_W + par * W_PAR + (size_t)p.rank * (W_PAR / NR) + lane * 16;
        unsigned char* fline = rws + WS_FL + par * FL_PAR + (lane & 7) * 16;
        u32* fx = (u32*)(rws + WS_FX + par * FX_PAR) + p.rank * 32;
        auto kc_of = [&](u32 t) { return (t - NXB) >> 6; };
        auto jb_of = [&](u32 t) { return (t - NXB + ROT * p.rank) & 63u; };
        auto load_w = [&](u32 t) {
            const u32 so = (kc_of(t) * 64 + jb_of(t)) * 4096;
#pragma unroll
            for (int j = 0; j < 4; ++j) r[j] = __builtin_amdgcn_raw_buffer_load_b128(wl, lane * 16 + j * 1024, so, 17);
        };
        auto wait_hdr = [&](u32 t, u32& h) -> bool {
            const u64* f = fl + jb_of(t) * 8 + kc_of(t);
            u64 v = __hip_atomic_load(f, __ATOMIC_RELAXED, __HIP_MEMORY_SCOPE_SYSTEM);
            if ((u32)v < call) {
                const u64 ts = __builtin_amdgcn_s_memrealtime();
                do {
                    if (timed_out(tmo, call, ts)) return false;
                    __builtin_amdgcn_s_sleep(2);
                    v = __hip_atomic_load(f, __ATOMIC_RELAXED, __HIP_MEMORY_SCOPE_SYSTEM);
                } while ((u32)v < call);
            }
            CFENCE();
            h = rfl((u32)(v >> 32)) & 0xffffu;
            return true;
        };
        u32 h0 = H_RAW, h1 = H_RAW, loc0 = LOC_OWN;
        u64 f1 = 0;
        if (t0 >= NXB) { if (!wait_hdr(t0, h0)) return; load_w(t0); }
        if (wi > STAG0) {
            const u64 until = tstart + (u64)((wi - STAG0) * STAGD) / 10u;
            while (__builtin_amdgcn_s_memrealtime() < until) __builtin_amdgcn_s_sleep(1);
        }
        int nb = 0;
#pragma unroll 1
        while (t0 < NBLK) {
            u32 tk = 0;
            if (lane == 0) tk = __hip_atomic_fetch_add(queue, 1u, __ATOMIC_RELAXED, __HIP_MEMORY_SCOPE_AGENT);
            unsigned char* d = t0 < NXB ? xd + (size_t)t0 * 4096 : wd + (size_t)(kc_of(t0) * 64 + jb_of(t0)) * 4096;
            const u32 nl = is_raw(h0) ? 64u : loc0 != LOC_OWN ? 0u : ((h0 & 255u) + 7u) >> 3;
            const u32 nl2 = (h0 & H_F5) ? 56u : 64u;
            st16_sys(d, r[0]);
            st16_sys(d + 1024, r[1]);
            if ((u32)lane < nl2) st16_sys(d + 2 * 1024, r[2]);
            if ((u32)lane < nl) st16_sys(d + 3 * 1024, r[3]);
            h1 = H_RAW;
            bool late = false;
            if (t1 < NXB) load_x(t1);
            else if (t1 < NBLK) {
                if (rfl((u32)f1) >= call) { h1 = rfl((u32)(f1 >> 32)) & 0xffffu; load_w(t1); }
                else { if (!wait_hdr(t1, h1)) return; load_w(t1); }
            }
            u64 f2 = 0;
            const bool w2 = t2 >= NXB && t2 < NBLK;
            if (w2) f2 = __hip_atomic_load(fl + jb_of(t2) * 8 + kc_of(t2), __ATOMIC_RELAXED, __HIP_MEMORY_SCOPE_SYSTEM);
            VM0();
            u32 loc1 = LOC_OWN;
            if (t0 < NXB) {
                if (lane == 0) __hip_atomic_store(fx + (t0 >> 5) * 256 + (t0 & 31), call, __ATOMIC_RELAXED, __HIP_MEMORY_SCOPE_SYSTEM);
            } else {
                const bool ride = RIDE && t1 < NBLK && !is_raw(h1) && (h1 & 255u) <= MAXRIDE;
                if (ride) loc1 = t0 - NXB;
                u32x4 v = r[3];
                if (lane == 7) { v.z = call; v.w = h0 | (loc0 << 16); }
                if (lane == 7 || (ride && lane < 7)) st16_sys(fline + (size_t)FLIDX(jb_of(t0), (u32)p.rank, kc_of(t0)) * 128, v);
            }
            if (late) {
                if (!wait_hdr(t1, h1)) return;
                load_w(t1);
            }
            t0 = t1; h0 = h1; loc0 = loc1;
            t1 = t2; f1 = f2;
            t2 = rfl(tk) - qbase;
        }
        (void)nb;
        return;
    }

    const int jb = xcd * 8 + (a & 7), ti = a >> 3;

    if (wave >= W_STG && wave < W_POLL) {
        const int sg = wave - W_STG;
        PRIO(3);
        {
            const __amdgpu_buffer_rsrc_t wr = make_rsrc(p.w, KS * NN);
            const u32 wlane = (u32)((lane >> 4) * 16 * NN + (lane & 15) * 4);
            u32 id = (u32)(sg * NCWG + a * 8 + xcd);
            u32 r[16];
            auto load_w = [&](u32 t) {
                const u32 so = (t >> 6) * 64 * NN + ((t + ROT * p.rank) & 63u) * 64;
#pragma unroll
                for (int j = 0; j < 16; ++j) r[j] = __builtin_amdgcn_raw_buffer_load_b32(wr, wlane + j * NN, so, 0);
            };
            if (PSLEEP) __builtin_amdgcn_s_sleep(PSLEEP);
            if (id < 512) load_w(id);
            __syncthreads();
            const u32 call = rfl(*(const u32*)(lds + L_CALL));
            const u32 par = call & 1u;
            unsigned char* scratch = lds + L_B + sg * 4096;
            for (; id < 512; id += NCWG * NSTAGE) {
                const u32 kc = id >> 6, pj = (id + ROT * p.rank) & 63u;
                u32 T[4][4];
#pragma unroll
                for (int t4 = 0; t4 < 4; ++t4) {
                    const u32 r0 = r[4 * t4], r1 = r[4 * t4 + 1], r2 = r[4 * t4 + 2], r3 = r[4 * t4 + 3];
                    const u32 aa = __builtin_amdgcn_perm(r1, r0, 0x05010400u), bb = __builtin_amdgcn_perm(r1, r0, 0x07030602u);
                    const u32 cc = __builtin_amdgcn_perm(r3, r2, 0x05010400u), ee = __builtin_amdgcn_perm(r3, r2, 0x07030602u);
                    T[0][t4] = __builtin_amdgcn_perm(cc, aa, 0x05040100u);
                    T[1][t4] = __builtin_amdgcn_perm(cc, aa, 0x07060302u);
                    T[2][t4] = __builtin_amdgcn_perm(ee, bb, 0x05040100u);
                    T[3][t4] = __builtin_amdgcn_perm(ee, bb, 0x07060302u);
                }
                if (id + NCWG * NSTAGE < 512) load_w(id + NCWG * NSTAGE);
                u32 hdr = H_RAW;
                u32x4 o[4];
                u32 sum = 0;
#pragma unroll
                for (int q = 0; q < 4; ++q)
#pragma unroll
                    for (int t4 = 0; t4 < 4; ++t4) sum = __builtin_amdgcn_sad_u8(T[q][t4] & 0x7f7f7f7fu, 0u, sum);
                const int bs = ((int)wave_sum(sum) - 82944) >> 12;
                const u32 base = (u32)min(max(bs, 0), 96);
                const u32 basex = base * 0x01010101u;
                bool done5 = false;
                if (kc >= F5FROM) {
                    const u32 b16 = base + 16u, b16x = b16 * 0x01010101u;
                    u32 D[16], M[4] = {0u, 0u, 0u, 0u}, total = 0;
                    if (lane < 32) *(u32x4*)(scratch + 2048 + lane * 16) = (u32x4){0u, 0u, 0u, 0u};
#pragma unroll
                    for (int i = 0; i < 16; ++i) {
                        const u32 d = T[i >> 2][i & 3];
                        const u32 u = (d | 0x80808080u) - b16x;
                        const u32 low4 = u & 0x0f0f0f0fu;
                        D[i] = (d & 0x80808080u) | low4;
                        const u32 t = u ^ 0x80808080u;
                        const u32 lo = t & (t << 1) & (t << 2) & (t << 3) & 0x80808080u;
                        u32 fix = (d ^ (low4 + b16x)) & 0x70707070u & ~((lo >> 1) | (lo >> 2) | (lo >> 3));
                        M[i >> 2] |= __builtin_amdgcn_udot4(lo >> 7, 0x08040201u, 0u, false) << (4 * (i & 3));
                        u64 m = __ballot(fix != 0u);
                        while (m) {
                            if (fix != 0u) {
                                const u32 sh = (u32)__builtin_ctz(fix) & 24u;
                                const u32 idx = total + lanes_below(m);
                                const u32 ent = (u32)((i >> 2) * 1024 + (i & 3) * 4) + (u32)lane * 16u + (sh >> 3) + (((fix >> (sh + 4u)) & 7u) << 12);
                                if (idx < MAXFIX) *(unsigned short*)(scratch + idx * 2) = (unsigned short)ent;
                                fix &= ~(0x70u << sh);
                            }
                            total += (u32)__builtin_popcountll(m);
                            m = __ballot(fix != 0u);
                        }
                    }
                    u32 curq = 0, nsym = 0;
#pragma unroll
                    for (int q = 0; q < 4; ++q) {
                        const u32 p0 = (u32)(q * 1024) + (u32)lane * 16u;
                        const u32 lastp1 = M[q] ? p0 + 32u - (u32)__builtin_clz(M[q]) : 0u;
                        const u32 smax = scan_max(lastp1);
                        u32 cur = max(lane_before(smax), curq);
                        const u32 g = M[q] ? p0 + (u32)__builtin_ctz(M[q]) - cur : 0u;
                        const u32 lead = __umul24(g, 4370u) >> 16;
                        const u32 cnt = (u32)__builtin_popcount(M[q]);
                        const u32 incl = scan_add(lead + cnt);
                        u32 at = nsym + incl - cnt;
                        cur += 15u * lead;
                        u32 mx = M[q];
                        while (__ballot(mx != 0u)) {
                            if (mx != 0u) {
                                const u32 pp = p0 + (u32)__builtin_ctz(mx);
                                if (at < 800u)
                                    __hip_atomic_fetch_or((u32*)__builtin_assume_aligned(scratch + 2048 + (at >> 3) * 4, 4), (15u - (pp - cur)) << ((at & 7u) * 4u),
                                                          __ATOMIC_RELAXED, __HIP_MEMORY_SCOPE_WORKGROUP);
                                ++at;
                                cur = pp + 1u;
                                mx &= mx - 1u;
                            }
                        }
                        nsym += (u32)__builtin_amdgcn_readlane(incl, 63);
                        curq = max(curq, (u32)__builtin_amdgcn_readlane(smax, 63));
                    }
                    if (total <= MAXFIX && nsym <= 768u) {
                        const unsigned short* gs = (const unsigned short*)(scratch + 2048 + lane * 6);
                        const u32 g0 = ~((u32)gs[0] | ((u32)gs[1] << 16)), g1 = ~(u32)gs[2];
                        u32 PP[10];
#pragma unroll
                        for (int gq = 0; gq < 2; ++gq) {
                            const u32* E = D + 8 * gq;
                            PP[5 * gq + 0] = E[0] | ((E[5] & 0x07070707u) << 4);
                            PP[5 * gq + 1] = E[1] | ((E[6] & 0x07070707u) << 4);
                            PP[5 * gq + 2] = E[2] | ((E[7] & 0x07070707u) << 4);
                            PP[5 * gq + 3] = E[3] | ((E[5] & 0x08080808u) << 1) | ((E[6] & 0x08080808u) << 2) | ((E[7] & 0x08080808u) << 3);
                            PP[5 * gq + 4] = E[4] | ((E[5] & 0x80808080u) >> 3) | ((E[6] & 0x80808080u) >> 2) | ((E[7] & 0x80808080u) >> 1);
                        }
                        *(u32x2*)(scratch + 2560 + lane * 8) = (u32x2){PP[8], PP[9]};
                        *(u32*)(scratch + 2560 + 512 + lane * 4) = g0;
                        *(unsigned short*)(scratch + 2560 + 768 + lane * 2) = (unsigned short)g1;
                        o[0] = (u32x4){PP[0], PP[1], PP[2], PP[3]};
                        o[1] = (u32x4){PP[4], PP[5], PP[6], PP[7]};
                        o[2] = *(const u32x4*)(scratch + 2560 + lane * 16);
                        o[3] = *(const u32x4*)(scratch + lane * 16);
                        hdr = total | (b16 << 8) | H_F5;
                        done5 = true;
                    }
                }
                u32 P[4][3];
                u32 total = 0;
                if (!done5) {
#pragma unroll
                    for (int q = 0; q < 4; ++q) {
                        u32 D[4];
#pragma unroll
                        for (int t4 = 0; t4 < 4; ++t4) {
                            const u32 d = T[q][t4];
                            const u32 code = ((d | 0x80808080u) - basex) & 0x1f1f1f1fu;
                            D[t4] = (d & 0x80808080u) | code;
                            u32 fix = (d ^ (code + basex)) & 0x60606060u;
                            u64 m = __ballot(fix != 0u);
                            while (m) {
                                if (fix != 0u) {
                                    const u32 sh = (u32)__builtin_ctz(fix) & 24u;
                                    const u32 idx = total + lanes_below(m);
                                    const u32 ent = (u32)(q * 1024 + t4 * 4) + (u32)lane * 16u + (sh >> 3) + (((fix >> (sh + 4u)) & 7u) << 12);
                                    if (idx < MAXFIX) *(unsigned short*)(scratch + idx * 2) = (unsigned short)ent;
                                    fix &= ~(0x60u << sh);
                                }
                                total += (u32)__builtin_popcountll(m);
                                m = __ballot(fix != 0u);
                            }
                        }
                        P[q][0] = D[0] | ((D[3] & 0x03030303u) << 5);
                        P[q][1] = D[1] | ((D[3] & 0x0c0c0c0cu) << 3);
                        P[q][2] = D[2] | ((D[3] & 0x10101010u) << 1) | ((D[3] & 0x80808080u) >> 1);
                    }
                }
                if (done5) {
                } else if (total <= MAXFIX) {
                    hdr = total | (base << 8);
                    o[0] = (u32x4){P[0][0], P[0][1], P[0][2], P[1][0]};
                    o[1] = (u32x4){P[1][1], P[1][2], P[2][0], P[2][1]};
                    o[2] = (u32x4){P[2][2], P[3][0], P[3][1], P[3][2]};
                    o[3] = *(const u32x4*)(scratch + lane * 16);
                } else
                {
#pragma unroll
                    for (int q = 0; q < 4; ++q) o[q] = (u32x4){T[q][0], T[q][1], T[q][2], T[q][3]};
                }
                unsigned char* d = my + WS_W + par * W_PAR + (size_t)p.rank * (W_PAR / NR) + (size_t)(kc * 64 + pj) * 4096 + lane * 16;
#pragma unroll
                for (int j = 0; j < 4; ++j) st16_sys(d + j * 1024, o[j]);
                VM0();
                if (lane == 0) __hip_atomic_store((u64*)(my + WS_FW + par * FW_PAR) + pj * 8 + kc, (u64)call | (((u64)hdr | ((u64)LOC_OWN << 16)) << 32), __ATOMIC_RELAXED, __HIP_MEMORY_SCOPE_SYSTEM);
            }
            const __amdgpu_buffer_rsrc_t xr = make_rsrc(p.x + (size_t)p.rank * MS * KS, MS * KS);
            for (u32 t = (u32)((a * 8 + xcd) * NSTAGE + sg); t < NXB; t += NCWG * NSTAGE) {
                const u32 so = (t >> 2) * (32 * KS) + (t & 3) * 128;
                u32x4 o[4];
#pragma unroll
                for (int j = 0; j < 4; ++j) o[j] = __builtin_amdgcn_raw_buffer_load_b128(xr, xlane + (j >> 1) * 64 + (j & 1) * 16, so, 0);
                unsigned char* d = my + WS_X + par * X_PAR + (size_t)p.rank * (X_PAR / NR) + (size_t)t * 4096 + lane * 16;
#pragma unroll
                for (int j = 0; j < 4; ++j) st16_sys(d + j * 1024, o[j]);
                VM0();
                if (lane == 0) __hip_atomic_store((u32*)(my + WS_FX + par * FX_PAR) + p.rank * 32 + (t >> 5) * 256 + (t & 31), call, __ATOMIC_RELAXED, __HIP_MEMORY_SCOPE_SYSTEM);
            }
            if (sg == 0) STAMP(6);
        }
        const u32 call = rfl(*(const u32*)(lds + L_CALL));
        const u32 par = call & 1u;
        const __amdgpu_buffer_rsrc_t rw = make_rsrc(my + WS_W + par * W_PAR, (u32)W_PAR);
        const __amdgpu_buffer_rsrc_t rall = make_rsrc(my, (u32)WS_END);
        const u32 lds0 = (u32)(size_t)(__attribute__((address_space(3))) unsigned char*)lds;
        const u32 b_off = (u32)(jb * 4096);
        for (u32 n = (u32)sg; n < 64; n += NSTAGE) {
            u32 t;
            PRIO(0);
            u32 ent;
            while (true) {
                u32 v0, v1;
                asm volatile("ds_read_b32 %0, %2\n\tds_read_b32 %1, %3\n\ts_waitcnt lgkmcnt(0)"
                             : "=&v"(v0), "=&v"(v1) : "v"(lds0 + (u32)L_TAIL), "v"(lds0 + (u32)L_ORD + n * 4u) : "memory");
                t = rfl(v0); ent = rfl(v1);
                if (t > n) break;
                __builtin_amdgcn_s_sleep(SSLEEP);
            }
            PRIO(3);
            if (t == DEAD) break;
            const u32 u = ent & 63u, hdr = (ent >> 6) & 0xffffu, loc = ent >> 22;
            const u32 so = u * (64 * 4096) + b_off;
            const bool raw = is_raw(hdr), f5 = (hdr & H_F5) != 0u;
            const u32 lo = loc == LOC_OWN ? (u32)(WS_W + par * W_PAR) + so + 3072u
                                          : (u32)(WS_FL + par * FL_PAR) + FLIDX((loc + ROT * (u >> 3)) & 63u, u >> 3, loc >> 6) * 128u;
            u32x4 v[4];
            v[0] = __builtin_amdgcn_raw_buffer_load_b128(rw, lane * 16, so, 17);
            v[1] = __builtin_amdgcn_raw_buffer_load_b128(rw, lane * 16 + 1024, so, 17);
            const u32x2 w2 = __builtin_amdgcn_raw_buffer_load_b64(rw, f5 ? lane * 8 + 2048 : lane * 16 + 2048, so, 17);
            const u32x2 w3 = __builtin_amdgcn_raw_buffer_load_b64(rw, f5 ? lane * 4 + 2560 : lane * 16 + 2056, so, 17);
            const u32 g1 = __builtin_amdgcn_raw_buffer_load_b16(rw, lane * 2 + 2816, so, 17);
            u32 e = __builtin_amdgcn_raw_buffer_load_b16(rall, lane * 2, lo, 17);
            v[2] = (u32x4){w2.x, w2.y, w3.x, w3.y};
            const u32 g0 = w3.x;
            if (raw) v[3] = __builtin_amdgcn_raw_buffer_load_b128(rw, lane * 16 + 3072, so, 17);
            if (n >= NSLOT) {
                bool dead = false;
                while (true) {
                    const u32 c = lds_ld(lds, L_CONS + (lane & 7) * 4);
                    if (__all(c > n - NSLOT)) break;
                    if (rfl(lds_ld(lds, L_TAIL)) == DEAD) { dead = true; break; }
                    __builtin_amdgcn_s_sleep(1);
                }
                if (dead) break;
                CFENCE();
            }
            unsigned char* slot = lds + L_B + (n & (NSLOT - 1)) * 4096;
            if (raw) {
#pragma unroll
                for (int q = 0; q < 4; ++q) *(u32x4*)(slot + q * 1024 + lane * 16) = v[q];
            } else if (f5) {
                const u32 basex = ((hdr >> 8) & 127u) * 0x01010101u;
                const u32 PP[10] = {v[0].x, v[0].y, v[0].z, v[0].w, v[1].x, v[1].y, v[1].z, v[1].w, v[2].x, v[2].y};
                u32 D[16];
#pragma unroll
                for (int gq = 0; gq < 2; ++gq) {
                    const u32* Q = PP + 5 * gq;
#pragma unroll
                    for (int k = 0; k < 5; ++k) D[8 * gq + k] = (Q[k] & 0x8f8f8f8fu) + basex;
                    D[8 * gq + 5] = (((Q[0] >> 4) & 0x07070707u) | ((Q[3] >> 1) & 0x08080808u) | ((Q[4] << 3) & 0x80808080u)) + basex;
                    D[8 * gq + 6] = (((Q[1] >> 4) & 0x07070707u) | ((Q[3] >> 2) & 0x08080808u) | ((Q[4] << 2) & 0x80808080u)) + basex;
                    D[8 * gq + 7] = (((Q[2] >> 4) & 0x07070707u) | ((Q[3] >> 3) & 0x08080808u) | ((Q[4] << 1) & 0x80808080u)) + basex;
                }
#pragma unroll
                for (int q = 0; q < 4; ++q) *(u32x4*)(slot + q * 1024 + lane * 16) = (u32x4){D[4 * q], D[4 * q + 1], D[4 * q + 2], D[4 * q + 3]};
                u32 sy[12], adv = 0;
#pragma unroll
                for (int j = 0; j < 12; ++j) {
                    sy[j] = ((j < 8 ? g0 : g1) >> (4 * (j & 7))) & 15u;
                    adv += min(sy[j] + 1u, 15u);
                }
                u32 cur = scan_add(adv) - adv - 1u;
#pragma unroll
                for (int j = 0; j < 12; ++j) {
                    cur += min(sy[j] + 1u, 15u);
                    __hip_atomic_fetch_sub((u32*)__builtin_assume_aligned(slot + (cur & 0xffcu), 4), sy[j] != 15u ? 16u << ((cur & 3u) * 8u) : 0u,
                                           __ATOMIC_RELAXED, __HIP_MEMORY_SCOPE_WORKGROUP);
                }
            } else {
                const u32 basex = ((hdr >> 8) & 127u) * 0x01010101u;
                const u32 P[4][3] = {{v[0].x, v[0].y, v[0].z}, {v[0].w, v[1].x, v[1].y}, {v[1].z, v[1].w, v[2].x}, {v[2].y, v[2].z, v[2].w}};
#pragma unroll
                for (int q = 0; q < 4; ++q) {
                    const u32 d3 = ((P[q][0] >> 5) & 0x03030303u) | ((P[q][1] >> 3) & 0x0c0c0c0cu) | ((P[q][2] >> 1) & 0x10101010u) | ((P[q][2] << 1) & 0x80808080u);
                    *(u32x4*)(slot + q * 1024 + lane * 16) =
                        (u32x4){(P[q][0] & 0x9f9f9f9fu) + basex, (P[q][1] & 0x9f9f9f9fu) + basex, (P[q][2] & 0x9f9f9f9fu) + basex, d3 + basex};
                }
            }
            if (!raw) {
                const u32 cnt = hdr & 255u;
                for (u32 e0 = 0; e0 < cnt; e0 += 64) {
                    if (e0) e = __builtin_amdgcn_raw_buffer_load_b16(rall, lane * 2 + e0 * 2, lo, 17);
                    if (e0 + (u32)lane < cnt)
                        __hip_atomic_fetch_xor((u32*)(slot + (e & 0xffcu)), ((e >> 12) & 7u) << ((e & 3u) * 8u + 4u), __ATOMIC_RELAXED, __HIP_MEMORY_SCOPE_WORKGROUP);
                }
            }
            CFENCE();
            if (lane == 0) __hip_atomic_fetch_or((u32*)__builtin_assume_aligned(lds + L_STG + (n >> 5) * 4, 4), 1u << (n & 31u), __ATOMIC_RELAXED, __HIP_MEMORY_SCOPE_WORKGROUP);
        }
        return;
    }

    __syncthreads();
    const u32 call = rfl(*(const u32*)(lds + L_CALL));
    const u32 par = call & 1u;

    if (wave == W_POLL) {
        PRIO(2);
        const u64* fwp = (lane >> 3) == p.rank ? (const u64*)(my + WS_FW + par * FW_PAR) + jb * 8 + (lane & 7)
                                               : (const u64*)(my + WS_FL + par * FL_PAR + (size_t)FLIDX((u32)jb, (u32)(lane >> 3), (u32)(lane & 7)) * 128 + 120);
        const __amdgpu_buffer_rsrc_t rf = make_rsrc(my + WS_FX + par * FX_PAR + ti * 1024, 1024);
        const u64* idle = (const u64*)(my + WS_FW + par * FW_PAR) + jb * 8;
        u64 seen = 0, xok = 0;
        u32 tail = 0, spins = 0;
        bool dead = false;
        const u64 t0 = __builtin_amdgcn_s_memrealtime();
        auto poll = [&]() -> u64 {
            return __hip_atomic_load(((seen >> lane) & 1ull) ? idle : fwp, __ATOMIC_RELAXED, __HIP_MEMORY_SCOPE_SYSTEM);
        };
        auto take = [&](u64 f) {
            if (xok != ~0ull) {
                const u32x4 g = __builtin_amdgcn_raw_buffer_load_b128(rf, lane * 16, 0, 17);
                u64 mx = __ballot(g.x >= call && g.y >= call && g.z >= call && g.w >= call);
                mx &= (mx >> 4) & 0x0f0f0f0f0f0f0f0full; mx &= mx >> 2; mx &= mx >> 1; mx &= 0x0101010101010101ull;
                xok = mx * 0xffull;
            }
            const u64 fresh = __ballot((u32)f >= call) & xok & ~seen;
            if (fresh) {
                if ((fresh >> lane) & 1) {
                    const u32 pos = tail + lanes_below(fresh);
                    *(u32*)(lds + L_ORD + pos * 4) = (u32)lane | ((u32)(f >> 32) << 6);
                }
                tail += (u32)__builtin_popcountll(fresh);
                seen |= fresh;
                CFENCE();
                if (lane == 0) lds_st(lds, L_TAIL, tail);
            } else {
                if ((++spins & 15u) == 0u && timed_out(tmo, call, t0)) {
                    VM0();
                    if (lane == 0) { lds_st(lds, L_BAD, 2u); lds_st(lds, L_TAIL, DEAD); }
                    dead = true;
                    return;
                }
                __builtin_amdgcn_s_sleep(POLLSLEEP);
            }
            CFENCE();
        };
        while (!dead && seen != ~0ull) take(poll());
        if (dead) return;
        {
            const u32 tv = __hip_atomic_load(tmo, __ATOMIC_RELAXED, __HIP_MEMORY_SCOPE_AGENT);
            if (lane == 0) lds_st(lds, L_BAD, tv == call ? 2u : 1u);
        }
        return;
    }

    const int wv = wave;
    const __amdgpu_buffer_rsrc_t rx = make_rsrc(my + WS_X + par * X_PAR, (u32)X_PAR);
    const float scale = p.sx[0] * p.sw[0];
    f32x16 acc0 = {}, acc1 = {};
    const u32 b_rd = (u32)(((lane >> 4) & 1) * 2048 + (lane >> 5) * 512 + (lane & 15) * 16);
    const u32 a_off = (u32)((ti * 8 + wv) * 8 * 2048 + lane * 16);
    const u32 lds0 = (u32)(size_t)(__attribute__((address_space(3))) unsigned char*)lds;
    int iters = 0;
    u32 n = 0, t = 0, ord = 0;
    u64 staged = 0;
    auto snapshot = [&]() {
        u32x4 c;
        asm volatile("ds_read_b128 %0, %2\n\tds_read_b32 %1, %3\n\ts_waitcnt lgkmcnt(0)"
                     : "=&v"(c), "=&v"(ord) : "v"(lds0 + (u32)L_STG), "v"(lds0 + (u32)L_ORD + (u32)lane * 4u) : "memory");
        t = rfl(c.z);
        staged = ((u64)rfl(c.y) << 32) | rfl(c.x);
    };
    struct BF { u32x4 b00, b01, b10, b11; };
    auto read_b = [&](u32 i) {
        const unsigned char* bb = lds + L_B + (i & (NSLOT - 1)) * 4096 + b_rd;
        BF f;
        f.b00 = *(const u32x4*)(bb); f.b01 = *(const u32x4*)(bb + 256);
        f.b10 = *(const u32x4*)(bb + 1024); f.b11 = *(const u32x4*)(bb + 1280);
        return f;
    };
    auto mma = [&](const u32x4 (&av)[2], const BF& f) {
        const i32x8 af = {(int)av[0].x, (int)av[0].y, (int)av[0].z, (int)av[0].w, (int)av[1].x, (int)av[1].y, (int)av[1].z, (int)av[1].w};
        const i32x8 bf0 = {(int)f.b00.x, (int)f.b00.y, (int)f.b00.z, (int)f.b00.w, (int)f.b01.x, (int)f.b01.y, (int)f.b01.z, (int)f.b01.w};
        const i32x8 bf1 = {(int)f.b10.x, (int)f.b10.y, (int)f.b10.z, (int)f.b10.w, (int)f.b11.x, (int)f.b11.y, (int)f.b11.z, (int)f.b11.w};
        acc0 = __builtin_amdgcn_mfma_scale_f32_32x32x64_f8f6f4(af, bf0, acc0, 0, 1, 0, 0, 0, 0);
        acc1 = __builtin_amdgcn_mfma_scale_f32_32x32x64_f8f6f4(af, bf1, acc1, 0, 1, 0, 0, 0, 0);
    };
    while (n < 64) {
        if (t <= n) {
            PRIO(0);
            while (true) { snapshot(); if (t > n) break; __builtin_amdgcn_s_sleep(MSLEEP); }
            PRIO(1);
        }
        if (t == DEAD) break;
        ++iters;
        const u32 m = min(t - n, 4u);
        u32x4 av[4][2];
#pragma unroll
        for (int j = 0; j < 4; ++j) {
            const u32 u = (u32)__builtin_amdgcn_readlane(ord, n + min((u32)j, m - 1u)) & 63u;
            const u32 so = (u >> 3) * (u32)(X_PAR / NR) + (u & 7) * 2048;
#pragma unroll
            for (int s = 0; s < 2; ++s) av[j][s] = __builtin_amdgcn_raw_buffer_load_b128(rx, a_off + s * 1024, so, 17);
        }
        bool dead = false;
        u32 j = 0;
        auto ready = [&](u32 i) { return ((staged >> i) & 1ull) != 0ull; };
        auto handle = [&](auto JC) {
            constexpr int J = decltype(JC)::value;
            while (!ready(n + J)) {
                snapshot();
                if (t == DEAD) { dead = true; return; }
                if (!ready(n + J)) __builtin_amdgcn_s_sleep(MSLEEP);
            }
            BF f0 = read_b(n + J);
            asm volatile("" : "+v"(f0.b00), "+v"(f0.b01), "+v"(f0.b10), "+v"(f0.b11));
            mma(av[J], f0);
            j = J + 1;
        };
        if (j == 0u) handle(IC<0>{});
        if (!dead && j == 1u && j < m) handle(IC<1>{});
        if (!dead && j == 2u && j < m) handle(IC<2>{});
        if (!dead && j == 3u && j < m) handle(IC<3>{});
        if (dead) break;
        n += m;
        LGKM0();
        if (lane == 0) lds_st(lds, L_CONS + wv * 4, n);
        snapshot();
    }

    u32 bad;
    while ((bad = rfl(lds_ld(lds, L_BAD))) == 0u) __builtin_amdgcn_s_sleep(1);
    const float s = bad == 2u ? __builtin_nanf("") : scale;
    float* o = p.out + (size_t)(ti * 256 + wv * 32 + (lane >> 5) * 4 + ((lane >> 4) & 1)) * NN + jb * 64 + (lane & 15) * 4;
#pragma unroll
    for (int r = 0; r < 16; r += 2) {
        const auto s0 = __builtin_amdgcn_permlane16_swap(__float_as_uint(acc0[r] * s), __float_as_uint(acc0[r + 1] * s), false, false);
        const auto s1 = __builtin_amdgcn_permlane16_swap(__float_as_uint(acc1[r] * s), __float_as_uint(acc1[r + 1] * s), false, false);
        st16_sys(o + (size_t)((r & 3) + 8 * (r >> 2)) * NN, (u32x4){s0[0], s1[0], s0[1], s1[1]});
    }
}

}

extern "C" void kernel_launch(int rank, int nranks, ncclComm_t comm, hipStream_t stream,
                              void* const* d_in, const int* in_sizes, int n_in,
                              void* d_out, int out_size,
                              const int* in_dtypes, int out_dtype,
                              void* const* d_out_all,
                              void* const* d_ws_all, size_t ws_bytes) {
    (void)comm; (void)in_sizes; (void)n_in; (void)out_size; (void)in_dtypes; (void)out_dtype; (void)d_out_all;
    if (nranks != NR || ws_bytes < WS_END || (unsigned)rank >= (unsigned)NR) {
        fprintf(stderr, "unexpected configuration\n");
        return;
    }
    Params p;
    p.x = (const unsigned char*)d_in[0];
    p.w = (const unsigned char*)d_in[1];
    p.sx = (const float*)d_in[2];
    p.sw = (const float*)d_in[3];
    p.out = (float*)d_out;
    for (int i = 0; i < NR; ++i) p.ws[i] = (unsigned char*)d_ws_all[i];
    p.rank = rank;
    p.pad = 0;
    hipLaunchKernelGGL(gemm_ag_fused, dim3(NWG), dim3(NTHREADS), 0, stream, p);
    HIP_CHECK(hipGetLastError());
}
```

```cpp
#include <hip/hip_runtime.h>
#include <rccl/rccl.h>
#include <cstdio>

#define HIP_CHECK(expr) do { hipError_t _e = (expr); if (_e != hipSuccess) { \
    fprintf(stderr, "HIP error %s at %s:%d\n", hipGetErrorString(_e), __FILE__, __LINE__); \
    return; } } while (0)

namespace {

typedef unsigned u32;
typedef unsigned long long u64;
typedef u32 u32x2 __attribute__((ext_vector_type(2)));
typedef u32 u32x4 __attribute__((ext_vector_type(4)));
typedef int i32x8 __attribute__((ext_vector_type(8)));
typedef float f32x16 __attribute__((ext_vector_type(16)));

#define NSEND_ 5
#define F5FROM 1
#define STAG0 32u
#define STAGD 80

constexpr int NR = 8;
constexpr int NN = 4096;
constexpr int KS = 512;
constexpr int MS = 512;
constexpr int NWG = 256;
constexpr int NCWG = 128;
constexpr int NWORK = 8;
constexpr int NSTAGE = 7;
constexpr int W_STG  = NWORK;
constexpr int W_POLL = W_STG + NSTAGE;
constexpr int NTHREADS = 64 * (W_POLL + 1);
constexpr int NSEND = NSEND_;
constexpr int WPD = (NWG - NCWG) * NSEND / (NR - 1);
constexpr u32 NXB = 64;
constexpr u32 NBLK = NXB + 512;

constexpr size_t WS_W   = 0;
constexpr size_t W_PAR  = (size_t)NR * 8 * 64 * 4096;
constexpr size_t WS_X   = 2 * W_PAR;
constexpr size_t X_PAR  = (size_t)NR * 16 * 8 * 2048;
constexpr size_t WS_FL  = WS_X + 2 * X_PAR;
constexpr size_t FL_PAR = 64 * 64 * 128;
constexpr size_t WS_FW  = WS_FL + 2 * FL_PAR;
constexpr size_t FW_PAR = 64 * 8 * 8;
constexpr size_t WS_FX  = WS_FW + 2 * FW_PAR;
constexpr size_t FX_PAR = 2 * 8 * 32 * 4;
constexpr size_t WS_Q   = WS_FX + 2 * FX_PAR;
constexpr size_t WS_CNT = WS_Q + NR * 128;
constexpr size_t WS_TMO = WS_CNT + NWG * 4;
constexpr size_t WS_END = WS_TMO + 128;

constexpr u64 DEADLINE_TICKS = 30000000ull;

constexpr u32 H_RAW = 127u << 8;
constexpr u32 H_F5 = 1u << 15;
constexpr u32 MAXFIX = 255;
constexpr u32 MAXRIDE = 60;
constexpr u32 LOC_OWN = 1023;

constexpr int NSLOT  = 32;
constexpr int L_B    = 0;
constexpr int L_ORD  = NSLOT * 4096;
constexpr int L_STG  = L_ORD + 256;
constexpr int L_TAIL = L_STG + 8;
constexpr int L_BAD  = L_STG + 12;
constexpr int L_CALL = L_STG + 16;
constexpr int L_CONS = L_CALL + 16;
constexpr int L_TAB  = L_CONS + 32;
constexpr int L_BYTES = L_TAB + 1024;
constexpr u32 DEAD = 0x80000000u;

struct Params {
    const unsigned char* x;
    const unsigned char* w;
    const float* sx;
    const float* sw;
    float* out;
    unsigned char* ws[NR];
    int rank;
    int pad;
};

__device__ __forceinline__ void st16_sys(void* p, u32x4 v) {
    asm volatile("global_store_dwordx4 %0, %1, off sc0 sc1\n\ts_nop 1" :: "v"(p), "v"(v) : "memory");
}
constexpr int SYS = 17;
__device__ __forceinline__ __amdgpu_buffer_rsrc_t make_rsrc(const void* base, u32 bytes) {
    return __builtin_amdgcn_make_buffer_rsrc(const_cast<void*>(base), 0, bytes, 0x00020000);
}
template <int V> struct IC { static constexpr int value = V; };
__device__ __forceinline__ u32 rfl(u32 v) { return __builtin_amdgcn_readfirstlane(v); }
__device__ __forceinline__ u32 lds_acquire(const unsigned char* lds, int off) {
    return __hip_atomic_load((const u32*)(lds + off), __ATOMIC_ACQUIRE, __HIP_MEMORY_SCOPE_WORKGROUP);
}
__device__ __forceinline__ void lds_release(unsigned char* lds, int off, u32 v) {
    __hip_atomic_store((u32*)(lds + off), v, __ATOMIC_RELEASE, __HIP_MEMORY_SCOPE_WORKGROUP);
}
__device__ __forceinline__ u32* lds_word(unsigned char* p) { return (u32*)__builtin_assume_aligned(p, 4); }
__device__ __forceinline__ u32 lanes_below(u64 m) {
    return __builtin_amdgcn_mbcnt_hi((u32)(m >> 32), __builtin_amdgcn_mbcnt_lo((u32)m, 0u));
}
__device__ __forceinline__ bool is_raw(u32 h) { return ((h >> 8) & 127u) == 127u; }
__device__ __forceinline__ u32 scan_add(u32 v) {
    v += __builtin_amdgcn_update_dpp(0u, v, 0x111, 0xf, 0xf, false);
    v += __builtin_amdgcn_update_dpp(0u, v, 0x112, 0xf, 0xf, false);
    v += __builtin_amdgcn_update_dpp(0u, v, 0x114, 0xf, 0xf, false);
    v += __builtin_amdgcn_update_dpp(0u, v, 0x118, 0xf, 0xf, false);
    v += __builtin_amdgcn_update_dpp(0u, v, 0x142, 0xa, 0xf, false);
    v += __builtin_amdgcn_update_dpp(0u, v, 0x143, 0xc, 0xf, false);
    return v;
}
__device__ __forceinline__ u32 scan_max(u32 v) {
    v = max(v, (u32)__builtin_amdgcn_update_dpp(0u, v, 0x111, 0xf, 0xf, false));
    v = max(v, (u32)__builtin_amdgcn_update_dpp(0u, v, 0x112, 0xf, 0xf, false));
    v = max(v, (u32)__builtin_amdgcn_update_dpp(0u, v, 0x114, 0xf, 0xf, false));
    v = max(v, (u32)__builtin_amdgcn_update_dpp(0u, v, 0x118, 0xf, 0xf, false));
    v = max(v, (u32)__builtin_amdgcn_update_dpp(0u, v, 0x142, 0xa, 0xf, false));
    v = max(v, (u32)__builtin_amdgcn_update_dpp(0u, v, 0x143, 0xc, 0xf, false));
    return v;
}
__device__ __forceinline__ u32 lane_before(u32 v) {
    u32 r = __builtin_amdgcn_update_dpp(0u, v, 0x138, 0xf, 0xf, false);
    asm volatile("" : "+v"(r));
    return r;
}
#define PRIO(x) __builtin_amdgcn_s_setprio(x)
#define VM0() asm volatile("s_waitcnt vmcnt(0)" ::: "memory")
#define CFENCE() asm volatile("" ::: "memory")

__device__ __forceinline__ bool timed_out(u32* tmo, u32 call, u64 t0) {
    if (__hip_atomic_load(tmo, __ATOMIC_RELAXED, __HIP_MEMORY_SCOPE_AGENT) == call
        || __builtin_amdgcn_s_memrealtime() - t0 > DEADLINE_TICKS) {
        __hip_atomic_store(tmo, call, __ATOMIC_RELAXED, __HIP_MEMORY_SCOPE_AGENT);
        return true;
    }
    return false;
}

__device__ __forceinline__ u32 build_list(const u32 (&F)[16], unsigned char* list, unsigned char* dump, int lane) {
    u32 M[2] = {0u, 0u};
#pragma unroll
    for (int i = 0; i < 16; ++i) {
        const u32 nz = (F[i] | (F[i] << 1) | (F[i] << 2)) & 0x40404040u;
        M[i >> 3] |= __builtin_amdgcn_udot4(nz >> 6, 0x08040201u, 0u, false) << (4 * (i & 7));
    }
#pragma unroll
    for (int q = 0; q < 4; ++q) *(u32x4*)(dump + q * 1024 + lane * 16) = (u32x4){F[4 * q], F[4 * q + 1], F[4 * q + 2], F[4 * q + 3]};
    const u32 cnt = (u32)__builtin_popcount(M[0]) + (u32)__builtin_popcount(M[1]);
    const u32 incl = scan_add(cnt);
    u32 idx = incl - cnt;
    u64 m = ((u64)M[1] << 32) | M[0];
    while (__ballot(m != 0ull)) {
        if (m != 0ull) {
            const u32 i = (u32)__builtin_ctzll(m);
            const u32 pos = (i >> 4) * 1024u + (u32)lane * 16u + (i & 15u);
            if (idx < MAXFIX) *(unsigned short*)(list + idx * 2) = (unsigned short)(pos | (((u32)dump[pos] >> 4) << 12));
            ++idx;
            m &= m - 1ull;
        }
    }
    return (u32)__builtin_amdgcn_readlane(incl, 63);
}

__device__ __forceinline__ u32 pack_block(const u32 (&T)[4][4], bool try5, unsigned char* scratch, unsigned char* dump, const u32* gaptab, int lane, u32x4 (&o)[4]) {
    u32 sum = 0;
#pragma unroll
    for (int i = 0; i < 16; ++i) sum = __builtin_amdgcn_sad_u8(T[i >> 2][i & 3] & 0x7f7f7f7fu, 0u, sum);
    const int bs = ((int)__builtin_amdgcn_readlane(scan_add(sum), 63) - 82944) >> 12;
    const u32 base = (u32)min(max(bs, 0), 96);
    if (try5) {
        const u32 b16 = base + 16u, b16x = b16 * 0x01010101u;
        u32 D[16], F[16], M[4] = {0u, 0u, 0u, 0u};
        if (lane < 32) *(u32x4*)(scratch + 2048 + lane * 16) = (u32x4){0u, 0u, 0u, 0u};
#pragma unroll
        for (int i = 0; i < 16; ++i) {
            const u32 d = T[i >> 2][i & 3];
            const u32 u = (d | 0x80808080u) - b16x;
            const u32 low4 = u & 0x0f0f0f0fu;
            D[i] = (d & 0x80808080u) | low4;
            const u32 t = u ^ 0x80808080u;
            const u32 lo = t & (t << 1) & (t << 2) & (t << 3) & 0x80808080u;
            M[i >> 2] |= __builtin_amdgcn_udot4(lo >> 7, 0x08040201u, 0u, false) << (4 * (i & 3));
            F[i] = (d ^ (low4 + b16x)) & 0x70707070u & ~((lo >> 1) | (lo >> 2) | (lo >> 3));
        }
        u32 smax[4], first[4], lead[4], cnt[4];
#pragma unroll
        for (int q = 0; q < 4; ++q) smax[q] = scan_max(M[q] ? (u32)(q * 1024) + (u32)lane * 16u + 32u - (u32)__builtin_clz(M[q]) : 0u);
        u32 curq = 0;
#pragma unroll
        for (int q = 0; q < 4; ++q) {
            const u32 prev = max(lane_before(smax[q]), curq);
            const u32 g = M[q] ? (u32)(q * 1024) + (u32)lane * 16u + (u32)__builtin_ctz(M[q]) - prev : 0u;
            lead[q] = __umul24(g, 4370u) >> 16;
            first[q] = 15u + 15u * lead[q] - g;
            cnt[q] = (u32)__builtin_popcount(M[q]);
            curq = max(curq, (u32)__builtin_amdgcn_readlane(smax[q], 63));
        }
        const u32 i01 = scan_add((lead[0] + cnt[0]) | ((lead[1] + cnt[1]) << 16)), i23 = scan_add((lead[2] + cnt[2]) | ((lead[3] + cnt[3]) << 16));
        const u32 t01 = (u32)__builtin_amdgcn_readlane(i01, 63), t23 = (u32)__builtin_amdgcn_readlane(i23, 63);
        const u32 incl[4] = {i01 & 0xffffu, (t01 & 0xffffu) + (i01 >> 16), (t01 & 0xffffu) + (t01 >> 16) + (i23 & 0xffffu),
                             (t01 & 0xffffu) + (t01 >> 16) + (t23 & 0xffffu) + (i23 >> 16)};
        const u32 nsym = (t01 & 0xffffu) + (t01 >> 16) + (t23 & 0xffffu) + (t23 >> 16);
        if (nsym <= 768u) {
#pragma unroll
            for (int q = 0; q < 4; ++q) {
                const u32 lo = M[q] & 255u, hi = M[q] >> 8, na = (u32)__builtin_popcount(lo);
                u64 X = (u64)first[q] | ((u64)gaptab[lo] << 4) | ((u64)gaptab[hi] << (4u * na + 4u));
                if (lo && hi) X |= (u64)(15u - (8u + (u32)__builtin_ctz(hi) - (32u - (u32)__builtin_clz(lo)))) << (4u * na);
                if (M[q]) {
                    const u32 at = incl[q] - cnt[q], sh = (at & 7u) * 4u, x0 = (u32)X, x1 = (u32)(X >> 32);
                    u32* G = lds_word(scratch + 2048 + (at >> 3) * 4);
                    __hip_atomic_fetch_or(G, x0 << sh, __ATOMIC_RELAXED, __HIP_MEMORY_SCOPE_WORKGROUP);
                    __hip_atomic_fetch_or(G + 1, sh ? __builtin_amdgcn_alignbit(x1, x0, 32u - sh) : x1, __ATOMIC_RELAXED, __HIP_MEMORY_SCOPE_WORKGROUP);
                    __hip_atomic_fetch_or(G + 2, sh ? x1 >> (32u - sh) : 0u, __ATOMIC_RELAXED, __HIP_MEMORY_SCOPE_WORKGROUP);
                }
            }
            const u32 total = build_list(F, scratch, dump, lane);
            if (total <= MAXFIX) {
                const unsigned short* gs = (const unsigned short*)(scratch + 2048 + lane * 6);
                const u32 g0 = ~((u32)gs[0] | ((u32)gs[1] << 16)), g1 = ~(u32)gs[2];
                u32 P[10];
#pragma unroll
                for (int h = 0; h < 2; ++h) {
                    const u32* E = D + 8 * h;
                    P[5 * h + 0] = E[0] | ((E[5] & 0x07070707u) << 4);
                    P[5 * h + 1] = E[1] | ((E[6] & 0x07070707u) << 4);
                    P[5 * h + 2] = E[2] | ((E[7] & 0x07070707u) << 4);
                    P[5 * h + 3] = E[3] | ((E[5] & 0x08080808u) << 1) | ((E[6] & 0x08080808u) << 2) | ((E[7] & 0x08080808u) << 3);
                    P[5 * h + 4] = E[4] | ((E[5] & 0x80808080u) >> 3) | ((E[6] & 0x80808080u) >> 2) | ((E[7] & 0x80808080u) >> 1);
                }
                *(u32x2*)(scratch + 2560 + lane * 8) = (u32x2){P[8], P[9]};
                *(u32*)(scratch + 2560 + 512 + lane * 4) = g0;
                *(unsigned short*)(scratch + 2560 + 768 + lane * 2) = (unsigned short)g1;
                o[0] = (u32x4){P[0], P[1], P[2], P[3]};
                o[1] = (u32x4){P[4], P[5], P[6], P[7]};
                o[2] = *(const u32x4*)(scratch + 2560 + lane * 16);
                o[3] = *(const u32x4*)(scratch + lane * 16);
                return total | (b16 << 8) | H_F5;
            }
        }
    }
    const u32 basex = base * 0x01010101u;
    u32 D[16], F[16];
#pragma unroll
    for (int i = 0; i < 16; ++i) {
        const u32 d = T[i >> 2][i & 3];
        const u32 code = ((d | 0x80808080u) - basex) & 0x1f1f1f1fu;
        D[i] = (d & 0x80808080u) | code;
        F[i] = (d ^ (code + basex)) & 0x60606060u;
    }
    const u32 total = build_list(F, scratch, dump, lane);
    if (total <= MAXFIX) {
        u32 P[4][3];
#pragma unroll
        for (int q = 0; q < 4; ++q) {
            P[q][0] = D[4 * q] | ((D[4 * q + 3] & 0x03030303u) << 5);
            P[q][1] = D[4 * q + 1] | ((D[4 * q + 3] & 0x0c0c0c0cu) << 3);
            P[q][2] = D[4 * q + 2] | ((D[4 * q + 3] & 0x10101010u) << 1) | ((D[4 * q + 3] & 0x80808080u) >> 1);
        }
        o[0] = (u32x4){P[0][0], P[0][1], P[0][2], P[1][0]};
        o[1] = (u32x4){P[1][1], P[1][2], P[2][0], P[2][1]};
        o[2] = (u32x4){P[2][2], P[3][0], P[3][1], P[3][2]};
        o[3] = *(const u32x4*)(scratch + lane * 16);
        return total | (base << 8);
    }
#pragma unroll
    for (int q = 0; q < 4; ++q) o[q] = (u32x4){T[q][0], T[q][1], T[q][2], T[q][3]};
    return H_RAW;
}

__global__ void __launch_bounds__(NTHREADS, NTHREADS / 256) gemm_ag_fused(Params p) {
    __shared__ __attribute__((aligned(16))) unsigned char lds[L_BYTES];
    const int tid = threadIdx.x;
    const int lane = tid & 63;
    const int wave = rfl(tid >> 6);
    const int b = blockIdx.x;
    const int xcd = b & 7, a = b >> 3;
    unsigned char* my = p.ws[p.rank];
    u32* tmo = (u32*)(my + WS_TMO);
    const u32 lds0 = (u32)(size_t)(__attribute__((address_space(3))) unsigned char*)lds;

    if (tid == 0) {
        u32* cnt = (u32*)(my + WS_CNT) + b;
        const u32 c = __hip_atomic_load(cnt, __ATOMIC_RELAXED, __HIP_MEMORY_SCOPE_AGENT) + 1u;
        __hip_atomic_store(cnt, c, __ATOMIC_RELAXED, __HIP_MEMORY_SCOPE_AGENT);
        *(u32*)(lds + L_CALL) = c;
    }
    if (wave == 1) {
        if (lane < 4) *(u32*)(lds + L_STG + lane * 4) = 0u;
        if (lane < 8) *(u32*)(lds + L_CONS + lane * 4) = 0u;
    }
    if (a < NCWG / 8 && tid >= 128 && tid < 384) {
        const u32 x = (u32)tid - 128u;
        u32 v = 0, k = 0;
        int prev = -1;
#pragma unroll
        for (int bit = 0; bit < 8; ++bit)
            if ((x >> bit) & 1u) {
                if (prev >= 0) { v |= (16u - (u32)(bit - prev)) << (4u * k); ++k; }
                prev = bit;
            }
        *(u32*)(lds + L_TAB + x * 4) = v;
    }
    const u32 xlane = (u32)((lane & 31) * KS + (lane >> 5) * 32);
    auto load_x = [&](__amdgpu_buffer_rsrc_t xr, u32 t, u32x4 (&r)[4]) {
        const u32 so = (t >> 2) * (32 * KS) + (t & 3) * 128;
#pragma unroll
        for (int j = 0; j < 4; ++j) r[j] = __builtin_amdgcn_raw_buffer_load_b128(xr, xlane + (j >> 1) * 64 + (j & 1) * 16, so, 0);
    };

    if (a >= NCWG / 8) {
        const int g = ((a - NCWG / 8) * 8 + xcd) * NSEND + wave;
        if (wave >= NSEND || g >= WPD * (NR - 1)) { __syncthreads(); return; }
        const int dd = 1 + g % (NR - 1);
        const u32 wi = (u32)(g / (NR - 1));
        const int dst = (p.rank + dd) & (NR - 1);
        unsigned char* rws = p.ws[dst];
        u32* queue = (u32*)(my + WS_Q + dd * 128);
        const __amdgpu_buffer_rsrc_t xr = make_rsrc(p.x + (size_t)dst * MS * KS, MS * KS);
        const u64 tstart = __builtin_amdgcn_s_memrealtime() + 130;
        u32 t0 = wi, t1 = wi + WPD, t2 = wi + 2 * WPD;
        u32x4 r[4];
        if (t0 < NXB) load_x(xr, t0, r);
        __syncthreads();
        const u32 call = rfl(*(const u32*)(lds + L_CALL));
        const u32 par = call & 1u;
        const u32 qbase = (call - 1u) * NBLK - 3u * WPD;
        const __amdgpu_buffer_rsrc_t wl = make_rsrc(my + WS_W + par * W_PAR + (size_t)p.rank * (W_PAR / NR), (u32)(W_PAR / NR));
        const u64* fl = (const u64*)(my + WS_FW + par * FW_PAR);
        unsigned char* xd = rws + WS_X + par * X_PAR + (size_t)p.rank * (X_PAR / NR) + lane * 16;
        unsigned char* wd = rws + WS_W + par * W_PAR + (size_t)p.rank * (W_PAR / NR) + lane * 16;
        unsigned char* fline = rws + WS_FL + par * FL_PAR + (size_t)p.rank * 8 * 128 + (lane & 7) * 16;
        u32* fx = (u32*)(rws + WS_FX + par * FX_PAR) + p.rank * 32;
        auto kc_of = [&](u32 t) { return (t - NXB) >> 6; };
        auto jb_of = [&](u32 t) { return (t - NXB) & 63u; };
        auto load_w = [&](u32 t) {
#pragma unroll
            for (int j = 0; j < 4; ++j) r[j] = __builtin_amdgcn_raw_buffer_load_b128(wl, lane * 16 + j * 1024, (t - NXB) * 4096, SYS);
        };
        auto wait_hdr = [&](u32 t, u32& h) -> bool {
            const u64* f = fl + jb_of(t) * 8 + kc_of(t);
            u64 v = __hip_atomic_load(f, __ATOMIC_RELAXED, __HIP_MEMORY_SCOPE_SYSTEM);
            if ((u32)v < call) {
                const u64 ts = __builtin_amdgcn_s_memrealtime();
                do {
                    if (timed_out(tmo, call, ts)) return false;
                    __builtin_amdgcn_s_sleep(2);
                    v = __hip_atomic_load(f, __ATOMIC_RELAXED, __HIP_MEMORY_SCOPE_SYSTEM);
                } while ((u32)v < call);
            }
            CFENCE();
            h = rfl((u32)(v >> 32)) & 0xffffu;
            return true;
        };
        u32 h0 = H_RAW, h1 = H_RAW, loc0 = LOC_OWN;
        u64 f1 = 0;
        if (t0 >= NXB) { if (!wait_hdr(t0, h0)) return; load_w(t0); }
        if (wi > STAG0) {
            const u64 until = tstart + (u64)((wi - STAG0) * STAGD) / 10u;
            while (__builtin_amdgcn_s_memrealtime() < until) __builtin_amdgcn_s_sleep(1);
        }
#pragma unroll 1
        while (t0 < NBLK) {
            u32 tk = 0;
            if (lane == 0) tk = __hip_atomic_fetch_add(queue, 1u, __ATOMIC_RELAXED, __HIP_MEMORY_SCOPE_AGENT);
            unsigned char* d = t0 < NXB ? xd + (size_t)t0 * 4096 : wd + (size_t)(t0 - NXB) * 4096;
            const u32 nl2 = (h0 & H_F5) ? 56u : 64u;
            const u32 nl3 = is_raw(h0) ? 64u : loc0 != LOC_OWN ? 0u : ((h0 & 255u) + 7u) >> 3;
            st16_sys(d, r[0]);
            st16_sys(d + 1024, r[1]);
            if ((u32)lane < nl2) st16_sys(d + 2 * 1024, r[2]);
            if ((u32)lane < nl3) st16_sys(d + 3 * 1024, r[3]);
            h1 = H_RAW;
            if (t1 < NXB) load_x(xr, t1, r);
            else if (t1 < NBLK) {
                if (rfl((u32)f1) >= call) h1 = rfl((u32)(f1 >> 32)) & 0xffffu;
                else if (!wait_hdr(t1, h1)) return;
                load_w(t1);
            }
            u64 f2 = 0;
            if (t2 >= NXB && t2 < NBLK) f2 = __hip_atomic_load(fl + jb_of(t2) * 8 + kc_of(t2), __ATOMIC_RELAXED, __HIP_MEMORY_SCOPE_SYSTEM);
            VM0();
            u32 loc1 = LOC_OWN;
            if (t0 < NXB) {
                if (lane == 0) __hip_atomic_store(fx + (t0 >> 5) * 256 + (t0 & 31), call, __ATOMIC_RELAXED, __HIP_MEMORY_SCOPE_SYSTEM);
            } else {
                const bool ride = t1 < NBLK && !is_raw(h1) && (h1 & 255u) <= MAXRIDE;
                if (ride) loc1 = t0 - NXB;
                u32x4 v = r[3];
                if (lane == 7) { v.z = call; v.w = h0 | (loc0 << 16); }
                if (lane == 7 || (ride && lane < 7)) st16_sys(fline + (size_t)(jb_of(t0) * 64 + kc_of(t0)) * 128, v);
            }
            t0 = t1; h0 = h1; loc0 = loc1;
            t1 = t2; f1 = f2;
            t2 = rfl(tk) - qbase;
        }
        return;
    }

    const int jb = xcd * 8 + (a & 7), ti = a >> 3;

    if (wave >= W_STG && wave < W_POLL) {
        const int sg = wave - W_STG;
        PRIO(3);
        const u32 id = (u32)(sg * NCWG + a * 8 + xcd);
        u32 r[16];
        if (id < 512) {
            const __amdgpu_buffer_rsrc_t wr = make_rsrc(p.w, KS * NN);
            const u32 wlane = (u32)((lane >> 4) * 16 * NN + (lane & 15) * 4);
            const u32 so = (id >> 6) * 64 * NN + (id & 63u) * 64;
#pragma unroll
            for (int j = 0; j < 16; ++j) r[j] = __builtin_amdgcn_raw_buffer_load_b32(wr, wlane + j * NN, so, 0);
        }
        __syncthreads();
        const u32 call = rfl(*(const u32*)(lds + L_CALL));
        const u32 par = call & 1u;
        if (id < 512) {
            u32 T[4][4];
#pragma unroll
            for (int t4 = 0; t4 < 4; ++t4) {
                const u32 r0 = r[4 * t4], r1 = r[4 * t4 + 1], r2 = r[4 * t4 + 2], r3 = r[4 * t4 + 3];
                const u32 aa = __builtin_amdgcn_perm(r1, r0, 0x05010400u), bb = __builtin_amdgcn_perm(r1, r0, 0x07030602u);
                const u32 cc = __builtin_amdgcn_perm(r3, r2, 0x05010400u), ee = __builtin_amdgcn_perm(r3, r2, 0x07030602u);
                T[0][t4] = __builtin_amdgcn_perm(cc, aa, 0x05040100u);
                T[1][t4] = __builtin_amdgcn_perm(cc, aa, 0x07060302u);
                T[2][t4] = __builtin_amdgcn_perm(ee, bb, 0x05040100u);
                T[3][t4] = __builtin_amdgcn_perm(ee, bb, 0x07060302u);
            }
            u32x4 o[4];
            const u32 hdr = pack_block(T, (id >> 6) >= F5FROM, lds + L_B + sg * 4096, lds + L_B + (sg + NSTAGE) * 4096, (const u32*)(lds + L_TAB), lane, o);
            unsigned char* d = my + WS_W + par * W_PAR + (size_t)p.rank * (W_PAR / NR) + (size_t)id * 4096 + lane * 16;
#pragma unroll
            for (int j = 0; j < 4; ++j) st16_sys(d + j * 1024, o[j]);
            VM0();
            if (lane == 0) __hip_atomic_store((u64*)(my + WS_FW + par * FW_PAR) + (id & 63u) * 8 + (id >> 6), (u64)call | ((u64)(hdr | (LOC_OWN << 16)) << 32),
                                              __ATOMIC_RELAXED, __HIP_MEMORY_SCOPE_SYSTEM);
        } else if (id - 512 < NXB) {
            const u32 t = id - 512;
            u32x4 o[4];
            load_x(make_rsrc(p.x + (size_t)p.rank * MS * KS, MS * KS), t, o);
            unsigned char* d = my + WS_X + par * X_PAR + (size_t)p.rank * (X_PAR / NR) + (size_t)t * 4096 + lane * 16;
#pragma unroll
            for (int j = 0; j < 4; ++j) st16_sys(d + j * 1024, o[j]);
            VM0();
            if (lane == 0) __hip_atomic_store((u32*)(my + WS_FX + par * FX_PAR) + p.rank * 32 + (t >> 5) * 256 + (t & 31), call, __ATOMIC_RELAXED, __HIP_MEMORY_SCOPE_SYSTEM);
        }
        const __amdgpu_buffer_rsrc_t rw = make_rsrc(my + WS_W + par * W_PAR, (u32)W_PAR);
        const __amdgpu_buffer_rsrc_t rall = make_rsrc(my, (u32)WS_END);
        for (u32 n = (u32)sg; n < 64; n += NSTAGE) {
            PRIO(0);
            u32 t, ent;
            while (true) {
                u32 v0, v1;
                asm volatile("ds_read_b32 %0, %2\n\tds_read_b32 %1, %3\n\ts_waitcnt lgkmcnt(0)"
                             : "=&v"(v0), "=&v"(v1) : "v"(lds0 + (u32)L_TAIL), "v"(lds0 + (u32)L_ORD + n * 4u) : "memory");
                t = rfl(v0); ent = rfl(v1);
                if (t > n) break;
                __builtin_amdgcn_s_sleep(1);
            }
            PRIO(3);
            if (t == DEAD) break;
            const u32 u = ent & 63u, hdr = (ent >> 6) & 0xffffu, loc = ent >> 22;
            const u32 so = u * (64 * 4096) + (u32)jb * 4096;
            const bool raw = is_raw(hdr), f5 = (hdr & H_F5) != 0u;
            const u32 lo = loc == LOC_OWN ? (u32)(WS_W + par * W_PAR) + so + 3072u
                                          : (u32)(WS_FL + par * FL_PAR) + (((loc & 63u) * 8u + (u >> 3)) * 8u + (loc >> 6)) * 128u;
            u32x4 v[4];
            v[0] = __builtin_amdgcn_raw_buffer_load_b128(rw, lane * 16, so, SYS);
            v[1] = __builtin_amdgcn_raw_buffer_load_b128(rw, lane * 16 + 1024, so, SYS);
            const u32x2 w2 = __builtin_amdgcn_raw_buffer_load_b64(rw, f5 ? lane * 8 + 2048 : lane * 16 + 2048, so, SYS);
            const u32x2 w3 = __builtin_amdgcn_raw_buffer_load_b64(rw, f5 ? lane * 4 + 2560 : lane * 16 + 2056, so, SYS);
            const u32 g1 = __builtin_amdgcn_raw_buffer_load_b16(rw, lane * 2 + 2816, so, SYS);
            u32 e = __builtin_amdgcn_raw_buffer_load_b16(rall, lane * 2, lo, SYS);
            v[2] = (u32x4){w2.x, w2.y, w3.x, w3.y};
            if (raw) v[3] = __builtin_amdgcn_raw_buffer_load_b128(rw, lane * 16 + 3072, so, SYS);
            if (n >= NSLOT) {
                bool dead = false;
                while (!__all(lds_acquire(lds, L_CONS + (lane & 7) * 4) > n - NSLOT)) {
                    if (rfl(lds_acquire(lds, L_TAIL)) == DEAD) { dead = true; break; }
                    __builtin_amdgcn_s_sleep(1);
                }
                if (dead) break;
            }
            unsigned char* slot = lds + L_B + (n & (NSLOT - 1)) * 4096;
            const u32 basex = ((hdr >> 8) & 127u) * 0x01010101u;
            if (raw) {
#pragma unroll
                for (int q = 0; q < 4; ++q) *(u32x4*)(slot + q * 1024 + lane * 16) = v[q];
            } else if (f5) {
                const u32 P[10] = {v[0].x, v[0].y, v[0].z, v[0].w, v[1].x, v[1].y, v[1].z, v[1].w, v[2].x, v[2].y};
                u32 D[16];
#pragma unroll
                for (int h = 0; h < 2; ++h) {
                    const u32* Q = P + 5 * h;
#pragma unroll
                    for (int k = 0; k < 5; ++k) D[8 * h + k] = (Q[k] & 0x8f8f8f8fu) + basex;
                    D[8 * h + 5] = (((Q[0] >> 4) & 0x07070707u) | ((Q[3] >> 1) & 0x08080808u) | ((Q[4] << 3) & 0x80808080u)) + basex;
                    D[8 * h + 6] = (((Q[1] >> 4) & 0x07070707u) | ((Q[3] >> 2) & 0x08080808u) | ((Q[4] << 2) & 0x80808080u)) + basex;
                    D[8 * h + 7] = (((Q[2] >> 4) & 0x07070707u) | ((Q[3] >> 3) & 0x08080808u) | ((Q[4] << 1) & 0x80808080u)) + basex;
                }
#pragma unroll
                for (int q = 0; q < 4; ++q) *(u32x4*)(slot + q * 1024 + lane * 16) = (u32x4){D[4 * q], D[4 * q + 1], D[4 * q + 2], D[4 * q + 3]};
                const u32 g0 = w3.x;
                u32 sy[12], adv = 0;
#pragma unroll
                for (int j = 0; j < 12; ++j) {
                    sy[j] = ((j < 8 ? g0 : g1) >> (4 * (j & 7))) & 15u;
                    adv += min(sy[j] + 1u, 15u);
                }
                u32 cur = scan_add(adv) - adv - 1u;
#pragma unroll
                for (int j = 0; j < 12; ++j) {
                    cur += min(sy[j] + 1u, 15u);
                    __hip_atomic_fetch_sub(lds_word(slot + (cur & 0xffcu)), sy[j] != 15u ? 16u << ((cur & 3u) * 8u) : 0u,
                                           __ATOMIC_RELAXED, __HIP_MEMORY_SCOPE_WORKGROUP);
                }
            } else {
                const u32 P[4][3] = {{v[0].x, v[0].y, v[0].z}, {v[0].w, v[1].x, v[1].y}, {v[1].z, v[1].w, v[2].x}, {v[2].y, v[2].z, v[2].w}};
#pragma unroll
                for (int q = 0; q < 4; ++q) {
                    const u32 d3 = ((P[q][0] >> 5) & 0x03030303u) | ((P[q][1] >> 3) & 0x0c0c0c0cu) | ((P[q][2] >> 1) & 0x10101010u) | ((P[q][2] << 1) & 0x80808080u);
                    *(u32x4*)(slot + q * 1024 + lane * 16) =
                        (u32x4){(P[q][0] & 0x9f9f9f9fu) + basex, (P[q][1] & 0x9f9f9f9fu) + basex, (P[q][2] & 0x9f9f9f9fu) + basex, d3 + basex};
                }
            }
            if (!raw) {
                const u32 cnt = hdr & 255u;
                for (u32 e0 = 0; e0 < cnt; e0 += 64) {
                    if (e0) e = __builtin_amdgcn_raw_buffer_load_b16(rall, lane * 2 + e0 * 2, lo, SYS);
                    if (e0 + (u32)lane < cnt)
                        __hip_atomic_fetch_xor(lds_word(slot + (e & 0xffcu)), ((e >> 12) & 7u) << ((e & 3u) * 8u + 4u), __ATOMIC_RELAXED, __HIP_MEMORY_SCOPE_WORKGROUP);
                }
            }
            if (lane == 0) __hip_atomic_fetch_or(lds_word(lds + L_STG + (n >> 5) * 4), 1u << (n & 31u), __ATOMIC_RELEASE, __HIP_MEMORY_SCOPE_WORKGROUP);
        }
        return;
    }

    __syncthreads();
    const u32 call = rfl(*(const u32*)(lds + L_CALL));
    const u32 par = call & 1u;

    if (wave == W_POLL) {
        PRIO(2);
        const u64* fwp = (lane >> 3) == p.rank ? (const u64*)(my + WS_FW + par * FW_PAR) + jb * 8 + (lane & 7)
                                               : (const u64*)(my + WS_FL + par * FL_PAR + (size_t)(jb * 64 + lane) * 128 + 120);
        const __amdgpu_buffer_rsrc_t rf = make_rsrc(my + WS_FX + par * FX_PAR + ti * 1024, 1024);
        u64 seen = 0, xok = 0;
        u32 tail = 0, spins = 0;
        const u64 t0 = __builtin_amdgcn_s_memrealtime();
        while (seen != ~0ull) {
            u64 f = 0;
            if (!((seen >> lane) & 1ull)) f = __hip_atomic_load(fwp, __ATOMIC_RELAXED, __HIP_MEMORY_SCOPE_SYSTEM);
            if (xok != ~0ull) {
                const u32x4 g = __builtin_amdgcn_raw_buffer_load_b128(rf, lane * 16, 0, SYS);
                u64 mx = __ballot(g.x >= call && g.y >= call && g.z >= call && g.w >= call);
                mx &= (mx >> 4) & 0x0f0f0f0f0f0f0f0full; mx &= mx >> 2; mx &= mx >> 1; mx &= 0x0101010101010101ull;
                xok = mx * 0xffull;
            }
            const u64 fresh = __ballot((u32)f >= call) & xok & ~seen;
            if (fresh) {
                if ((fresh >> lane) & 1) {
                    const u32 pos = tail + lanes_below(fresh);
                    *(u32*)(lds + L_ORD + pos * 4) = (u32)lane | ((u32)(f >> 32) << 6);
                }
                tail += (u32)__builtin_popcountll(fresh);
                seen |= fresh;
                if (lane == 0) lds_release(lds, L_TAIL, tail);
            } else {
                if ((++spins & 15u) == 0u && timed_out(tmo, call, t0)) {
                    VM0();
                    if (lane == 0) { lds_release(lds, L_BAD, 2u); lds_release(lds, L_TAIL, DEAD); }
                    return;
                }
                __builtin_amdgcn_s_sleep(4);
            }
            CFENCE();
        }
        const u32 tv = __hip_atomic_load(tmo, __ATOMIC_RELAXED, __HIP_MEMORY_SCOPE_AGENT);
        if (lane == 0) lds_release(lds, L_BAD, tv == call ? 2u : 1u);
        return;
    }

    const __amdgpu_buffer_rsrc_t rx = make_rsrc(my + WS_X + par * X_PAR, (u32)X_PAR);
    const float scale = p.sx[0] * p.sw[0];
    f32x16 acc0 = {}, acc1 = {};
    const u32 b_rd = (u32)(((lane >> 4) & 1) * 2048 + (lane >> 5) * 512 + (lane & 15) * 16);
    const u32 a_off = (u32)((ti * 8 + wave) * 8 * 2048 + lane * 16);
    u32 n = 0, t = 0, ord = 0;
    u64 staged = 0;
    auto snapshot = [&]() {
        u32x4 c;
        asm volatile("ds_read_b128 %0, %2\n\tds_read_b32 %1, %3\n\ts_waitcnt lgkmcnt(0)"
                     : "=&v"(c), "=&v"(ord) : "v"(lds0 + (u32)L_STG), "v"(lds0 + (u32)L_ORD + (u32)lane * 4u) : "memory");
        t = rfl(c.z);
        staged = ((u64)rfl(c.y) << 32) | rfl(c.x);
    };
    while (n < 64) {
        if (t <= n) {
            PRIO(0);
            while (true) { snapshot(); if (t > n) break; __builtin_amdgcn_s_sleep(1); }
            PRIO(1);
        }
        if (t == DEAD) break;
        const u32 m = min(t - n, 4u);
        u32x4 av[4][2];
#pragma unroll
        for (int j = 0; j < 4; ++j) {
            const u32 u = (u32)__builtin_amdgcn_readlane(ord, n + min((u32)j, m - 1u)) & 63u;
            const u32 so = (u >> 3) * (u32)(X_PAR / NR) + (u & 7) * 2048;
#pragma unroll
            for (int s = 0; s < 2; ++s) av[j][s] = __builtin_amdgcn_raw_buffer_load_b128(rx, a_off + s * 1024, so, SYS);
        }
        bool dead = false;
#pragma unroll
        for (int j = 0; j < 4; ++j) {
            if ((u32)j < m && !dead) {
                while (!((staged >> (n + j)) & 1ull)) {
                    snapshot();
                    if (t == DEAD) { dead = true; break; }
                    if (!((staged >> (n + j)) & 1ull)) __builtin_amdgcn_s_sleep(1);
                }
                if (!dead) {
                    const unsigned char* bb = lds + L_B + ((n + j) & (NSLOT - 1)) * 4096 + b_rd;
                    u32x4 b00 = *(const u32x4*)(bb), b01 = *(const u32x4*)(bb + 256);
                    u32x4 b10 = *(const u32x4*)(bb + 1024), b11 = *(const u32x4*)(bb + 1280);
                    asm volatile("" : "+v"(b00), "+v"(b01), "+v"(b10), "+v"(b11));
                    const i32x8 af = {(int)av[j][0].x, (int)av[j][0].y, (int)av[j][0].z, (int)av[j][0].w, (int)av[j][1].x, (int)av[j][1].y, (int)av[j][1].z, (int)av[j][1].w};
                    const i32x8 bf0 = {(int)b00.x, (int)b00.y, (int)b00.z, (int)b00.w, (int)b01.x, (int)b01.y, (int)b01.z, (int)b01.w};
                    const i32x8 bf1 = {(int)b10.x, (int)b10.y, (int)b10.z, (int)b10.w, (int)b11.x, (int)b11.y, (int)b11.z, (int)b11.w};
                    acc0 = __builtin_amdgcn_mfma_scale_f32_32x32x64_f8f6f4(af, bf0, acc0, 0  , 1  , 0, 0, 0, 0);
                    acc1 = __builtin_amdgcn_mfma_scale_f32_32x32x64_f8f6f4(af, bf1, acc1, 0, 1, 0, 0, 0, 0);
                }
            }
        }
        if (dead) break;
        n += m;
        if (lane == 0) lds_release(lds, L_CONS + wave * 4, n);
        snapshot();
    }

    u32 bad;
    while ((bad = rfl(lds_acquire(lds, L_BAD))) == 0u) __builtin_amdgcn_s_sleep(1);
    const float s = bad == 2u ? __builtin_nanf("") : scale;
    float* o = p.out + (size_t)(ti * 256 + wave * 32 + (lane >> 5) * 4 + ((lane >> 4) & 1)) * NN + jb * 64 + (lane & 15) * 4;
#pragma unroll
    for (int r = 0; r < 16; r += 2) {
        const auto s0 = __builtin_amdgcn_permlane16_swap(__float_as_uint(acc0[r] * s), __float_as_uint(acc0[r + 1] * s), false, false);
        const auto s1 = __builtin_amdgcn_permlane16_swap(__float_as_uint(acc1[r] * s), __float_as_uint(acc1[r + 1] * s), false, false);
        st16_sys(o + (size_t)((r & 3) + 8 * (r >> 2)) * NN, (u32x4){s0[0], s1[0], s0[1], s1[1]});
    }
}

}

extern "C" void kernel_launch(int rank, int nranks, ncclComm_t comm, hipStream_t stream,
                              void* const* d_in, const int* in_sizes, int n_in,
                              void* d_out, int out_size,
                              const int* in_dtypes, int out_dtype,
                              void* const* d_out_all,
                              void* const* d_ws_all, size_t ws_bytes) {
    (void)comm; (void)in_sizes; (void)n_in; (void)out_size; (void)in_dtypes; (void)out_dtype; (void)d_out_all;
    if (nranks != NR || ws_bytes < WS_END || (unsigned)rank >= (unsigned)NR) {
        fprintf(stderr, "unexpected configuration\n");
        return;
    }
    Params p;
    p.x = (const unsigned char*)d_in[0];
    p.w = (const unsigned char*)d_in[1];
    p.sx = (const float*)d_in[2];
    p.sw = (const float*)d_in[3];
    p.out = (float*)d_out;
    for (int i = 0; i < NR; ++i) p.ws[i] = (unsigned char*)d_ws_all[i];
    p.rank = rank;
    p.pad = 0;
    hipLaunchKernelGGL(gemm_ag_fused, dim3(NWG), dim3(NTHREADS), 0, stream, p);
    HIP_CHECK(hipGetLastError());
}
```

```cpp
#include <hip/hip_runtime.h>
#include <rccl/rccl.h>
#include <cstdio>

#define HIP_CHECK(expr) do { hipError_t _e = (expr); if (_e != hipSuccess) { \
    fprintf(stderr, "HIP error %s at %s:%d\n", hipGetErrorString(_e), __FILE__, __LINE__); \
    return; } } while (0)

namespace {

typedef unsigned u32;
typedef unsigned long long u64;
typedef u32 u32x2 __attribute__((ext_vector_type(2)));
typedef u32 u32x4 __attribute__((ext_vector_type(4)));
typedef int i32x8 __attribute__((ext_vector_type(8)));
typedef float f32x16 __attribute__((ext_vector_type(16)));

#define NSEND_ 5
#define PSLEEP 0
#define STAGOFF 130
#define F5ID 32
#define STAG0 32u
#define STAGD 80

constexpr int NR = 8;
constexpr int NN = 4096;
constexpr int KS = 512;
constexpr int MS = 512;
constexpr int NWG = 256;
constexpr int NCWG = 128;
constexpr int NWORK = 8;
constexpr int NSTAGE = 7;
constexpr int W_STG  = NWORK;
constexpr int W_POLL = W_STG + NSTAGE;
constexpr int NTHREADS = 64 * (W_POLL + 1);
constexpr int NSEND = NSEND_;
constexpr int WPD = (NWG - NCWG) * NSEND / (NR - 1);
constexpr u32 NXB = 64;
constexpr u32 NBLK = NXB + 512;

constexpr size_t WS_W   = 0;
constexpr size_t W_PAR  = (size_t)NR * 8 * 64 * 4096;
constexpr size_t WS_X   = 2 * W_PAR;
constexpr size_t X_PAR  = (size_t)NR * 16 * 8 * 2048;
constexpr size_t WS_FL  = WS_X + 2 * X_PAR;
constexpr size_t FL_PAR = 64 * 64 * 128;
constexpr size_t WS_FW  = WS_FL + 2 * FL_PAR;
constexpr size_t FW_PAR = 64 * 8 * 8;
constexpr size_t WS_FX  = WS_FW + 2 * FW_PAR;
constexpr size_t FX_PAR = 8 * 64 * 128;
constexpr size_t WS_Q   = WS_FX + 2 * FX_PAR;
constexpr size_t WS_CNT = WS_Q + NR * 128;
constexpr size_t WS_TMO = WS_CNT + NWG * 32;
constexpr size_t WS_END = WS_TMO + 128;

constexpr u64 DEADLINE_TICKS = 30000000ull;

constexpr u32 H_RAW = 127u << 8;
constexpr u32 H_F5 = 1u << 15;
constexpr u32 MAXFIX = 255;
constexpr u32 MAXRIDE = 60;
constexpr u32 LOC_OWN = 1023;

constexpr int NSLOT  = 32;
constexpr int L_B    = 0;
constexpr int L_ORD  = NSLOT * 4096;
constexpr int L_STG  = L_ORD + 256;
constexpr int L_TAIL = L_STG + 8;
constexpr int L_BAD  = L_STG + 12;
constexpr int L_CALL = L_STG + 16;
constexpr int L_CONS = L_CALL + 16;
constexpr int L_TAB  = L_CONS + 32;
constexpr int L_BYTES = L_TAB + 1024;
constexpr u32 DEAD = 0x80000000u;

struct Params {
    const unsigned char* x;
    const unsigned char* w;
    const float* sx;
    const float* sw;
    float* out;
    unsigned char* ws[NR];
    int rank;
    int pad;
};

__device__ __forceinline__ void st16_sys(void* p, u32x4 v) {
    asm volatile("global_store_dwordx4 %0, %1, off sc0 sc1\n\ts_nop 1" :: "v"(p), "v"(v) : "memory");
}
constexpr int SYS = 17;
__device__ __forceinline__ __amdgpu_buffer_rsrc_t make_rsrc(const void* base, u32 bytes) {
    return __builtin_amdgcn_make_buffer_rsrc(const_cast<void*>(base), 0, bytes, 0x00020000);
}
template <int V> struct IC { static constexpr int value = V; };
__device__ __forceinline__ u32 rfl(u32 v) { return __builtin_amdgcn_readfirstlane(v); }
__device__ __forceinline__ u32 lds_acquire(const unsigned char* lds, int off) {
    return __hip_atomic_load((const u32*)(lds + off), __ATOMIC_ACQUIRE, __HIP_MEMORY_SCOPE_WORKGROUP);
}
__device__ __forceinline__ void lds_release(unsigned char* lds, int off, u32 v) {
    __hip_atomic_store((u32*)(lds + off), v, __ATOMIC_RELEASE, __HIP_MEMORY_SCOPE_WORKGROUP);
}
__device__ __forceinline__ u32* lds_word(unsigned char* p) { return (u32*)__builtin_assume_aligned(p, 4); }
__device__ __forceinline__ u32 lanes_below(u64 m) {
    return __builtin_amdgcn_mbcnt_hi((u32)(m >> 32), __builtin_amdgcn_mbcnt_lo((u32)m, 0u));
}
__device__ __forceinline__ bool is_raw(u32 h) { return ((h >> 8) & 127u) == 127u; }
__device__ __forceinline__ u32 scan_add(u32 v) {
    v += __builtin_amdgcn_update_dpp(0u, v, 0x111, 0xf, 0xf, false);
    v += __builtin_amdgcn_update_dpp(0u, v, 0x112, 0xf, 0xf, false);
    v += __builtin_amdgcn_update_dpp(0u, v, 0x114, 0xf, 0xf, false);
    v += __builtin_amdgcn_update_dpp(0u, v, 0x118, 0xf, 0xf, false);
    v += __builtin_amdgcn_update_dpp(0u, v, 0x142, 0xa, 0xf, false);
    v += __builtin_amdgcn_update_dpp(0u, v, 0x143, 0xc, 0xf, false);
    return v;
}
__device__ __forceinline__ u32 scan_max(u32 v) {
    v = max(v, (u32)__builtin_amdgcn_update_dpp(0u, v, 0x111, 0xf, 0xf, false));
    v = max(v, (u32)__builtin_amdgcn_update_dpp(0u, v, 0x112, 0xf, 0xf, false));
    v = max(v, (u32)__builtin_amdgcn_update_dpp(0u, v, 0x114, 0xf, 0xf, false));
    v = max(v, (u32)__builtin_amdgcn_update_dpp(0u, v, 0x118, 0xf, 0xf, false));
    v = max(v, (u32)__builtin_amdgcn_update_dpp(0u, v, 0x142, 0xa, 0xf, false));
    v = max(v, (u32)__builtin_amdgcn_update_dpp(0u, v, 0x143, 0xc, 0xf, false));
    return v;
}
__device__ __forceinline__ u32 lane_before(u32 v) {
    u32 r = __builtin_amdgcn_update_dpp(0u, v, 0x138, 0xf, 0xf, false);
    asm volatile("" : "+v"(r));
    return r;
}
#define PRIO(x) __builtin_amdgcn_s_setprio(x)
#define VM0() asm volatile("s_waitcnt vmcnt(0)" ::: "memory")
#define CFENCE() asm volatile("" ::: "memory")

__device__ __forceinline__ bool timed_out(u32* tmo, u32 call, u64 t0) {
    if (__hip_atomic_load(tmo, __ATOMIC_RELAXED, __HIP_MEMORY_SCOPE_AGENT) == call
        || __builtin_amdgcn_s_memrealtime() - t0 > DEADLINE_TICKS) {
        __hip_atomic_store(tmo, call, __ATOMIC_RELAXED, __HIP_MEMORY_SCOPE_AGENT);
        return true;
    }
    return false;
}

__device__ __forceinline__ u32 build_list(const u32 (&F)[16], unsigned char* list, unsigned char* dump, int lane) {
    u32 M[2] = {0u, 0u};
#pragma unroll
    for (int i = 0; i < 16; ++i) {
        const u32 nz = (F[i] | (F[i] << 1) | (F[i] << 2)) & 0x40404040u;
        M[i >> 3] |= __builtin_amdgcn_udot4(nz >> 6, 0x08040201u, 0u, false) << (4 * (i & 7));
    }
#pragma unroll
    for (int q = 0; q < 4; ++q) *(u32x4*)(dump + q * 1024 + lane * 16) = (u32x4){F[4 * q], F[4 * q + 1], F[4 * q + 2], F[4 * q + 3]};
    const u32 cnt = (u32)__builtin_popcount(M[0]) + (u32)__builtin_popcount(M[1]);
    const u32 incl = scan_add(cnt);
    u32 idx = incl - cnt;
    u64 m = ((u64)M[1] << 32) | M[0];
    while (__ballot(m != 0ull)) {
        if (m != 0ull) {
            const u32 i = (u32)__builtin_ctzll(m);
            const u32 pos = (i >> 4) * 1024u + (u32)lane * 16u + (i & 15u);
            if (idx < MAXFIX) *(unsigned short*)(list + idx * 2) = (unsigned short)(pos | (((u32)dump[pos] >> 4) << 12));
            ++idx;
            m &= m - 1ull;
        }
    }
    return (u32)__builtin_amdgcn_readlane(incl, 63);
}

__device__ __forceinline__ u32 pack_block(const u32 (&T)[4][4], bool try5, unsigned char* scratch, unsigned char* dump, const u32* gaptab, int lane, u32x4 (&o)[4]) {
    u32 sum = 0;
#pragma unroll
    for (int i = 0; i < 16; ++i) sum = __builtin_amdgcn_sad_u8(T[i >> 2][i & 3] & 0x7f7f7f7fu, 0u, sum);
    const int bs = ((int)__builtin_amdgcn_readlane(scan_add(sum), 63) - 82944) >> 12;
    const u32 base = (u32)min(max(bs, 0), 96);
    if (try5) {
        const u32 b16 = base + 16u, b16x = b16 * 0x01010101u;
        u32 D[16], F[16], M[4] = {0u, 0u, 0u, 0u};
        if (lane < 32) *(u32x4*)(scratch + 2048 + lane * 16) = (u32x4){0u, 0u, 0u, 0u};
#pragma unroll
        for (int i = 0; i < 16; ++i) {
            const u32 d = T[i >> 2][i & 3];
            const u32 u = (d | 0x80808080u) - b16x;
            const u32 low4 = u & 0x0f0f0f0fu;
            D[i] = (d & 0x80808080u) | low4;
            const u32 t = u ^ 0x80808080u;
            const u32 lo = t & (t << 1) & (t << 2) & (t << 3) & 0x80808080u;
            M[i >> 2] |= __builtin_amdgcn_udot4(lo >> 7, 0x08040201u, 0u, false) << (4 * (i & 3));
            F[i] = (d ^ (low4 + b16x)) & 0x70707070u & ~((lo >> 1) | (lo >> 2) | (lo >> 3));
        }
        u32 smax[4], first[4], lead[4], cnt[4];
#pragma unroll
        for (int q = 0; q < 4; ++q) smax[q] = scan_max(M[q] ? (u32)(q * 1024) + (u32)lane * 16u + 32u - (u32)__builtin_clz(M[q]) : 0u);
        u32 curq = 0;
#pragma unroll
        for (int q = 0; q < 4; ++q) {
            const u32 prev = max(lane_before(smax[q]), curq);
            const u32 g = M[q] ? (u32)(q * 1024) + (u32)lane * 16u + (u32)__builtin_ctz(M[q]) - prev : 0u;
            lead[q] = __umul24(g, 4370u) >> 16;
            first[q] = 15u + 15u * lead[q] - g;
            cnt[q] = (u32)__builtin_popcount(M[q]);
            curq = max(curq, (u32)__builtin_amdgcn_readlane(smax[q], 63));
        }
        const u32 i01 = scan_add((lead[0] + cnt[0]) | ((lead[1] + cnt[1]) << 16)), i23 = scan_add((lead[2] + cnt[2]) | ((lead[3] + cnt[3]) << 16));
        const u32 t01 = (u32)__builtin_amdgcn_readlane(i01, 63), t23 = (u32)__builtin_amdgcn_readlane(i23, 63);
        const u32 incl[4] = {i01 & 0xffffu, (t01 & 0xffffu) + (i01 >> 16), (t01 & 0xffffu) + (t01 >> 16) + (i23 & 0xffffu),
                             (t01 & 0xffffu) + (t01 >> 16) + (t23 & 0xffffu) + (i23 >> 16)};
        const u32 nsym = (t01 & 0xffffu) + (t01 >> 16) + (t23 & 0xffffu) + (t23 >> 16);
        if (nsym <= 768u) {
#pragma unroll
            for (int q = 0; q < 4; ++q) {
                const u32 lo = M[q] & 255u, hi = M[q] >> 8, na = (u32)__builtin_popcount(lo);
                u64 X = (u64)first[q] | ((u64)gaptab[lo] << 4) | ((u64)gaptab[hi] << (4u * na + 4u));
                if (lo && hi) X |= (u64)(15u - (8u + (u32)__builtin_ctz(hi) - (32u - (u32)__builtin_clz(lo)))) << (4u * na);
                if (M[q]) {
                    const u32 at = incl[q] - cnt[q], sh = (at & 7u) * 4u, x0 = (u32)X, x1 = (u32)(X >> 32);
                    u32* G = lds_word(scratch + 2048 + (at >> 3) * 4);
                    __hip_atomic_fetch_or(G, x0 << sh, __ATOMIC_RELAXED, __HIP_MEMORY_SCOPE_WORKGROUP);
                    __hip_atomic_fetch_or(G + 1, sh ? __builtin_amdgcn_alignbit(x1, x0, 32u - sh) : x1, __ATOMIC_RELAXED, __HIP_MEMORY_SCOPE_WORKGROUP);
                    __hip_atomic_fetch_or(G + 2, sh ? x1 >> (32u - sh) : 0u, __ATOMIC_RELAXED, __HIP_MEMORY_SCOPE_WORKGROUP);
                }
            }
            const u32 total = build_list(F, scratch, dump, lane);
            if (total <= MAXFIX) {
                const unsigned short* gs = (const unsigned short*)(scratch + 2048 + lane * 6);
                const u32 g0 = ~((u32)gs[0] | ((u32)gs[1] << 16)), g1 = ~(u32)gs[2];
                u32 P[10];
#pragma unroll
                for (int h = 0; h < 2; ++h) {
                    const u32* E = D + 8 * h;
                    P[5 * h + 0] = E[0] | ((E[5] & 0x07070707u) << 4);
                    P[5 * h + 1] = E[1] | ((E[6] & 0x07070707u) << 4);
                    P[5 * h + 2] = E[2] | ((E[7] & 0x07070707u) << 4);
                    P[5 * h + 3] = E[3] | ((E[5] & 0x08080808u) << 1) | ((E[6] & 0x08080808u) << 2) | ((E[7] & 0x08080808u) << 3);
                    P[5 * h + 4] = E[4] | ((E[5] & 0x80808080u) >> 3) | ((E[6] & 0x80808080u) >> 2) | ((E[7] & 0x80808080u) >> 1);
                }
                *(u32x2*)(scratch + 2560 + lane * 8) = (u32x2){P[8], P[9]};
                *(u32*)(scratch + 2560 + 512 + lane * 4) = g0;
                *(unsigned short*)(scratch + 2560 + 768 + lane * 2) = (unsigned short)g1;
                o[0] = (u32x4){P[0], P[1], P[2], P[3]};
                o[1] = (u32x4){P[4], P[5], P[6], P[7]};
                o[2] = *(const u32x4*)(scratch + 2560 + lane * 16);
                o[3] = *(const u32x4*)(scratch + lane * 16);
                return total | (b16 << 8) | H_F5;
            }
        }
    }
    const u32 basex = base * 0x01010101u;
    u32 D[16], F[16];
#pragma unroll
    for (int i = 0; i < 16; ++i) {
        const u32 d = T[i >> 2][i & 3];
        const u32 code = ((d | 0x80808080u) - basex) & 0x1f1f1f1fu;
        D[i] = (d & 0x80808080u) | code;
        F[i] = (d ^ (code + basex)) & 0x60606060u;
    }
    const u32 total = build_list(F, scratch, dump, lane);
    if (total <= MAXFIX) {
        u32 P[4][3];
#pragma unroll
        for (int q = 0; q < 4; ++q) {
            P[q][0] = D[4 * q] | ((D[4 * q + 3] & 0x03030303u) << 5);
            P[q][1] = D[4 * q + 1] | ((D[4 * q + 3] & 0x0c0c0c0cu) << 3);
            P[q][2] = D[4 * q + 2] | ((D[4 * q + 3] & 0x10101010u) << 1) | ((D[4 * q + 3] & 0x80808080u) >> 1);
        }
        o[0] = (u32x4){P[0][0], P[0][1], P[0][2], P[1][0]};
        o[1] = (u32x4){P[1][1], P[1][2], P[2][0], P[2][1]};
        o[2] = (u32x4){P[2][2], P[3][0], P[3][1], P[3][2]};
        o[3] = *(const u32x4*)(scratch + lane * 16);
        return total | (base << 8);
    }
#pragma unroll
    for (int q = 0; q < 4; ++q) o[q] = (u32x4){T[q][0], T[q][1], T[q][2], T[q][3]};
    return H_RAW;
}

__global__ void __launch_bounds__(NTHREADS, NTHREADS / 256) gemm_ag_fused(Params p) {
    __shared__ __attribute__((aligned(16))) unsigned char lds[L_BYTES];
    const int tid = threadIdx.x;
    const int lane = tid & 63;
    const int wave = rfl(tid >> 6);
    const int b = blockIdx.x;
    const int xcd = b & 7, a = b >> 3;
    unsigned char* my = p.ws[p.rank];
    u32* tmo = (u32*)(my + WS_TMO);
    const u32 lds0 = (u32)(size_t)(__attribute__((address_space(3))) unsigned char*)lds;

    u32* cntp = (u32*)(my + WS_CNT) + b * 8 + (a >= NCWG / 8 ? wave & 7 : 0);
    if (a < NCWG / 8 && tid == 0) {
        const u32 c = __hip_atomic_load(cntp, __ATOMIC_RELAXED, __HIP_MEMORY_SCOPE_AGENT) + 1u;
        __hip_atomic_store(cntp, c, __ATOMIC_RELAXED, __HIP_MEMORY_SCOPE_AGENT);
        *(u32*)(lds + L_CALL) = c;
    }
    if (a < NCWG / 8 && wave == 1) {
        if (lane < 4) *(u32*)(lds + L_STG + lane * 4) = 0u;
        if (lane < 8) *(u32*)(lds + L_CONS + lane * 4) = 0u;
    }
    if (a < NCWG / 8 && tid >= 128 && tid < 384) {
        const u32 x = (u32)tid - 128u;
        u32 v = 0, k = 0;
        int prev = -1;
#pragma unroll
        for (int bit = 0; bit < 8; ++bit)
            if ((x >> bit) & 1u) {
                if (prev >= 0) { v |= (16u - (u32)(bit - prev)) << (4u * k); ++k; }
                prev = bit;
            }
        *(u32*)(lds + L_TAB + x * 4) = v;
    }
    const u32 xlane = (u32)((lane & 31) * KS + (lane >> 5) * 32);
    auto load_x = [&](__amdgpu_buffer_rsrc_t xr, u32 t, u32x4 (&r)[4]) {
        const u32 so = (t >> 2) * (32 * KS) + (t & 3) * 128;
#pragma unroll
        for (int j = 0; j < 4; ++j) r[j] = __builtin_amdgcn_raw_buffer_load_b128(xr, xlane + (j >> 1) * 64 + (j & 1) * 16, so, 0);
    };

    if (a >= NCWG / 8) {
        const int g = ((a - NCWG / 8) * 8 + xcd) * NSEND + wave;
        if (wave >= NSEND || g >= WPD * (NR - 1)) return;
        const u32 cnt_old = __hip_atomic_load(cntp, __ATOMIC_RELAXED, __HIP_MEMORY_SCOPE_AGENT);
        const int dd = 1 + g % (NR - 1);
        const u32 wi = (u32)(g / (NR - 1));
        const int dst = (p.rank + dd) & (NR - 1);
        unsigned char* rws = p.ws[dst];
        u32* queue = (u32*)(my + WS_Q + dd * 128);
        const __amdgpu_buffer_rsrc_t xr = make_rsrc(p.x + (size_t)dst * MS * KS, MS * KS);
        const u64 tstart = __builtin_amdgcn_s_memrealtime() + STAGOFF;
        u32 t0 = wi, t1 = wi + WPD, t2 = wi + 2 * WPD;
        u32x4 r[4];
        if (t0 < NXB) load_x(xr, t0, r);
        const u32 call = rfl(cnt_old) + 1u;
        if (lane == 0) __hip_atomic_store(cntp, call, __ATOMIC_RELAXED, __HIP_MEMORY_SCOPE_AGENT);
        const u32 par = call & 1u;
        const u32 qbase = (call - 1u) * NBLK - 3u * WPD;
        const __amdgpu_buffer_rsrc_t wl = make_rsrc(my + WS_W + par * W_PAR + (size_t)p.rank * (W_PAR / NR), (u32)(W_PAR / NR));
        const u64* fl = (const u64*)(my + WS_FW + par * FW_PAR);
        unsigned char* xd = rws + WS_X + par * X_PAR + (size_t)p.rank * (X_PAR / NR) + lane * 16;
        unsigned char* wd = rws + WS_W + par * W_PAR + (size_t)p.rank * (W_PAR / NR) + lane * 16;
        unsigned char* fline = rws + WS_FL + par * FL_PAR + (size_t)p.rank * 8 * 128 + (lane & 7) * 16;
        unsigned char* xline = rws + WS_FX + par * FX_PAR + (size_t)p.rank * 64 * 128 + (lane & 7) * 16;
        auto kc_of = [&](u32 t) { return (t - NXB) >> 6; };
        auto jb_of = [&](u32 t) { return (t - NXB) & 63u; };
        auto load_w = [&](u32 t) {
#pragma unroll
            for (int j = 0; j < 4; ++j) r[j] = __builtin_amdgcn_raw_buffer_load_b128(wl, lane * 16 + j * 1024, (t - NXB) * 4096, SYS);
        };
        auto wait_hdr = [&](u32 t, u32& h) -> bool {
            const u64* f = fl + jb_of(t) * 8 + kc_of(t);
            u64 v = __hip_atomic_load(f, __ATOMIC_RELAXED, __HIP_MEMORY_SCOPE_SYSTEM);
            if ((u32)v < call) {
                const u64 ts = __builtin_amdgcn_s_memrealtime();
                do {
                    if (timed_out(tmo, call, ts)) return false;
                    __builtin_amdgcn_s_sleep(2);
                    v = __hip_atomic_load(f, __ATOMIC_RELAXED, __HIP_MEMORY_SCOPE_SYSTEM);
                } while ((u32)v < call);
            }
            CFENCE();
            h = rfl((u32)(v >> 32)) & 0xffffu;
            return true;
        };
        u32 h0 = H_RAW, h1 = H_RAW, loc0 = LOC_OWN;
        u64 f1 = 0;
        if (t0 >= NXB) { if (!wait_hdr(t0, h0)) return; load_w(t0); }
        if (wi > STAG0) {
            const u64 until = tstart + (u64)((wi - STAG0) * STAGD) / 10u;
            while (__builtin_amdgcn_s_memrealtime() < until) __builtin_amdgcn_s_sleep(1);
        }
#pragma unroll 1
        while (t0 < NBLK) {
            u32 tk = 0;
            if (lane == 0) tk = __hip_atomic_fetch_add(queue, 1u, __ATOMIC_RELAXED, __HIP_MEMORY_SCOPE_AGENT);
            unsigned char* d = t0 < NXB ? xd + (size_t)t0 * 4096 : wd + (size_t)(t0 - NXB) * 4096;
            const u32 nl2 = (h0 & H_F5) ? 56u : 64u;
            const u32 nl3 = is_raw(h0) ? 64u : loc0 != LOC_OWN ? 0u : ((h0 & 255u) + 7u) >> 3;
            st16_sys(d, r[0]);
            st16_sys(d + 1024, r[1]);
            if ((u32)lane < nl2) st16_sys(d + 2 * 1024, r[2]);
            if ((u32)lane < nl3) st16_sys(d + 3 * 1024, r[3]);
            h1 = H_RAW;
            if (t1 < NXB) load_x(xr, t1, r);
            else if (t1 < NBLK) {
                if (rfl((u32)f1) >= call) h1 = rfl((u32)(f1 >> 32)) & 0xffffu;
                else if (!wait_hdr(t1, h1)) return;
                load_w(t1);
            }
            u64 f2 = 0;
            if (t2 >= NXB && t2 < NBLK) f2 = __hip_atomic_load(fl + jb_of(t2) * 8 + kc_of(t2), __ATOMIC_RELAXED, __HIP_MEMORY_SCOPE_SYSTEM);
            VM0();
            const bool ride = t1 >= NXB && t1 < NBLK && !is_raw(h1) && (h1 & 255u) <= MAXRIDE;
            const u32 loc1 = !ride ? LOC_OWN : t0 < NXB ? 512u + t0 : t0 - NXB;
            {
                u32x4 v = r[3];
                if (lane == 7) { v.z = call; v.w = h0 | (loc0 << 16); }
                unsigned char* line = t0 < NXB ? xline + (size_t)t0 * 128 : fline + (size_t)(jb_of(t0) * 64 + kc_of(t0)) * 128;
                if (lane == 7 || (ride && lane < 7)) st16_sys(line, v);
            }
            t0 = t1; h0 = h1; loc0 = loc1;
            t1 = t2; f1 = f2;
            t2 = rfl(tk) - qbase;
        }
        return;
    }

    const int jb = xcd * 8 + (a & 7), ti = a >> 3;

    if (wave >= W_STG && wave < W_POLL) {
        const int sg = wave - W_STG;
        PRIO(3);
        const u32 id = (u32)(sg * NCWG + a * 8 + xcd);
        u32 r[16];
        if (PSLEEP) __builtin_amdgcn_s_sleep(PSLEEP);
        if (id < 512) {
            const __amdgpu_buffer_rsrc_t wr = make_rsrc(p.w, KS * NN);
            const u32 wlane = (u32)((lane >> 4) * 16 * NN + (lane & 15) * 4);
            const u32 so = (id >> 6) * 64 * NN + (id & 63u) * 64;
#pragma unroll
            for (int j = 0; j < 16; ++j) r[j] = __builtin_amdgcn_raw_buffer_load_b32(wr, wlane + j * NN, so, 0);
        }
        __syncthreads();
        const u32 call = rfl(*(const u32*)(lds + L_CALL));
        const u32 par = call & 1u;
        if (id < 512) {
            u32 T[4][4];
#pragma unroll
            for (int t4 = 0; t4 < 4; ++t4) {
                const u32 r0 = r[4 * t4], r1 = r[4 * t4 + 1], r2 = r[4 * t4 + 2], r3 = r[4 * t4 + 3];
                const u32 aa = __builtin_amdgcn_perm(r1, r0, 0x05010400u), bb = __builtin_amdgcn_perm(r1, r0, 0x07030602u);
                const u32 cc = __builtin_amdgcn_perm(r3, r2, 0x05010400u), ee = __builtin_amdgcn_perm(r3, r2, 0x07030602u);
                T[0][t4] = __builtin_amdgcn_perm(cc, aa, 0x05040100u);
                T[1][t4] = __builtin_amdgcn_perm(cc, aa, 0x07060302u);
                T[2][t4] = __builtin_amdgcn_perm(ee, bb, 0x05040100u);
                T[3][t4] = __builtin_amdgcn_perm(ee, bb, 0x07060302u);
            }
            u32x4 o[4];
            const u32 hdr = pack_block(T, id >= F5ID, lds + L_B + sg * 4096, lds + L_B + (sg + NSTAGE) * 4096, (const u32*)(lds + L_TAB), lane, o);
            unsigned char* d = my + WS_W + par * W_PAR + (size_t)p.rank * (W_PAR / NR) + (size_t)id * 4096 + lane * 16;
#pragma unroll
            for (int j = 0; j < 4; ++j) st16_sys(d + j * 1024, o[j]);
            VM0();
            if (lane == 0) __hip_atomic_store((u64*)(my + WS_FW + par * FW_PAR) + (id & 63u) * 8 + (id >> 6), (u64)call | ((u64)(hdr | (LOC_OWN << 16)) << 32),
                                              __ATOMIC_RELAXED, __HIP_MEMORY_SCOPE_SYSTEM);
        } else if (id - 512 < NXB) {
            const u32 t = id - 512;
            u32x4 o[4];
            load_x(make_rsrc(p.x + (size_t)p.rank * MS * KS, MS * KS), t, o);
            unsigned char* d = my + WS_X + par * X_PAR + (size_t)p.rank * (X_PAR / NR) + (size_t)t * 4096 + lane * 16;
#pragma unroll
            for (int j = 0; j < 4; ++j) st16_sys(d + j * 1024, o[j]);
            VM0();
            if (lane == 0) __hip_atomic_store((u32*)(my + WS_FX + par * FX_PAR + (size_t)(p.rank * 64 + t) * 128 + 120), call, __ATOMIC_RELAXED, __HIP_MEMORY_SCOPE_SYSTEM);
        }
        const __amdgpu_buffer_rsrc_t rw = make_rsrc(my + WS_W + par * W_PAR, (u32)W_PAR);
        const __amdgpu_buffer_rsrc_t rall = make_rsrc(my, (u32)WS_END);
        for (u32 n = (u32)sg; n < 64; n += NSTAGE) {
            PRIO(0);
            u32 t, ent;
            while (true) {
                u32 v0, v1;
                asm volatile("ds_read_b32 %0, %2\n\tds_read_b32 %1, %3\n\ts_waitcnt lgkmcnt(0)"
                             : "=&v"(v0), "=&v"(v1) : "v"(lds0 + (u32)L_TAIL), "v"(lds0 + (u32)L_ORD + n * 4u) : "memory");
                t = rfl(v0); ent = rfl(v1);
                if (t > n) break;
                __builtin_amdgcn_s_sleep(1);
            }
            PRIO(3);
            if (t == DEAD) break;
            const u32 u = ent & 63u, hdr = (ent >> 6) & 0xffffu, loc = ent >> 22;
            const u32 so = u * (64 * 4096) + (u32)jb * 4096;
            const bool raw = is_raw(hdr), f5 = (hdr & H_F5) != 0u;
            const u32 lo = loc == LOC_OWN ? (u32)(WS_W + par * W_PAR) + so + 3072u
                         : loc < 512u     ? (u32)(WS_FL + par * FL_PAR) + (((loc & 63u) * 8u + (u >> 3)) * 8u + (loc >> 6)) * 128u
                                          : (u32)(WS_FX + par * FX_PAR) + ((u >> 3) * 64u + loc - 512u) * 128u;
            u32x4 v[4];
            v[0] = __builtin_amdgcn_raw_buffer_load_b128(rw, lane * 16, so, SYS);
            v[1] = __builtin_amdgcn_raw_buffer_load_b128(rw, lane * 16 + 1024, so, SYS);
            const u32x2 w2 = __builtin_amdgcn_raw_buffer_load_b64(rw, f5 ? lane * 8 + 2048 : lane * 16 + 2048, so, SYS);
            const u32x2 w3 = __builtin_amdgcn_raw_buffer_load_b64(rw, f5 ? lane * 4 + 2560 : lane * 16 + 2056, so, SYS);
            const u32 g1 = __builtin_amdgcn_raw_buffer_load_b16(rw, lane * 2 + 2816, so, SYS);
            u32 e = __builtin_amdgcn_raw_buffer_load_b16(rall, lane * 2, lo, SYS);
            v[2] = (u32x4){w2.x, w2.y, w3.x, w3.y};
            if (raw) v[3] = __builtin_amdgcn_raw_buffer_load_b128(rw, lane * 16 + 3072, so, SYS);
            if (n >= NSLOT) {
                bool dead = false;
                while (!__all(lds_acquire(lds, L_CONS + (lane & 7) * 4) > n - NSLOT)) {
                    if (rfl(lds_acquire(lds, L_TAIL)) == DEAD) { dead = true; break; }
                    __builtin_amdgcn_s_sleep(1);
                }
                if (dead) break;
            }
            unsigned char* slot = lds + L_B + (n & (NSLOT - 1)) * 4096;
            const u32 basex = ((hdr >> 8) & 127u) * 0x01010101u;
            if (raw) {
#pragma unroll
                for (int q = 0; q < 4; ++q) *(u32x4*)(slot + q * 1024 + lane * 16) = v[q];
            } else if (f5) {
                const u32 P[10] = {v[0].x, v[0].y, v[0].z, v[0].w, v[1].x, v[1].y, v[1].z, v[1].w, v[2].x, v[2].y};
                u32 D[16];
#pragma unroll
                for (int h = 0; h < 2; ++h) {
                    const u32* Q = P + 5 * h;
#pragma unroll
                    for (int k = 0; k < 5; ++k) D[8 * h + k] = (Q[k] & 0x8f8f8f8fu) + basex;
                    D[8 * h + 5] = (((Q[0] >> 4) & 0x07070707u) | ((Q[3] >> 1) & 0x08080808u) | ((Q[4] << 3) & 0x80808080u)) + basex;
                    D[8 * h + 6] = (((Q[1] >> 4) & 0x07070707u) | ((Q[3] >> 2) & 0x08080808u) | ((Q[4] << 2) & 0x80808080u)) + basex;
                    D[8 * h + 7] = (((Q[2] >> 4) & 0x07070707u) | ((Q[3] >> 3) & 0x08080808u) | ((Q[4] << 1) & 0x80808080u)) + basex;
                }
#pragma unroll
                for (int q = 0; q < 4; ++q) *(u32x4*)(slot + q * 1024 + lane * 16) = (u32x4){D[4 * q], D[4 * q + 1], D[4 * q + 2], D[4 * q + 3]};
                const u32 g0 = w3.x;
                u32 sy[12], adv = 0;
#pragma unroll
                for (int j = 0; j < 12; ++j) {
                    sy[j] = ((j < 8 ? g0 : g1) >> (4 * (j & 7))) & 15u;
                    adv += min(sy[j] + 1u, 15u);
                }
                u32 cur = scan_add(adv) - adv - 1u;
#pragma unroll
                for (int j = 0; j < 12; ++j) {
                    cur += min(sy[j] + 1u, 15u);
                    __hip_atomic_fetch_sub(lds_word(slot + (cur & 0xffcu)), sy[j] != 15u ? 16u << ((cur & 3u) * 8u) : 0u,
                                           __ATOMIC_RELAXED, __HIP_MEMORY_SCOPE_WORKGROUP);
                }
            } else {
                const u32 P[4][3] = {{v[0].x, v[0].y, v[0].z}, {v[0].w, v[1].x, v[1].y}, {v[1].z, v[1].w, v[2].x}, {v[2].y, v[2].z, v[2].w}};
#pragma unroll
                for (int q = 0; q < 4; ++q) {
                    const u32 d3 = ((P[q][0] >> 5) & 0x03030303u) | ((P[q][1] >> 3) & 0x0c0c0c0cu) | ((P[q][2] >> 1) & 0x10101010u) | ((P[q][2] << 1) & 0x80808080u);
                    *(u32x4*)(slot + q * 1024 + lane * 16) =
                        (u32x4){(P[q][0] & 0x9f9f9f9fu) + basex, (P[q][1] & 0x9f9f9f9fu) + basex, (P[q][2] & 0x9f9f9f9fu) + basex, d3 + basex};
                }
            }
            if (!raw) {
                const u32 cnt = hdr & 255u;
                for (u32 e0 = 0; e0 < cnt; e0 += 64) {
                    if (e0) e = __builtin_amdgcn_raw_buffer_load_b16(rall, lane * 2 + e0 * 2, lo, SYS);
                    if (e0 + (u32)lane < cnt)
                        __hip_atomic_fetch_xor(lds_word(slot + (e & 0xffcu)), ((e >> 12) & 7u) << ((e & 3u) * 8u + 4u), __ATOMIC_RELAXED, __HIP_MEMORY_SCOPE_WORKGROUP);
                }
            }
            if (lane == 0) __hip_atomic_fetch_or(lds_word(lds + L_STG + (n >> 5) * 4), 1u << (n & 31u), __ATOMIC_RELEASE, __HIP_MEMORY_SCOPE_WORKGROUP);
        }
        return;
    }

    __syncthreads();
    const u32 call = rfl(*(const u32*)(lds + L_CALL));
    const u32 par = call & 1u;

    if (wave == W_POLL) {
        PRIO(2);
        const u64* fwp = (lane >> 3) == p.rank ? (const u64*)(my + WS_FW + par * FW_PAR) + jb * 8 + (lane & 7)
                                               : (const u64*)(my + WS_FL + par * FL_PAR + (size_t)(jb * 64 + lane) * 128 + 120);
        const __amdgpu_buffer_rsrc_t rf = make_rsrc(my + WS_FX + par * FX_PAR, (u32)FX_PAR);
        const u32 xf = (u32)(((lane >> 3) * 64 + ti * 32 + (lane & 7) * 4) * 128 + 120);
        u64 seen = 0, xok = 0;
        u32 tail = 0, spins = 0;
        const u64 t0 = __builtin_amdgcn_s_memrealtime();
        while (seen != ~0ull) {
            u64 f = 0;
            if (!((seen >> lane) & 1ull)) f = __hip_atomic_load(fwp, __ATOMIC_RELAXED, __HIP_MEMORY_SCOPE_SYSTEM);
            if (xok != ~0ull) {
                u32 gm = call;
                if (!((xok >> lane) & 1ull)) {
                    const u32 g0 = __builtin_amdgcn_raw_buffer_load_b32(rf, xf, 0, SYS), g1 = __builtin_amdgcn_raw_buffer_load_b32(rf, xf + 128, 0, SYS);
                    const u32 g2 = __builtin_amdgcn_raw_buffer_load_b32(rf, xf + 256, 0, SYS), g3 = __builtin_amdgcn_raw_buffer_load_b32(rf, xf + 384, 0, SYS);
                    gm = min(min(g0, g1), min(g2, g3));
                }
                u64 mx = __ballot(gm >= call);
                mx &= (mx >> 4) & 0x0f0f0f0f0f0f0f0full; mx &= mx >> 2; mx &= mx >> 1; mx &= 0x0101010101010101ull;
                xok = mx * 0xffull;
            }
            const u64 fresh = __ballot((u32)f >= call) & xok & ~seen;
            if (fresh) {
                if ((fresh >> lane) & 1) {
                    const u32 pos = tail + lanes_below(fresh);
                    *(u32*)(lds + L_ORD + pos * 4) = (u32)lane | ((u32)(f >> 32) << 6);
                }
                tail += (u32)__builtin_popcountll(fresh);
                seen |= fresh;
                if (lane == 0) lds_release(lds, L_TAIL, tail);
            } else {
                if ((++spins & 15u) == 0u && timed_out(tmo, call, t0)) {
                    VM0();
                    if (lane == 0) { lds_release(lds, L_BAD, 2u); lds_release(lds, L_TAIL, DEAD); }
                    return;
                }
                __builtin_amdgcn_s_sleep(4);
            }
            CFENCE();
        }
        const u32 tv = __hip_atomic_load(tmo, __ATOMIC_RELAXED, __HIP_MEMORY_SCOPE_AGENT);
        if (lane == 0) lds_release(lds, L_BAD, tv == call ? 2u : 1u);
        return;
    }

    const __amdgpu_buffer_rsrc_t rx = make_rsrc(my + WS_X + par * X_PAR, (u32)X_PAR);
    const float scale = p.sx[0] * p.sw[0];
    f32x16 acc0 = {}, acc1 = {};
    const u32 b_rd = (u32)(((lane >> 4) & 1) * 2048 + (lane >> 5) * 512 + (lane & 15) * 16);
    const u32 a_off = (u32)((ti * 8 + wave) * 8 * 2048 + lane * 16);
    u32 n = 0, t = 0, ord = 0;
    u64 staged = 0;
    auto snapshot = [&]() {
        u32x4 c;
        asm volatile("ds_read_b128 %0, %2\n\tds_read_b32 %1, %3\n\ts_waitcnt lgkmcnt(0)"
                     : "=&v"(c), "=&v"(ord) : "v"(lds0 + (u32)L_STG), "v"(lds0 + (u32)L_ORD + (u32)lane * 4u) : "memory");
        t = rfl(c.z);
        staged = ((u64)rfl(c.y) << 32) | rfl(c.x);
    };
    while (n < 64) {
        if (t <= n) {
            PRIO(0);
            while (true) { snapshot(); if (t > n) break; __builtin_amdgcn_s_sleep(1); }
            PRIO(1);
        }
        if (t == DEAD) break;
        const u32 m = min(t - n, 4u);
        u32x4 av[4][2];
#pragma unroll
        for (int j = 0; j < 4; ++j) {
            const u32 u = (u32)__builtin_amdgcn_readlane(ord, n + min((u32)j, m - 1u)) & 63u;
            const u32 so = (u >> 3) * (u32)(X_PAR / NR) + (u & 7) * 2048;
#pragma unroll
            for (int s = 0; s < 2; ++s) av[j][s] = __builtin_amdgcn_raw_buffer_load_b128(rx, a_off + s * 1024, so, SYS);
        }
        bool dead = false;
#pragma unroll
        for (int j = 0; j < 4; ++j) {
            if ((u32)j < m && !dead) {
                while (!((staged >> (n + j)) & 1ull)) {
                    snapshot();
                    if (t == DEAD) { dead = true; break; }
                    if (!((staged >> (n + j)) & 1ull)) __builtin_amdgcn_s_sleep(1);
                }
                if (!dead) {
                    const unsigned char* bb = lds + L_B + ((n + j) & (NSLOT - 1)) * 4096 + b_rd;
                    u32x4 b00 = *(const u32x4*)(bb), b01 = *(const u32x4*)(bb + 256);
                    u32x4 b10 = *(const u32x4*)(bb + 1024), b11 = *(const u32x4*)(bb + 1280);
                    asm volatile("" : "+v"(b00), "+v"(b01), "+v"(b10), "+v"(b11));
                    const i32x8 af = {(int)av[j][0].x, (int)av[j][0].y, (int)av[j][0].z, (int)av[j][0].w, (int)av[j][1].x, (int)av[j][1].y, (int)av[j][1].z, (int)av[j][1].w};
                    const i32x8 bf0 = {(int)b00.x, (int)b00.y, (int)b00.z, (int)b00.w, (int)b01.x, (int)b01.y, (int)b01.z, (int)b01.w};
                    const i32x8 bf1 = {(int)b10.x, (int)b10.y, (int)b10.z, (int)b10.w, (int)b11.x, (int)b11.y, (int)b11.z, (int)b11.w};
                    acc0 = __builtin_amdgcn_mfma_scale_f32_32x32x64_f8f6f4(af, bf0, acc0, 0  , 1  , 0, 0, 0, 0);
                    acc1 = __builtin_amdgcn_mfma_scale_f32_32x32x64_f8f6f4(af, bf1, acc1, 0, 1, 0, 0, 0, 0);
                }
            }
        }
        if (dead) break;
        n += m;
        if (lane == 0) lds_release(lds, L_CONS + wave * 4, n);
        snapshot();
    }

    u32 bad;
    while ((bad = rfl(lds_acquire(lds, L_BAD))) == 0u) __builtin_amdgcn_s_sleep(1);
    const float s = bad == 2u ? __builtin_nanf("") : scale;
    float* o = p.out + (size_t)(ti * 256 + wave * 32 + (lane >> 5) * 4 + ((lane >> 4) & 1)) * NN + jb * 64 + (lane & 15) * 4;
#pragma unroll
    for (int r = 0; r < 16; r += 2) {
        const auto s0 = __builtin_amdgcn_permlane16_swap(__float_as_uint(acc0[r] * s), __float_as_uint(acc0[r + 1] * s), false, false);
        const auto s1 = __builtin_amdgcn_permlane16_swap(__float_as_uint(acc1[r] * s), __float_as_uint(acc1[r + 1] * s), false, false);
        st16_sys(o + (size_t)((r & 3) + 8 * (r >> 2)) * NN, (u32x4){s0[0], s1[0], s0[1], s1[1]});
    }
}

}

extern "C" void kernel_launch(int rank, int nranks, ncclComm_t comm, hipStream_t stream,
                              void* const* d_in, const int* in_sizes, int n_in,
                              void* d_out, int out_size,
                              const int* in_dtypes, int out_dtype,
                              void* const* d_out_all,
                              void* const* d_ws_all, size_t ws_bytes) {
    (void)comm; (void)in_sizes; (void)n_in; (void)out_size; (void)in_dtypes; (void)out_dtype; (void)d_out_all;
    if (nranks != NR || ws_bytes < WS_END || (unsigned)rank >= (unsigned)NR) {
        fprintf(stderr, "unexpected configuration\n");
        return;
    }
    Params p;
    p.x = (const unsigned char*)d_in[0];
    p.w = (const unsigned char*)d_in[1];
    p.sx = (const float*)d_in[2];
    p.sw = (const float*)d_in[3];
    p.out = (float*)d_out;
    for (int i = 0; i < NR; ++i) p.ws[i] = (unsigned char*)d_ws_all[i];
    p.rank = rank;
    p.pad = 0;
    hipLaunchKernelGGL(gemm_ag_fused, dim3(NWG), dim3(NTHREADS), 0, stream, p);
    HIP_CHECK(hipGetLastError());
}
```

```cpp
#include <hip/hip_runtime.h>
#include <rccl/rccl.h>
#include <cstdio>

#define HIP_CHECK(expr) do { hipError_t _e = (expr); if (_e != hipSuccess) { \
    fprintf(stderr, "HIP error %s at %s:%d\n", hipGetErrorString(_e), __FILE__, __LINE__); \
    return; } } while (0)

namespace {

typedef unsigned u32;
typedef unsigned long long u64;
typedef u32 u32x2 __attribute__((ext_vector_type(2)));
typedef u32 u32x4 __attribute__((ext_vector_type(4)));
typedef int i32x8 __attribute__((ext_vector_type(8)));
typedef float f32x16 __attribute__((ext_vector_type(16)));

constexpr int NR = 8;
constexpr int NN = 4096;
constexpr int KS = 512;
constexpr int MS = 512;
constexpr int NWG = 256;
constexpr int NCWG = 128;
constexpr int NWORK = 8;
constexpr int NSTAGE = 7;
constexpr int W_STG  = NWORK;
constexpr int W_POLL = W_STG + NSTAGE;
constexpr int NTHREADS = 64 * (W_POLL + 1);
constexpr int NSEND = 5;
constexpr int WPD = (NWG - NCWG) * NSEND / (NR - 1);
constexpr u32 NXB = 64;
constexpr u32 NBLK = NXB + 512;
constexpr u32 F5_FROM = 32;
constexpr u32 F5_TO = 448;
constexpr u32 STAGGER_FROM = 32;
constexpr u32 STAGGER_NS = 80;

constexpr size_t WS_W   = 0;
constexpr size_t W_PAR  = (size_t)NR * 8 * 64 * 4096;
constexpr size_t WS_X   = 2 * W_PAR;
constexpr size_t X_PAR  = (size_t)NR * 16 * 8 * 2048;
constexpr size_t WS_FL  = WS_X + 2 * X_PAR;
constexpr size_t FL_PAR = 64 * 64 * 128;
constexpr size_t WS_FW  = WS_FL + 2 * FL_PAR;
constexpr size_t FW_PAR = 64 * 8 * 8;
constexpr size_t WS_FX  = WS_FW + 2 * FW_PAR;
constexpr size_t FX_PAR = 8 * 64 * 128;
constexpr size_t WS_Q   = WS_FX + 2 * FX_PAR;
constexpr size_t WS_CNT = WS_Q + NR * 128;
constexpr size_t WS_TMO = WS_CNT + NWG * 32;
constexpr size_t WS_END = WS_TMO + 128;

constexpr u64 DEADLINE_TICKS = 30000000ull;

constexpr u32 H_RAW = 127u << 8;
constexpr u32 H_F5 = 1u << 15;
constexpr u32 MAXFIX = 255;
constexpr u32 MAXRIDE = 60;
constexpr u32 LOC_OWN = 1023;

constexpr int NSLOT  = 32;
constexpr int L_B    = 0;
constexpr int L_ORD  = NSLOT * 4096;
constexpr int L_STG  = L_ORD + 256;
constexpr int L_TAIL = L_STG + 8;
constexpr int L_BAD  = L_STG + 12;
constexpr int L_CALL = L_STG + 16;
constexpr int L_CONS = L_CALL + 16;
constexpr int L_TAB  = L_CONS + 32;
constexpr int L_BYTES = L_TAB + 1024;
constexpr u32 DEAD = 0x80000000u;

struct Params {
    const unsigned char* x;
    const unsigned char* w;
    const float* sx;
    const float* sw;
    float* out;
    unsigned char* ws[NR];
    int rank;
    int pad;
};

__device__ __forceinline__ void st16_sys(void* p, u32x4 v) {
    asm volatile("global_store_dwordx4 %0, %1, off sc0 sc1\n\ts_nop 1" :: "v"(p), "v"(v) : "memory");
}
constexpr int SYS = 17;
__device__ __forceinline__ __amdgpu_buffer_rsrc_t make_rsrc(const void* base, u32 bytes) {
    return __builtin_amdgcn_make_buffer_rsrc(const_cast<void*>(base), 0, bytes, 0x00020000);
}
__device__ __forceinline__ u32 rfl(u32 v) { return __builtin_amdgcn_readfirstlane(v); }
__device__ __forceinline__ u32 lds_acquire(const unsigned char* lds, int off) {
    return __hip_atomic_load((const u32*)(lds + off), __ATOMIC_ACQUIRE, __HIP_MEMORY_SCOPE_WORKGROUP);
}
__device__ __forceinline__ void lds_release(unsigned char* lds, int off, u32 v) {
    __hip_atomic_store((u32*)(lds + off), v, __ATOMIC_RELEASE, __HIP_MEMORY_SCOPE_WORKGROUP);
}
__device__ __forceinline__ u32* lds_word(unsigned char* p) { return (u32*)__builtin_assume_aligned(p, 4); }
__device__ __forceinline__ u32 lanes_below(u64 m) {
    return __builtin_amdgcn_mbcnt_hi((u32)(m >> 32), __builtin_amdgcn_mbcnt_lo((u32)m, 0u));
}
__device__ __forceinline__ bool is_raw(u32 h) { return ((h >> 8) & 127u) == 127u; }
__device__ __forceinline__ u32 scan_add(u32 v) {
    v += __builtin_amdgcn_update_dpp(0u, v, 0x111, 0xf, 0xf, false);
    v += __builtin_amdgcn_update_dpp(0u, v, 0x112, 0xf, 0xf, false);
    v += __builtin_amdgcn_update_dpp(0u, v, 0x114, 0xf, 0xf, false);
    v += __builtin_amdgcn_update_dpp(0u, v, 0x118, 0xf, 0xf, false);
    v += __builtin_amdgcn_update_dpp(0u, v, 0x142, 0xa, 0xf, false);
    v += __builtin_amdgcn_update_dpp(0u, v, 0x143, 0xc, 0xf, false);
    return v;
}
__device__ __forceinline__ u32 scan_max(u32 v) {
    v = max(v, (u32)__builtin_amdgcn_update_dpp(0u, v, 0x111, 0xf, 0xf, false));
    v = max(v, (u32)__builtin_amdgcn_update_dpp(0u, v, 0x112, 0xf, 0xf, false));
    v = max(v, (u32)__builtin_amdgcn_update_dpp(0u, v, 0x114, 0xf, 0xf, false));
    v = max(v, (u32)__builtin_amdgcn_update_dpp(0u, v, 0x118, 0xf, 0xf, false));
    v = max(v, (u32)__builtin_amdgcn_update_dpp(0u, v, 0x142, 0xa, 0xf, false));
    v = max(v, (u32)__builtin_amdgcn_update_dpp(0u, v, 0x143, 0xc, 0xf, false));
    return v;
}
__device__ __forceinline__ u32 lane_before(u32 v) {
    u32 r = __builtin_amdgcn_update_dpp(0u, v, 0x138, 0xf, 0xf, false);
    asm volatile("" : "+v"(r));
    return r;
}
#define PRIO(x) __builtin_amdgcn_s_setprio(x)
#define VM0() asm volatile("s_waitcnt vmcnt(0)" ::: "memory")
#define CFENCE() asm volatile("" ::: "memory")

__device__ __forceinline__ bool timed_out(u32* tmo, u32 call, u64 t0) {
    if (__hip_atomic_load(tmo, __ATOMIC_RELAXED, __HIP_MEMORY_SCOPE_AGENT) == call
        || __builtin_amdgcn_s_memrealtime() - t0 > DEADLINE_TICKS) {
        __hip_atomic_store(tmo, call, __ATOMIC_RELAXED, __HIP_MEMORY_SCOPE_AGENT);
        return true;
    }
    return false;
}

__device__ __forceinline__ u32 build_list(const u32 (&F)[16], unsigned char* list, unsigned char* dump, int lane) {
    u32 M[2] = {0u, 0u};
#pragma unroll
    for (int i = 0; i < 16; ++i) {
        const u32 nz = (F[i] | (F[i] << 1) | (F[i] << 2)) & 0x40404040u;
        M[i >> 3] |= __builtin_amdgcn_udot4(nz >> 6, 0x08040201u, 0u, false) << (4 * (i & 7));
    }
#pragma unroll
    for (int q = 0; q < 4; ++q) *(u32x4*)(dump + q * 1024 + lane * 16) = (u32x4){F[4 * q], F[4 * q + 1], F[4 * q + 2], F[4 * q + 3]};
    const u32 cnt = (u32)__builtin_popcount(M[0]) + (u32)__builtin_popcount(M[1]);
    const u32 incl = scan_add(cnt);
    u32 idx = incl - cnt;
    u64 m = ((u64)M[1] << 32) | M[0];
    while (__ballot(m != 0ull)) {
        if (m != 0ull) {
            const u32 i = (u32)__builtin_ctzll(m);
            const u32 pos = (i >> 4) * 1024u + (u32)lane * 16u + (i & 15u);
            if (idx < MAXFIX) *(unsigned short*)(list + idx * 2) = (unsigned short)(pos | (((u32)dump[pos] >> 4) << 12));
            ++idx;
            m &= m - 1ull;
        }
    }
    return (u32)__builtin_amdgcn_readlane(incl, 63);
}

__device__ __forceinline__ u32 pack_block(const u32 (&T)[4][4], bool try5, unsigned char* scratch, unsigned char* dump, const u32* gaptab, int lane, u32x4 (&o)[4]) {
    u32 sum = 0;
#pragma unroll
    for (int i = 0; i < 16; ++i) sum = __builtin_amdgcn_sad_u8(T[i >> 2][i & 3] & 0x7f7f7f7fu, 0u, sum);
    const int bs = ((int)__builtin_amdgcn_readlane(scan_add(sum), 63) - 82944) >> 12;
    const u32 base = (u32)min(max(bs, 0), 96);
    if (try5) {
        const u32 b16 = base + 16u, b16x = b16 * 0x01010101u;
        u32 D[16], F[16], M[4] = {0u, 0u, 0u, 0u};
        if (lane < 32) *(u32x4*)(scratch + 2048 + lane * 16) = (u32x4){0u, 0u, 0u, 0u};
#pragma unroll
        for (int i = 0; i < 16; ++i) {
            const u32 d = T[i >> 2][i & 3];
            const u32 u = (d | 0x80808080u) - b16x;
            const u32 low4 = u & 0x0f0f0f0fu;
            D[i] = (d & 0x80808080u) | low4;
            const u32 t = u ^ 0x80808080u;
            const u32 lo = t & (t << 1) & (t << 2) & (t << 3) & 0x80808080u;
            M[i >> 2] |= __builtin_amdgcn_udot4(lo >> 7, 0x08040201u, 0u, false) << (4 * (i & 3));
            F[i] = (d ^ (low4 + b16x)) & 0x70707070u & ~((lo >> 1) | (lo >> 2) | (lo >> 3));
        }
        u32 smax[4], first[4], lead[4], cnt[4];
#pragma unroll
        for (int q = 0; q < 4; ++q) smax[q] = scan_max(M[q] ? (u32)(q * 1024) + (u32)lane * 16u + 32u - (u32)__builtin_clz(M[q]) : 0u);
        u32 curq = 0;
#pragma unroll
        for (int q = 0; q < 4; ++q) {
            const u32 prev = max(lane_before(smax[q]), curq);
            const u32 g = M[q] ? (u32)(q * 1024) + (u32)lane * 16u + (u32)__builtin_ctz(M[q]) - prev : 0u;
            lead[q] = __umul24(g, 4370u) >> 16;
            first[q] = 15u + 15u * lead[q] - g;
            cnt[q] = (u32)__builtin_popcount(M[q]);
            curq = max(curq, (u32)__builtin_amdgcn_readlane(smax[q], 63));
        }
        const u32 i01 = scan_add((lead[0] + cnt[0]) | ((lead[1] + cnt[1]) << 16)), i23 = scan_add((lead[2] + cnt[2]) | ((lead[3] + cnt[3]) << 16));
        const u32 t01 = (u32)__builtin_amdgcn_readlane(i01, 63), t23 = (u32)__builtin_amdgcn_readlane(i23, 63);
        const u32 incl[4] = {i01 & 0xffffu, (t01 & 0xffffu) + (i01 >> 16), (t01 & 0xffffu) + (t01 >> 16) + (i23 & 0xffffu),
                             (t01 & 0xffffu) + (t01 >> 16) + (t23 & 0xffffu) + (i23 >> 16)};
        const u32 nsym = (t01 & 0xffffu) + (t01 >> 16) + (t23 & 0xffffu) + (t23 >> 16);
        if (nsym <= 768u) {
#pragma unroll
            for (int q = 0; q < 4; ++q) {
                const u32 lo = M[q] & 255u, hi = M[q] >> 8, na = (u32)__builtin_popcount(lo);
                u64 X = (u64)first[q] | ((u64)gaptab[lo] << 4) | ((u64)gaptab[hi] << (4u * na + 4u));
                if (lo && hi) X |= (u64)(15u - (8u + (u32)__builtin_ctz(hi) - (32u - (u32)__builtin_clz(lo)))) << (4u * na);
                if (M[q]) {
                    const u32 at = incl[q] - cnt[q], sh = (at & 7u) * 4u, x0 = (u32)X, x1 = (u32)(X >> 32);
                    u32* G = lds_word(scratch + 2048 + (at >> 3) * 4);
                    __hip_atomic_fetch_or(G, x0 << sh, __ATOMIC_RELAXED, __HIP_MEMORY_SCOPE_WORKGROUP);
                    __hip_atomic_fetch_or(G + 1, sh ? __builtin_amdgcn_alignbit(x1, x0, 32u - sh) : x1, __ATOMIC_RELAXED, __HIP_MEMORY_SCOPE_WORKGROUP);
                    __hip_atomic_fetch_or(G + 2, sh ? x1 >> (32u - sh) : 0u, __ATOMIC_RELAXED, __HIP_MEMORY_SCOPE_WORKGROUP);
                }
            }
            const u32 total = build_list(F, scratch, dump, lane);
            if (total <= MAXFIX) {
                const unsigned short* gs = (const unsigned short*)(scratch + 2048 + lane * 6);
                const u32 g0 = ~((u32)gs[0] | ((u32)gs[1] << 16)), g1 = ~(u32)gs[2];
                u32 P[10];
#pragma unroll
                for (int h = 0; h < 2; ++h) {
                    const u32* E = D + 8 * h;
                    P[5 * h + 0] = E[0] | ((E[5] & 0x07070707u) << 4);
                    P[5 * h + 1] = E[1] | ((E[6] & 0x07070707u) << 4);
                    P[5 * h + 2] = E[2] | ((E[7] & 0x07070707u) << 4);
                    P[5 * h + 3] = E[3] | ((E[5] & 0x08080808u) << 1) | ((E[6] & 0x08080808u) << 2) | ((E[7] & 0x08080808u) << 3);
                    P[5 * h + 4] = E[4] | ((E[5] & 0x80808080u) >> 3) | ((E[6] & 0x80808080u) >> 2) | ((E[7] & 0x80808080u) >> 1);
                }
                *(u32x2*)(scratch + 2560 + lane * 8) = (u32x2){P[8], P[9]};
                *(u32*)(scratch + 2560 + 512 + lane * 4) = g0;
                *(unsigned short*)(scratch + 2560 + 768 + lane * 2) = (unsigned short)g1;
                o[0] = (u32x4){P[0], P[1], P[2], P[3]};
                o[1] = (u32x4){P[4], P[5], P[6], P[7]};
                o[2] = *(const u32x4*)(scratch + 2560 + lane * 16);
                o[3] = *(const u32x4*)(scratch + lane * 16);
                return total | (b16 << 8) | H_F5;
            }
        }
    }
    const u32 basex = base * 0x01010101u;
    u32 D[16], F[16];
#pragma unroll
    for (int i = 0; i < 16; ++i) {
        const u32 d = T[i >> 2][i & 3];
        const u32 code = ((d | 0x80808080u) - basex) & 0x1f1f1f1fu;
        D[i] = (d & 0x80808080u) | code;
        F[i] = (d ^ (code + basex)) & 0x60606060u;
    }
    const u32 total = build_list(F, scratch, dump, lane);
    if (total <= MAXFIX) {
        u32 P[4][3];
#pragma unroll
        for (int q = 0; q < 4; ++q) {
            P[q][0] = D[4 * q] | ((D[4 * q + 3] & 0x03030303u) << 5);
            P[q][1] = D[4 * q + 1] | ((D[4 * q + 3] & 0x0c0c0c0cu) << 3);
            P[q][2] = D[4 * q + 2] | ((D[4 * q + 3] & 0x10101010u) << 1) | ((D[4 * q + 3] & 0x80808080u) >> 1);
        }
        o[0] = (u32x4){P[0][0], P[0][1], P[0][2], P[1][0]};
        o[1] = (u32x4){P[1][1], P[1][2], P[2][0], P[2][1]};
        o[2] = (u32x4){P[2][2], P[3][0], P[3][1], P[3][2]};
        o[3] = *(const u32x4*)(scratch + lane * 16);
        return total | (base << 8);
    }
#pragma unroll
    for (int q = 0; q < 4; ++q) o[q] = (u32x4){T[q][0], T[q][1], T[q][2], T[q][3]};
    return H_RAW;
}

__global__ void __launch_bounds__(NTHREADS, NTHREADS / 256) gemm_ag_fused(Params p) {
    __shared__ __attribute__((aligned(16))) unsigned char lds[L_BYTES];
    const int tid = threadIdx.x;
    const int lane = tid & 63;
    const int wave = rfl(tid >> 6);
    const int b = blockIdx.x;
    const int xcd = b & 7, a = b >> 3;
    unsigned char* my = p.ws[p.rank];
    u32* tmo = (u32*)(my + WS_TMO);
    const u32 lds0 = (u32)(size_t)(__attribute__((address_space(3))) unsigned char*)lds;

    u32* cntp = (u32*)(my + WS_CNT) + b * 8 + (a >= NCWG / 8 ? wave & 7 : 0);
    if (a < NCWG / 8 && tid == 0) {
        const u32 c = __hip_atomic_load(cntp, __ATOMIC_RELAXED, __HIP_MEMORY_SCOPE_AGENT) + 1u;
        __hip_atomic_store(cntp, c, __ATOMIC_RELAXED, __HIP_MEMORY_SCOPE_AGENT);
        *(u32*)(lds + L_CALL) = c;
    }
    if (a < NCWG / 8 && wave == 1) {
        if (lane < 4) *(u32*)(lds + L_STG + lane * 4) = 0u;
        if (lane < 8) *(u32*)(lds + L_CONS + lane * 4) = 0u;
    }
    if (a < NCWG / 8 && tid >= 128 && tid < 384) {
        const u32 x = (u32)tid - 128u;
        u32 v = 0, k = 0;
        int prev = -1;
#pragma unroll
        for (int bit = 0; bit < 8; ++bit)
            if ((x >> bit) & 1u) {
                if (prev >= 0) { v |= (16u - (u32)(bit - prev)) << (4u * k); ++k; }
                prev = bit;
            }
        *(u32*)(lds + L_TAB + x * 4) = v;
    }
    const u32 xlane = (u32)((lane & 31) * KS + (lane >> 5) * 32);
    auto load_x = [&](__amdgpu_buffer_rsrc_t xr, u32 t, u32x4 (&r)[4]) {
        const u32 so = (t >> 2) * (32 * KS) + (t & 3) * 128;
#pragma unroll
        for (int j = 0; j < 4; ++j) r[j] = __builtin_amdgcn_raw_buffer_load_b128(xr, xlane + (j >> 1) * 64 + (j & 1) * 16, so, 0);
    };

    if (a >= NCWG / 8) {
        const int g = ((a - NCWG / 8) * 8 + xcd) * NSEND + wave;
        if (wave >= NSEND || g >= WPD * (NR - 1)) return;
        const u32 cnt_old = __hip_atomic_load(cntp, __ATOMIC_RELAXED, __HIP_MEMORY_SCOPE_AGENT);
        const int dd = 1 + g % (NR - 1);
        const u32 wi = (u32)(g / (NR - 1));
        const int dst = (p.rank + dd) & (NR - 1);
        unsigned char* rws = p.ws[dst];
        u32* queue = (u32*)(my + WS_Q + dd * 128);
        const __amdgpu_buffer_rsrc_t xr = make_rsrc(p.x + (size_t)dst * MS * KS, MS * KS);
        const u64 tstart = __builtin_amdgcn_s_memrealtime() + 130;
        u32 t0 = wi, t1 = wi + WPD, t2 = wi + 2 * WPD;
        u32x4 r[4];
        if (t0 < NXB) load_x(xr, t0, r);
        const u32 call = rfl(cnt_old) + 1u;
        if (lane == 0) __hip_atomic_store(cntp, call, __ATOMIC_RELAXED, __HIP_MEMORY_SCOPE_AGENT);
        const u32 par = call & 1u;
        const u32 qbase = (call - 1u) * NBLK - 3u * WPD;
        const __amdgpu_buffer_rsrc_t wl = make_rsrc(my + WS_W + par * W_PAR + (size_t)p.rank * (W_PAR / NR), (u32)(W_PAR / NR));
        const u64* fl = (const u64*)(my + WS_FW + par * FW_PAR);
        unsigned char* xd = rws + WS_X + par * X_PAR + (size_t)p.rank * (X_PAR / NR) + lane * 16;
        unsigned char* wd = rws + WS_W + par * W_PAR + (size_t)p.rank * (W_PAR / NR) + lane * 16;
        unsigned char* fline = rws + WS_FL + par * FL_PAR + (size_t)p.rank * 8 * 128 + (lane & 7) * 16;
        unsigned char* xline = rws + WS_FX + par * FX_PAR + (size_t)p.rank * 64 * 128 + (lane & 7) * 16;
        auto kc_of = [&](u32 t) { return (t - NXB) >> 6; };
        auto jb_of = [&](u32 t) { return (t - NXB) & 63u; };
        auto load_w = [&](u32 t) {
#pragma unroll
            for (int j = 0; j < 4; ++j) r[j] = __builtin_amdgcn_raw_buffer_load_b128(wl, lane * 16 + j * 1024, (t - NXB) * 4096, SYS);
        };
        auto wait_hdr = [&](u32 t, u32& h) -> bool {
            const u64* f = fl + jb_of(t) * 8 + kc_of(t);
            u64 v = __hip_atomic_load(f, __ATOMIC_RELAXED, __HIP_MEMORY_SCOPE_SYSTEM);
            if ((u32)v < call) {
                const u64 ts = __builtin_amdgcn_s_memrealtime();
                do {
                    if (timed_out(tmo, call, ts)) return false;
                    __builtin_amdgcn_s_sleep(2);
                    v = __hip_atomic_load(f, __ATOMIC_RELAXED, __HIP_MEMORY_SCOPE_SYSTEM);
                } while ((u32)v < call);
            }
            CFENCE();
            h = rfl((u32)(v >> 32)) & 0xffffu;
            return true;
        };
        u32 h0 = H_RAW, h1 = H_RAW, loc0 = LOC_OWN;
        u64 f1 = 0;
        if (t0 >= NXB) { if (!wait_hdr(t0, h0)) return; load_w(t0); }
        if (wi > STAGGER_FROM) {
            const u64 until = tstart + (u64)((wi - STAGGER_FROM) * STAGGER_NS) / 10u;
            while (__builtin_amdgcn_s_memrealtime() < until) __builtin_amdgcn_s_sleep(1);
        }
#pragma unroll 1
        while (t0 < NBLK) {
            u32 tk = 0;
            if (lane == 0) tk = __hip_atomic_fetch_add(queue, 1u, __ATOMIC_RELAXED, __HIP_MEMORY_SCOPE_AGENT);
            unsigned char* d = t0 < NXB ? xd + (size_t)t0 * 4096 : wd + (size_t)(t0 - NXB) * 4096;
            const u32 nl2 = (h0 & H_F5) ? 56u : 64u;
            const u32 nl3 = is_raw(h0) ? 64u : loc0 != LOC_OWN ? 0u : ((h0 & 255u) + 7u) >> 3;
            st16_sys(d, r[0]);
            st16_sys(d + 1024, r[1]);
            if ((u32)lane < nl2) st16_sys(d + 2 * 1024, r[2]);
            if ((u32)lane < nl3) st16_sys(d + 3 * 1024, r[3]);
            h1 = H_RAW;
            if (t1 < NXB) load_x(xr, t1, r);
            else if (t1 < NBLK) {
                if (rfl((u32)f1) >= call) h1 = rfl((u32)(f1 >> 32)) & 0xffffu;
                else if (!wait_hdr(t1, h1)) return;
                load_w(t1);
            }
            u64 f2 = 0;
            if (t2 >= NXB && t2 < NBLK) f2 = __hip_atomic_load(fl + jb_of(t2) * 8 + kc_of(t2), __ATOMIC_RELAXED, __HIP_MEMORY_SCOPE_SYSTEM);
            VM0();
            const bool ride = t1 >= NXB && t1 < NBLK && !is_raw(h1) && (h1 & 255u) <= MAXRIDE;
            const u32 loc1 = !ride ? LOC_OWN : t0 < NXB ? 512u + t0 : t0 - NXB;
            {
                u32x4 v = r[3];
                if (lane == 7) { v.z = call; v.w = h0 | (loc0 << 16); }
                unsigned char* line = t0 < NXB ? xline + (size_t)t0 * 128 : fline + (size_t)(jb_of(t0) * 64 + kc_of(t0)) * 128;
                if (lane == 7 || (ride && lane < 7)) st16_sys(line, v);
            }
            t0 = t1; h0 = h1; loc0 = loc1;
            t1 = t2; f1 = f2;
            t2 = rfl(tk) - qbase;
        }
        return;
    }

    const int jb = xcd * 8 + (a & 7), ti = a >> 3;

    if (wave >= W_STG && wave < W_POLL) {
        const int sg = wave - W_STG;
        PRIO(3);
        const u32 id = (u32)(sg * NCWG + a * 8 + xcd);
        u32 r[16];
        if (id < 512) {
            const __amdgpu_buffer_rsrc_t wr = make_rsrc(p.w, KS * NN);
            const u32 wlane = (u32)((lane >> 4) * 16 * NN + (lane & 15) * 4);
            const u32 so = (id >> 6) * 64 * NN + (id & 63u) * 64;
#pragma unroll
            for (int j = 0; j < 16; ++j) r[j] = __builtin_amdgcn_raw_buffer_load_b32(wr, wlane + j * NN, so, 0);
        }
        __syncthreads();
        const u32 call = rfl(*(const u32*)(lds + L_CALL));
        const u32 par = call & 1u;
        if (id < 512) {
            u32 T[4][4];
#pragma unroll
            for (int t4 = 0; t4 < 4; ++t4) {
                const u32 r0 = r[4 * t4], r1 = r[4 * t4 + 1], r2 = r[4 * t4 + 2], r3 = r[4 * t4 + 3];
                const u32 aa = __builtin_amdgcn_perm(r1, r0, 0x05010400u), bb = __builtin_amdgcn_perm(r1, r0, 0x07030602u);
                const u32 cc = __builtin_amdgcn_perm(r3, r2, 0x05010400u), ee = __builtin_amdgcn_perm(r3, r2, 0x07030602u);
                T[0][t4] = __builtin_amdgcn_perm(cc, aa, 0x05040100u);
                T[1][t4] = __builtin_amdgcn_perm(cc, aa, 0x07060302u);
                T[2][t4] = __builtin_amdgcn_perm(ee, bb, 0x05040100u);
                T[3][t4] = __builtin_amdgcn_perm(ee, bb, 0x07060302u);
            }
            u32x4 o[4];
            const u32 hdr = pack_block(T, id >= F5_FROM && id < F5_TO, lds + L_B + sg * 4096, lds + L_B + (sg + NSTAGE) * 4096, (const u32*)(lds + L_TAB), lane, o);
            unsigned char* d = my + WS_W + par * W_PAR + (size_t)p.rank * (W_PAR / NR) + (size_t)id * 4096 + lane * 16;
#pragma unroll
            for (int j = 0; j < 4; ++j) st16_sys(d + j * 1024, o[j]);
            VM0();
            if (lane == 0) __hip_atomic_store((u64*)(my + WS_FW + par * FW_PAR) + (id & 63u) * 8 + (id >> 6), (u64)call | ((u64)(hdr | (LOC_OWN << 16)) << 32),
                                              __ATOMIC_RELAXED, __HIP_MEMORY_SCOPE_SYSTEM);
        } else if (id - 512 < NXB) {
            const u32 t = id - 512;
            u32x4 o[4];
            load_x(make_rsrc(p.x + (size_t)p.rank * MS * KS, MS * KS), t, o);
            unsigned char* d = my + WS_X + par * X_PAR + (size_t)p.rank * (X_PAR / NR) + (size_t)t * 4096 + lane * 16;
#pragma unroll
            for (int j = 0; j < 4; ++j) st16_sys(d + j * 1024, o[j]);
            VM0();
            if (lane == 0) __hip_atomic_store((u32*)(my + WS_FX + par * FX_PAR + (size_t)(p.rank * 64 + t) * 128 + 120), call, __ATOMIC_RELAXED, __HIP_MEMORY_SCOPE_SYSTEM);
        }
        const __amdgpu_buffer_rsrc_t rw = make_rsrc(my + WS_W + par * W_PAR, (u32)W_PAR);
        const __amdgpu_buffer_rsrc_t rall = make_rsrc(my, (u32)WS_END);
        for (u32 n = (u32)sg; n < 64; n += NSTAGE) {
            PRIO(0);
            u32 t, ent;
            while (true) {
                u32 v0, v1;
                asm volatile("ds_read_b32 %0, %2\n\tds_read_b32 %1, %3\n\ts_waitcnt lgkmcnt(0)"
                             : "=&v"(v0), "=&v"(v1) : "v"(lds0 + (u32)L_TAIL), "v"(lds0 + (u32)L_ORD + n * 4u) : "memory");
                t = rfl(v0); ent = rfl(v1);
                if (t > n) break;
                __builtin_amdgcn_s_sleep(1);
            }
            PRIO(3);
            if (t == DEAD) break;
            const u32 u = ent & 63u, hdr = (ent >> 6) & 0xffffu, loc = ent >> 22;
            const u32 so = u * (64 * 4096) + (u32)jb * 4096;
            const bool raw = is_raw(hdr), f5 = (hdr & H_F5) != 0u;
            const u32 lo = loc == LOC_OWN ? (u32)(WS_W + par * W_PAR) + so + 3072u
                         : loc < 512u     ? (u32)(WS_FL + par * FL_PAR) + (((loc & 63u) * 8u + (u >> 3)) * 8u + (loc >> 6)) * 128u
                                          : (u32)(WS_FX + par * FX_PAR) + ((u >> 3) * 64u + loc - 512u) * 128u;
            u32x4 v[4];
            v[0] = __builtin_amdgcn_raw_buffer_load_b128(rw, lane * 16, so, SYS);
            v[1] = __builtin_amdgcn_raw_buffer_load_b128(rw, lane * 16 + 1024, so, SYS);
            const u32x2 w2 = __builtin_amdgcn_raw_buffer_load_b64(rw, f5 ? lane * 8 + 2048 : lane * 16 + 2048, so, SYS);
            const u32x2 w3 = __builtin_amdgcn_raw_buffer_load_b64(rw, f5 ? lane * 4 + 2560 : lane * 16 + 2056, so, SYS);
            const u32 g1 = __builtin_amdgcn_raw_buffer_load_b16(rw, lane * 2 + 2816, so, SYS);
            u32 e = __builtin_amdgcn_raw_buffer_load_b16(rall, lane * 2, lo, SYS);
            v[2] = (u32x4){w2.x, w2.y, w3.x, w3.y};
            if (raw) v[3] = __builtin_amdgcn_raw_buffer_load_b128(rw, lane * 16 + 3072, so, SYS);
            if (n >= NSLOT) {
                bool dead = false;
                while (!__all(lds_acquire(lds, L_CONS + (lane & 7) * 4) > n - NSLOT)) {
                    if (rfl(lds_acquire(lds, L_TAIL)) == DEAD) { dead = true; break; }
                    __builtin_amdgcn_s_sleep(1);
                }
                if (dead) break;
            }
            unsigned char* slot = lds + L_B + (n & (NSLOT - 1)) * 4096;
            const u32 basex = ((hdr >> 8) & 127u) * 0x01010101u;
            if (raw) {
#pragma unroll
                for (int q = 0; q < 4; ++q) *(u32x4*)(slot + q * 1024 + lane * 16) = v[q];
            } else if (f5) {
                const u32 P[10] = {v[0].x, v[0].y, v[0].z, v[0].w, v[1].x, v[1].y, v[1].z, v[1].w, v[2].x, v[2].y};
                u32 D[16];
#pragma unroll
                for (int h = 0; h < 2; ++h) {
                    const u32* Q = P + 5 * h;
#pragma unroll
                    for (int k = 0; k < 5; ++k) D[8 * h + k] = (Q[k] & 0x8f8f8f8fu) + basex;
                    D[8 * h + 5] = (((Q[0] >> 4) & 0x07070707u) | ((Q[3] >> 1) & 0x08080808u) | ((Q[4] << 3) & 0x80808080u)) + basex;
                    D[8 * h + 6] = (((Q[1] >> 4) & 0x07070707u) | ((Q[3] >> 2) & 0x08080808u) | ((Q[4] << 2) & 0x80808080u)) + basex;
                    D[8 * h + 7] = (((Q[2] >> 4) & 0x07070707u) | ((Q[3] >> 3) & 0x08080808u) | ((Q[4] << 1) & 0x80808080u)) + basex;
                }
#pragma unroll
                for (int q = 0; q < 4; ++q) *(u32x4*)(slot + q * 1024 + lane * 16) = (u32x4){D[4 * q], D[4 * q + 1], D[4 * q + 2], D[4 * q + 3]};
                const u32 g0 = w3.x;
                u32 sy[12], adv = 0;
#pragma unroll
                for (int j = 0; j < 12; ++j) {
                    sy[j] = ((j < 8 ? g0 : g1) >> (4 * (j & 7))) & 15u;
                    adv += min(sy[j] + 1u, 15u);
                }
                u32 cur = scan_add(adv) - adv - 1u;
#pragma unroll
                for (int j = 0; j < 12; ++j) {
                    cur += min(sy[j] + 1u, 15u);
                    __hip_atomic_fetch_sub(lds_word(slot + (cur & 0xffcu)), sy[j] != 15u ? 16u << ((cur & 3u) * 8u) : 0u,
                                           __ATOMIC_RELAXED, __HIP_MEMORY_SCOPE_WORKGROUP);
                }
            } else {
                const u32 P[4][3] = {{v[0].x, v[0].y, v[0].z}, {v[0].w, v[1].x, v[1].y}, {v[1].z, v[1].w, v[2].x}, {v[2].y, v[2].z, v[2].w}};
#pragma unroll
                for (int q = 0; q < 4; ++q) {
                    const u32 d3 = ((P[q][0] >> 5) & 0x03030303u) | ((P[q][1] >> 3) & 0x0c0c0c0cu) | ((P[q][2] >> 1) & 0x10101010u) | ((P[q][2] << 1) & 0x80808080u);
                    *(u32x4*)(slot + q * 1024 + lane * 16) =
                        (u32x4){(P[q][0] & 0x9f9f9f9fu) + basex, (P[q][1] & 0x9f9f9f9fu) + basex, (P[q][2] & 0x9f9f9f9fu) + basex, d3 + basex};
                }
            }
            if (!raw) {
                const u32 cnt = hdr & 255u;
                for (u32 e0 = 0; e0 < cnt; e0 += 64) {
                    if (e0) e = __builtin_amdgcn_raw_buffer_load_b16(rall, lane * 2 + e0 * 2, lo, SYS);
                    if (e0 + (u32)lane < cnt)
                        __hip_atomic_fetch_xor(lds_word(slot + (e & 0xffcu)), ((e >> 12) & 7u) << ((e & 3u) * 8u + 4u), __ATOMIC_RELAXED, __HIP_MEMORY_SCOPE_WORKGROUP);
                }
            }
            if (lane == 0) __hip_atomic_fetch_or(lds_word(lds + L_STG + (n >> 5) * 4), 1u << (n & 31u), __ATOMIC_RELEASE, __HIP_MEMORY_SCOPE_WORKGROUP);
        }
        return;
    }

    __syncthreads();
    const u32 call = rfl(*(const u32*)(lds + L_CALL));
    const u32 par = call & 1u;

    if (wave == W_POLL) {
        PRIO(2);
        const u64* fwp = (lane >> 3) == p.rank ? (const u64*)(my + WS_FW + par * FW_PAR) + jb * 8 + (lane & 7)
                                               : (const u64*)(my + WS_FL + par * FL_PAR + (size_t)(jb * 64 + lane) * 128 + 120);
        const __amdgpu_buffer_rsrc_t rf = make_rsrc(my + WS_FX + par * FX_PAR, (u32)FX_PAR);
        const u32 xf = (u32)(((lane >> 3) * 64 + ti * 32 + (lane & 7) * 4) * 128 + 120);
        u64 seen = 0, xok = 0;
        u32 tail = 0, spins = 0;
        const u64 t0 = __builtin_amdgcn_s_memrealtime();
        while (seen != ~0ull) {
            u64 f = 0;
            if (!((seen >> lane) & 1ull)) f = __hip_atomic_load(fwp, __ATOMIC_RELAXED, __HIP_MEMORY_SCOPE_SYSTEM);
            if (xok != ~0ull) {
                u32 gm = call;
                if (!((xok >> lane) & 1ull)) {
                    const u32 g0 = __builtin_amdgcn_raw_buffer_load_b32(rf, xf, 0, SYS), g1 = __builtin_amdgcn_raw_buffer_load_b32(rf, xf + 128, 0, SYS);
                    const u32 g2 = __builtin_amdgcn_raw_buffer_load_b32(rf, xf + 256, 0, SYS), g3 = __builtin_amdgcn_raw_buffer_load_b32(rf, xf + 384, 0, SYS);
                    gm = min(min(g0, g1), min(g2, g3));
                }
                u64 mx = __ballot(gm >= call);
                mx &= (mx >> 4) & 0x0f0f0f0f0f0f0f0full; mx &= mx >> 2; mx &= mx >> 1; mx &= 0x0101010101010101ull;
                xok = mx * 0xffull;
            }
            const u64 fresh = __ballot((u32)f >= call) & xok & ~seen;
            if (fresh) {
                if ((fresh >> lane) & 1) {
                    const u32 pos = tail + lanes_below(fresh);
                    *(u32*)(lds + L_ORD + pos * 4) = (u32)lane | ((u32)(f >> 32) << 6);
                }
                tail += (u32)__builtin_popcountll(fresh);
                seen |= fresh;
                if (lane == 0) lds_release(lds, L_TAIL, tail);
            } else {
                if ((++spins & 15u) == 0u && timed_out(tmo, call, t0)) {
                    VM0();
                    if (lane == 0) { lds_release(lds, L_BAD, 2u); lds_release(lds, L_TAIL, DEAD); }
                    return;
                }
                __builtin_amdgcn_s_sleep(4);
            }
            CFENCE();
        }
        const u32 tv = __hip_atomic_load(tmo, __ATOMIC_RELAXED, __HIP_MEMORY_SCOPE_AGENT);
        if (lane == 0) lds_release(lds, L_BAD, tv == call ? 2u : 1u);
        return;
    }

    const __amdgpu_buffer_rsrc_t rx = make_rsrc(my + WS_X + par * X_PAR, (u32)X_PAR);
    const float scale = p.sx[0] * p.sw[0];
    f32x16 acc0 = {}, acc1 = {};
    const u32 b_rd = (u32)(((lane >> 4) & 1) * 2048 + (lane >> 5) * 512 + (lane & 15) * 16);
    const u32 a_off = (u32)((ti * 8 + wave) * 8 * 2048 + lane * 16);
    u32 n = 0, t = 0, ord = 0;
    u64 staged = 0;
    auto snapshot = [&]() {
        u32x4 c;
        asm volatile("ds_read_b128 %0, %2\n\tds_read_b32 %1, %3\n\ts_waitcnt lgkmcnt(0)"
                     : "=&v"(c), "=&v"(ord) : "v"(lds0 + (u32)L_STG), "v"(lds0 + (u32)L_ORD + (u32)lane * 4u) : "memory");
        t = rfl(c.z);
        staged = ((u64)rfl(c.y) << 32) | rfl(c.x);
    };
    while (n < 64) {
        if (t <= n) {
            PRIO(0);
            while (true) { snapshot(); if (t > n) break; __builtin_amdgcn_s_sleep(1); }
            PRIO(1);
        }
        if (t == DEAD) break;
        const u32 m = min(t - n, 4u);
        u32x4 av[4][2];
#pragma unroll
        for (int j = 0; j < 4; ++j) {
            const u32 u = (u32)__builtin_amdgcn_readlane(ord, n + min((u32)j, m - 1u)) & 63u;
            const u32 so = (u >> 3) * (u32)(X_PAR / NR) + (u & 7) * 2048;
#pragma unroll
            for (int s = 0; s < 2; ++s) av[j][s] = __builtin_amdgcn_raw_buffer_load_b128(rx, a_off + s * 1024, so, SYS);
        }
        bool dead = false;
#pragma unroll
        for (int j = 0; j < 4; ++j) {
            if ((u32)j < m && !dead) {
                while (!((staged >> (n + j)) & 1ull)) {
                    snapshot();
                    if (t == DEAD) { dead = true; break; }
                    if (!((staged >> (n + j)) & 1ull)) __builtin_amdgcn_s_sleep(1);
                }
                if (!dead) {
                    const unsigned char* bb = lds + L_B + ((n + j) & (NSLOT - 1)) * 4096 + b_rd;
                    u32x4 b00 = *(const u32x4*)(bb), b01 = *(const u32x4*)(bb + 256);
                    u32x4 b10 = *(const u32x4*)(bb + 1024), b11 = *(const u32x4*)(bb + 1280);
                    asm volatile("" : "+v"(b00), "+v"(b01), "+v"(b10), "+v"(b11));
                    const i32x8 af = {(int)av[j][0].x, (int)av[j][0].y, (int)av[j][0].z, (int)av[j][0].w, (int)av[j][1].x, (int)av[j][1].y, (int)av[j][1].z, (int)av[j][1].w};
                    const i32x8 bf0 = {(int)b00.x, (int)b00.y, (int)b00.z, (int)b00.w, (int)b01.x, (int)b01.y, (int)b01.z, (int)b01.w};
                    const i32x8 bf1 = {(int)b10.x, (int)b10.y, (int)b10.z, (int)b10.w, (int)b11.x, (int)b11.y, (int)b11.z, (int)b11.w};
                    acc0 = __builtin_amdgcn_mfma_scale_f32_32x32x64_f8f6f4(af, bf0, acc0, 0  , 1  , 0, 0, 0, 0);
                    acc1 = __builtin_amdgcn_mfma_scale_f32_32x32x64_f8f6f4(af, bf1, acc1, 0, 1, 0, 0, 0, 0);
                }
            }
        }
        if (dead) break;
        n += m;
        if (lane == 0) lds_release(lds, L_CONS + wave * 4, n);
        snapshot();
    }

    u32 bad;
    while ((bad = rfl(lds_acquire(lds, L_BAD))) == 0u) __builtin_amdgcn_s_sleep(1);
    const float s = bad == 2u ? __builtin_nanf("") : scale;
    float* o = p.out + (size_t)(ti * 256 + wave * 32 + (lane >> 5) * 4 + ((lane >> 4) & 1)) * NN + jb * 64 + (lane & 15) * 4;
#pragma unroll
    for (int r = 0; r < 16; r += 2) {
        const auto s0 = __builtin_amdgcn_permlane16_swap(__float_as_uint(acc0[r] * s), __float_as_uint(acc0[r + 1] * s), false, false);
        const auto s1 = __builtin_amdgcn_permlane16_swap(__float_as_uint(acc1[r] * s), __float_as_uint(acc1[r + 1] * s), false, false);
        st16_sys(o + (size_t)((r & 3) + 8 * (r >> 2)) * NN, (u32x4){s0[0], s1[0], s0[1], s1[1]});
    }
}

}

extern "C" void kernel_launch(int rank, int nranks, ncclComm_t comm, hipStream_t stream,
                              void* const* d_in, const int* in_sizes, int n_in,
                              void* d_out, int out_size,
                              const int* in_dtypes, int out_dtype,
                              void* const* d_out_all,
                              void* const* d_ws_all, size_t ws_bytes) {
    (void)comm; (void)in_sizes; (void)n_in; (void)out_size; (void)in_dtypes; (void)out_dtype; (void)d_out_all;
    if (nranks != NR || ws_bytes < WS_END || (unsigned)rank >= (unsigned)NR) {
        fprintf(stderr, "unexpected configuration\n");
        return;
    }
    Params p;
    p.x = (const unsigned char*)d_in[0];
    p.w = (const unsigned char*)d_in[1];
    p.sx = (const float*)d_in[2];
    p.sw = (const float*)d_in[3];
    p.out = (float*)d_out;
    for (int i = 0; i < NR; ++i) p.ws[i] = (unsigned char*)d_ws_all[i];
    p.rank = rank;
    p.pad = 0;
    hipLaunchKernelGGL(gemm_ag_fused, dim3(NWG), dim3(NTHREADS), 0, stream, p);
    HIP_CHECK(hipGetLastError());
}
```
